```python
import numpy as np

kernel_name = "gpu_gemm_rs_m4096_k4096_n16384_e4m3e5m2_dequant"


ABSMAX_RTOL = 0.05

COMPUTE_DTYPE = "float32"
EPILOGUE = "dequant"
IN_QUANT = None
OUT_QUANT = None
IN_FP8 = "e4m3e5m2"
IN_DTYPES = {"x": "e4m3", "w_mat": "e5m2",
             "scale_x": "float32", "scale_w": "float32"}
WORLD_SIZE = 8

def _snap_fp8(x, mant_bits, max_abs, min_normal):
    a = np.abs(x.astype(np.float64))
    _m, _e = np.frexp(a)
    q = np.where(a >= min_normal, np.ldexp(1.0, _e - 1 - mant_bits),
                 min_normal * 2.0 ** -mant_bits)
    snapped = np.minimum(np.round(a / q) * q, max_abs)
    return (np.sign(x) * snapped).astype(np.float32)

def setup_inputs():

    x = _snap_fp8(np.random.randn(4096, 4096).astype(np.float32), 3, 448.0, 2.0 ** -6)
    w_mat = _snap_fp8(np.random.randn(4096, 16384).astype(np.float32), 2, 57344.0, 2.0 ** -14)
    scale_x = (0.02 * (1.0 + np.random.rand(1))).astype(np.float32)
    scale_w = (0.02 * (1.0 + np.random.rand(1))).astype(np.float32)
    return {"x": x, "w_mat": w_mat, "scale_x": scale_x, "scale_w": scale_w}

def reference(x, w_mat, scale_x, scale_w):
    acc = (x @ w_mat).astype(np.float32)
    y = (acc * (scale_x * scale_w)).astype(np.float32)
    return y
```

```cpp
#include <hip/hip_runtime.h>
#include <rccl/rccl.h>
#include <cstdio>
#include <cstdint>

#define HIP_CHECK(expr) do { hipError_t _e = (expr); if (_e != hipSuccess) { \
    fprintf(stderr, "HIP error %s at %s:%d\n", hipGetErrorString(_e), __FILE__, __LINE__); \
    return; } } while (0)

typedef int v8i __attribute__((ext_vector_type(8)));
typedef int v4i __attribute__((ext_vector_type(4)));
typedef float v16f __attribute__((ext_vector_type(16)));
typedef float v4f __attribute__((ext_vector_type(4)));
#define GAS __attribute__((address_space(1)))
typedef GAS char gchar;
typedef GAS unsigned gu32;
typedef GAS v4i gv4i;
typedef GAS v4f gv4f;
typedef GAS float gf32;

namespace {

constexpr int NR = 8;
constexpr int GM = 4096, GK = 4096, GN = 16384;
constexpr int KS = GK / NR;
constexpr int NS = GN / NR;
constexpr int MS = GM / NR;
constexpr int BM = 256, BN = 128, BK = 128;
constexpr int TM = GM / BM;
constexpr int TN = NS / BN;
constexpr int NKB = GK / BK;
constexpr int KBS = KS / BK;
constexpr int RPO = BM / NR;
constexpr int NWG = TM * TN;
constexpr int NPUSH = 32;
constexpr int NTHREADS = 512;

constexpr size_t XIMG = (size_t)BM * BK;
constexpr size_t WIMG = (size_t)BN * BK;
constexpr size_t X_STAGE = (size_t)TM * NKB * XIMG;
constexpr size_t W_STAGE = (size_t)TN * NKB * WIMG;
constexpr size_t CTRL_BYTES = 1u << 20;
constexpr size_t STAGE_BYTES = X_STAGE + W_STAGE;
constexpr size_t WS_NEED = CTRL_BYTES + 2 * STAGE_BYTES;

constexpr int L_TICKET = 0;
constexpr int L_TMO = 1;
constexpr int L_WCNT = 8;
constexpr int L_OCNT = 16;
constexpr int L_XCNT = 32;
constexpr int XGROUPS = 4;
constexpr int WG_PER_GROUP = NPUSH / XGROUPS;
constexpr unsigned long long DEADLINE_TICKS = 100000000ull;

constexpr int STAGE_LDS = (int)(XIMG + WIMG);
constexpr int LDS_BYTES = 3 * STAGE_LDS;

__device__ __forceinline__ gchar* gptr(void* p) { return (gchar*)p; }
__device__ __forceinline__ gu32* ctrl_word(void* ws, int line) {
    return (gu32*)(gptr(ws) + (size_t)line * 128);
}

#define WAIT_VM(n) asm volatile("s_waitcnt vmcnt(" #n ")" ::: "memory")

__device__ __forceinline__ bool wave_wait(gu32* addr, unsigned target, bool active,
                                          gu32* tmo, unsigned call) {
    const unsigned long long t0 = __builtin_amdgcn_s_memrealtime();
    for (;;) {
        bool ok = true;
        if (active)
            ok = __hip_atomic_load(addr, __ATOMIC_RELAXED, __HIP_MEMORY_SCOPE_SYSTEM) >= target;
        if (__all(ok)) return true;
        if (__hip_atomic_load(tmo, __ATOMIC_RELAXED, __HIP_MEMORY_SCOPE_AGENT) == call ||
            __builtin_amdgcn_s_memrealtime() - t0 > DEADLINE_TICKS) {
            __hip_atomic_store(tmo, call, __ATOMIC_RELAXED, __HIP_MEMORY_SCOPE_AGENT);
            return false;
        }
        __builtin_amdgcn_s_sleep(2);
    }
}

__global__ __launch_bounds__(NTHREADS, 2)
void fused_gemm_kernel(const unsigned char* __restrict__ xs,
                       const unsigned char* __restrict__ wsrc,
                       const float* __restrict__ sx, const float* __restrict__ sw,
                       void* const* __restrict__ out_all, void* const* __restrict__ ws_all,
                       int rank) {
    __shared__ __attribute__((aligned(16))) char smem[LDS_BYTES];
    const int tid = threadIdx.x;
    const int lane = tid & 63;
    const int wave = __builtin_amdgcn_readfirstlane(tid >> 6);
    void* ws_me = ws_all[rank];
    gu32* tmo = ctrl_word(ws_me, L_TMO);

    unsigned* s_bcast = reinterpret_cast<unsigned*>(smem);
    if (tid == 0) {
        unsigned t = __hip_atomic_fetch_add(ctrl_word(ws_me, L_TICKET), 1u, __ATOMIC_RELAXED,
                                            __HIP_MEMORY_SCOPE_AGENT);
        s_bcast[0] = t;
    }
    __syncthreads();
    const unsigned ticket = __builtin_amdgcn_readfirstlane(s_bcast[0]);
    __syncthreads();
    const unsigned call = ticket / NWG + 1u;
    const int d = (int)(ticket % NWG);
    const size_t stage_off = CTRL_BYTES + (size_t)(call & 1u) * STAGE_BYTES;

    int tile;
    if (d < NPUSH) {
        const int grp = d / WG_PER_GROUP;
        const int gwv = (d % WG_PER_GROUP) * 8 + wave;
        const int ro = gwv >> 1;
        const int kk0 = (gwv & 1) * 2;
        v4i xv[4][2];
        {
            const int q = ro * 8 + (lane >> 3);
            const int grow0 = (q >> 5) * MS + (q & 31);
#pragma unroll
            for (int st = 0; st < 4; ++st) {
                const int tm = grp + XGROUPS * st;
                const unsigned char* src = xs + (size_t)(grow0 + RPO * tm) * KS + kk0 * BK + (lane & 7) * 16;
                xv[st][0] = *reinterpret_cast<const v4i*>(src);
                xv[st][1] = *reinterpret_cast<const v4i*>(src + BK);
            }
        }
        const int gw = d * 8 + wave;
#pragma unroll 1
        for (int ui = 0; ui < 2; ++ui) {
            const int u = gw + ui * 256;
            const int p = (rank + 1 + (u & 7)) & 7;
            const int rest = u >> 3;
            const int tn = rest >> 2, kk = rest & 3;
            const int kb16 = lane & 7, nb16 = lane >> 3;
            const unsigned char* src = wsrc + (size_t)(kk * BK + kb16 * 16) * GN + p * NS + tn * BN + nb16 * 16;
            v4i in[16];
#pragma unroll
            for (int j = 0; j < 16; ++j) in[j] = *reinterpret_cast<const v4i*>(src + (size_t)j * GN);
            v4i o[16];
#pragma unroll
            for (int q = 0; q < 4; ++q) {
#pragma unroll
                for (int c = 0; c < 4; ++c) {
                    const unsigned a0 = in[4 * q + 0][c], a1 = in[4 * q + 1][c];
                    const unsigned a2 = in[4 * q + 2][c], a3 = in[4 * q + 3][c];
                    const unsigned t0 = __builtin_amdgcn_perm(a1, a0, 0x05010400u);
                    const unsigned t1 = __builtin_amdgcn_perm(a1, a0, 0x07030602u);
                    const unsigned t2 = __builtin_amdgcn_perm(a3, a2, 0x05010400u);
                    const unsigned t3 = __builtin_amdgcn_perm(a3, a2, 0x07030602u);
                    o[4 * c + 0][q] = (int)__builtin_amdgcn_perm(t2, t0, 0x05040100u);
                    o[4 * c + 1][q] = (int)__builtin_amdgcn_perm(t2, t0, 0x07060302u);
                    o[4 * c + 2][q] = (int)__builtin_amdgcn_perm(t3, t1, 0x05040100u);
                    o[4 * c + 3][q] = (int)__builtin_amdgcn_perm(t3, t1, 0x07060302u);
                }
            }
            gchar* dst = gptr(ws_all[p]) + stage_off + X_STAGE +
                         (size_t)(tn * NKB + rank * KBS + kk) * WIMG + lane * 16;
#pragma unroll
            for (int i = 0; i < 16; ++i) *(gv4i*)(dst + i * 1024) = o[i];
        }
        WAIT_VM(0);
        __syncthreads();
        if (tid == 0) {
            __builtin_amdgcn_fence(__ATOMIC_RELEASE, "");
            WAIT_VM(0);
#pragma unroll 1
            for (int i = 0; i < NR; ++i) {
                const int p = (rank + 1 + i) & 7;
                __hip_atomic_fetch_add(ctrl_word(ws_all[p], L_WCNT + rank), 1u, __ATOMIC_RELAXED,
                                       __HIP_MEMORY_SCOPE_SYSTEM);
            }
        }
#pragma unroll
        for (int st = 0; st < 4; ++st) {
            const int tm = grp + XGROUPS * st;
            const size_t off = stage_off + (size_t)(tm * NKB + rank * KBS + kk0) * XIMG + ro * 1024 + lane * 16;
#pragma unroll
            for (int i = 0; i < NR; ++i) {
                const int p = (rank + 1 + ((i + wave) & 7)) & 7;
                gchar* dst = gptr(ws_all[p]) + off;
                *(gv4i*)(dst) = xv[st][0];
                *(gv4i*)(dst + XIMG) = xv[st][1];
            }
            if (st > 0) {
                WAIT_VM(16);
                __builtin_amdgcn_s_barrier();
                if (tid == 0) {
                    __builtin_amdgcn_fence(__ATOMIC_RELEASE, "");
                    WAIT_VM(0);
                    const int tmp = tm - XGROUPS;
#pragma unroll 1
                    for (int i = 0; i < NR; ++i) {
                        const int p = (rank + 1 + i) & 7;
                        __hip_atomic_fetch_add(ctrl_word(ws_all[p], L_XCNT + tmp * 8 + rank), 1u,
                                               __ATOMIC_RELAXED, __HIP_MEMORY_SCOPE_SYSTEM);
                    }
                }
            }
        }
        WAIT_VM(0);
        __syncthreads();
        if (tid == 0) {
            __builtin_amdgcn_fence(__ATOMIC_RELEASE, "");
            WAIT_VM(0);
            const int tmp = grp + XGROUPS * 3;
#pragma unroll 1
            for (int i = 0; i < NR; ++i) {
                const int p = (rank + 1 + i) & 7;
                __hip_atomic_fetch_add(ctrl_word(ws_all[p], L_XCNT + tmp * 8 + rank), 1u,
                                       __ATOMIC_RELAXED, __HIP_MEMORY_SCOPE_SYSTEM);
            }
        }
        tile = (NWG - NPUSH) + d;
    } else {
        tile = d - NPUSH;
    }
    const int tm = tile / TN, tn = tile % TN;

    if (wave == 0) {
        const int s = lane & 7;
        gu32* addr = lane < 8 ? ctrl_word(ws_me, L_WCNT + s) : ctrl_word(ws_me, L_XCNT + tm * 8 + s);
        const unsigned target = (lane < 8 ? (unsigned)NPUSH : (unsigned)WG_PER_GROUP) * call;
        wave_wait(addr, target, lane < 16, tmo, call);
        __builtin_amdgcn_fence(__ATOMIC_ACQUIRE, "");
        const unsigned bad = __hip_atomic_load(tmo, __ATOMIC_RELAXED, __HIP_MEMORY_SCOPE_AGENT) == call;
        WAIT_VM(0);
        if (lane == 0) s_bcast[0] = bad;
    }
    __syncthreads();
    const bool dead = __builtin_amdgcn_readfirstlane(s_bcast[0]) != 0;
    __syncthreads();

    const int wm = wave >> 1, wn = wave & 1;
    v16f acc[2][2];
#pragma unroll
    for (int a = 0; a < 2; ++a)
#pragma unroll
        for (int b = 0; b < 2; ++b)
#pragma unroll
            for (int i = 0; i < 16; ++i) acc[a][b][i] = 0.f;

    if (!dead) {
        const gchar* xg = gptr(ws_me) + stage_off + (size_t)tm * NKB * XIMG;
        const gchar* wg = gptr(ws_me) + stage_off + X_STAGE + (size_t)tn * NKB * WIMG;
        const int swz = ((tid & 7) ^ ((tid >> 4) & 7)) * 16;
        const int xoff = (tid >> 3) * 128 + swz;
        int woff[2];
#pragma unroll
        for (int i = 0; i < 2; ++i) {
            const int n = i * 64 + (tid >> 3);
            woff[i] = (((n & 15) * 8) + (n >> 4)) * 128 + swz;
        }
        auto stage = [&](int kb, int buf) {
            const gchar* xa = xg + (size_t)kb * XIMG + xoff;
            const gchar* wa = wg + (size_t)kb * WIMG;
            char* la = smem + buf * STAGE_LDS + tid * 16;
#pragma unroll
            for (int i = 0; i < 4; ++i)
                __builtin_amdgcn_global_load_lds((const gu32*)(xa + i * 8192),
                                                 (__attribute__((address_space(3))) unsigned*)(la + i * 8192), 16, 0, 0);
#pragma unroll
            for (int i = 0; i < 2; ++i)
                __builtin_amdgcn_global_load_lds((const gu32*)(wa + woff[i]),
                                                 (__attribute__((address_space(3))) unsigned*)(la + (int)XIMG + i * 8192), 16, 0, 0);
        };
        const int r32 = lane & 31, h = lane >> 5;
        const int sw = (r32 >> 1) & 7;
        int foff[2][2];
#pragma unroll
        for (int ks = 0; ks < 2; ++ks)
#pragma unroll
            for (int e = 0; e < 2; ++e) foff[ks][e] = r32 * 128 + (((ks * 4 + h * 2 + e) ^ sw) << 4);

        stage(0, 0);
        stage(1, 1);
        int buf = 0;
#pragma unroll 1
        for (int kb = 0; kb < NKB; ++kb) {
            if (kb + 1 < NKB) { WAIT_VM(6); } else { WAIT_VM(0); }
            __builtin_amdgcn_s_barrier();
            if (kb + 2 < NKB) stage(kb + 2, buf == 0 ? 2 : buf - 1);
            const char* la = smem + buf * STAGE_LDS + (wm * 64) * 128;
            const char* lb = smem + buf * STAGE_LDS + (int)XIMG + (wn * 64) * 128;
#pragma unroll
            for (int ks = 0; ks < 2; ++ks) {
                v8i af[2], bf[2];
#pragma unroll
                for (int i = 0; i < 2; ++i) {
                    const v4i a0 = *reinterpret_cast<const v4i*>(la + i * 4096 + foff[ks][0]);
                    const v4i a1 = *reinterpret_cast<const v4i*>(la + i * 4096 + foff[ks][1]);
                    const v4i b0 = *reinterpret_cast<const v4i*>(lb + i * 4096 + foff[ks][0]);
                    const v4i b1 = *reinterpret_cast<const v4i*>(lb + i * 4096 + foff[ks][1]);
                    af[i] = __builtin_shufflevector(a0, a1, 0, 1, 2, 3, 4, 5, 6, 7);
                    bf[i] = __builtin_shufflevector(b0, b1, 0, 1, 2, 3, 4, 5, 6, 7);
                }
#pragma unroll
                for (int a = 0; a < 2; ++a)
#pragma unroll
                    for (int b = 0; b < 2; ++b)
                        acc[a][b] = __builtin_amdgcn_mfma_scale_f32_32x32x64_f8f6f4(
                            af[a], bf[b], acc[a][b], 0  , 1  , 0, 127, 0, 127);
            }
            buf = buf == 2 ? 0 : buf + 1;
        }
    }
    __syncthreads();

    const float scale = sx[0] * sw[0];
    {
        float* ct = reinterpret_cast<float*>(smem);
        const int r32 = lane & 31, h = lane >> 5;
        const float nanv = __builtin_nanf("");
#pragma unroll
        for (int a = 0; a < 2; ++a)
#pragma unroll
            for (int b = 0; b < 2; ++b)
#pragma unroll
                for (int i = 0; i < 16; ++i) {
                    const int row = wm * 64 + a * 32 + (i & 3) + 8 * (i >> 2) + 4 * h;
                    const int col = wn * 64 + b * 32 + r32;
                    ct[row * BN + col] = dead ? nanv : acc[a][b][i] * scale;
                }
    }
    __syncthreads();
    {
        const int owner = wave;
        const float* ct = reinterpret_cast<const float*>(smem);
        const int r32 = lane & 31, h = lane >> 5;
        gf32* dst = (gf32*)out_all[owner] + (size_t)(RPO * tm + h) * GN + rank * NS + tn * BN + r32 * 4;
        const float* src = ct + (owner * RPO + h) * BN + r32 * 4;
#pragma unroll
        for (int i = 0; i < 16; ++i) {
            const v4f v = *reinterpret_cast<const v4f*>(src + i * 2 * BN);
            *(gv4f*)(dst + (size_t)i * 2 * GN) = v;
        }
    }
    WAIT_VM(0);
    __syncthreads();
    if (tid == 0) {
        __builtin_amdgcn_fence(__ATOMIC_RELEASE, "");
        WAIT_VM(0);
#pragma unroll 1
        for (int i = 0; i < NR - 1; ++i) {
            const int p = (rank + 1 + i) & 7;
            __hip_atomic_fetch_add(ctrl_word(ws_all[p], L_OCNT + rank), 1u, __ATOMIC_RELAXED,
                                   __HIP_MEMORY_SCOPE_SYSTEM);
        }
    }
    if (tile == NWG - 1 && wave == 0) {
        const int s = lane & 7;
        const bool ok = wave_wait(ctrl_word(ws_me, L_OCNT + s), (unsigned)NWG * call,
                                  lane < 8 && s != rank, tmo, call);
        __builtin_amdgcn_fence(__ATOMIC_ACQUIRE, "");
        if (!ok && lane == 0) {
            gf32* o = (gf32*)out_all[rank] + (size_t)(RPO * tm) * GN + rank * NS + tn * BN;
            *o = __builtin_nanf("");
        }
    }
}

}

extern "C" void kernel_launch(int rank, int nranks, ncclComm_t comm,
                              hipStream_t stream,
                              void* const* d_in, const int* in_sizes, int n_in,
                              void* d_out, int out_size,
                              const int* in_dtypes, int out_dtype,
                              void* const* d_out_all,
                              void* const* d_ws_all, size_t ws_bytes) {
    (void)comm; (void)in_dtypes; (void)out_dtype; (void)d_out;
    if (nranks != NR || (unsigned)rank >= (unsigned)NR || n_in < 4 || ws_bytes < WS_NEED ||
        in_sizes[0] != GM * KS || in_sizes[1] != KS * GN || out_size != MS * GN) {
        fprintf(stderr, "kernel_launch: unexpected problem geometry\n");
        return;
    }
    hipLaunchKernelGGL(fused_gemm_kernel, dim3(NWG), dim3(NTHREADS), 0, stream,
                       (const unsigned char*)d_in[0], (const unsigned char*)d_in[1],
                       (const float*)d_in[2], (const float*)d_in[3], d_out_all, d_ws_all, rank);
    HIP_CHECK(hipGetLastError());
}
```

```cpp
#include <hip/hip_runtime.h>
#include <rccl/rccl.h>
#include <cstdio>
#include <cstdint>
#include <type_traits>

#define HIP_CHECK(expr) do { hipError_t _e = (expr); if (_e != hipSuccess) { \
    fprintf(stderr, "HIP error %s at %s:%d\n", hipGetErrorString(_e), __FILE__, __LINE__); \
    return; } } while (0)

typedef int v8i __attribute__((ext_vector_type(8)));
typedef int v4i __attribute__((ext_vector_type(4)));
typedef float v16f __attribute__((ext_vector_type(16)));
typedef float v4f __attribute__((ext_vector_type(4)));
#define GAS __attribute__((address_space(1)))
typedef GAS char gchar;
typedef GAS unsigned gu32;
typedef GAS v4i gv4i;
typedef GAS v4f gv4f;
typedef GAS float gf32;

namespace {

constexpr int NR = 8;
constexpr int GM = 4096, GK = 4096, GN = 16384;
constexpr int KS = GK / NR;
constexpr int NS = GN / NR;
constexpr int MS = GM / NR;
constexpr int BM = 256, BN = 128, BK = 128;
constexpr int TM = GM / BM;
constexpr int TN = NS / BN;
constexpr int NKB = GK / BK;
constexpr int KBS = KS / BK;
constexpr int RPO = BM / NR;
constexpr int NWG = TM * TN;
constexpr int NPUSH = 32;
constexpr int NTHREADS = 512;

constexpr size_t XIMG = (size_t)BM * BK;
constexpr size_t WIMG = (size_t)BN * BK;
constexpr size_t X_STAGE = (size_t)TM * NKB * XIMG;
constexpr size_t W_STAGE = (size_t)TN * NKB * WIMG;
constexpr size_t CTRL_BYTES = 1u << 20;
constexpr size_t STAGE_BYTES = X_STAGE + W_STAGE;
constexpr size_t WS_NEED = CTRL_BYTES + 2 * STAGE_BYTES;

constexpr int L_TICKET = 0;
constexpr int L_TMO = 1;
constexpr int L_PDONE = 2;
constexpr int L_WCNT = 8;
constexpr int L_OCNT = 16;
constexpr int L_XCNT = 32;
constexpr int XGROUPS = 4;
constexpr int WG_PER_GROUP = NPUSH / XGROUPS;
constexpr int NWU = TN * KBS;
static_assert(NWG - NPUSH >= NWU, "w units");
constexpr unsigned long long DEADLINE_TICKS = 100000000ull;

constexpr int STAGE_LDS = (int)(XIMG + WIMG);
constexpr int LDS_BYTES = 3 * STAGE_LDS;

__device__ __forceinline__ gchar* gptr(void* p) { return (gchar*)p; }
__device__ __forceinline__ gu32* ctrl_word(void* ws, int line) {
    return (gu32*)(gptr(ws) + (size_t)line * 128);
}

#define WAIT_VM_(n) asm volatile("s_waitcnt vmcnt(" #n ")" ::: "memory")
#define WAIT_VM(n) WAIT_VM_(n)
#define XLEAD 8
#define THR_N 1
#define THR_K 1
#define TS(e) do { } while (0)

__device__ __forceinline__ void st16_wt(gchar* p, v4i v) {
    asm volatile("global_store_dwordx4 %0, %1, off sc0 sc1\n\ts_nop 1" :: "v"(p), "v"(v) : "memory");
}

__device__ __forceinline__ bool wave_wait(gu32* addr, unsigned target, bool active,
                                          gu32* tmo, unsigned call) {
    const unsigned long long t0 = __builtin_amdgcn_s_memrealtime();
    for (;;) {
        bool ok = true;
        if (active)
            ok = __hip_atomic_load(addr, __ATOMIC_RELAXED, __HIP_MEMORY_SCOPE_SYSTEM) >= target;
        if (__all(ok)) return true;
        if (__hip_atomic_load(tmo, __ATOMIC_RELAXED, __HIP_MEMORY_SCOPE_AGENT) == call ||
            __builtin_amdgcn_s_memrealtime() - t0 > DEADLINE_TICKS) {
            __hip_atomic_store(tmo, call, __ATOMIC_RELAXED, __HIP_MEMORY_SCOPE_AGENT);
            return false;
        }
        __builtin_amdgcn_s_sleep(2);
    }
}

__global__ __launch_bounds__(NTHREADS, 2)
void fused_gemm_kernel(const unsigned char* __restrict__ xs,
                       const unsigned char* __restrict__ wsrc,
                       const float* __restrict__ sx, const float* __restrict__ sw,
                       void* const* __restrict__ out_all, void* const* __restrict__ ws_all,
                       int rank) {
    __shared__ __attribute__((aligned(16))) char smem[LDS_BYTES];
    const int tid = threadIdx.x;
    const int lane = tid & 63;
    const int wave = __builtin_amdgcn_readfirstlane(tid >> 6);
    void* ws_me = ws_all[rank];
    gu32* tmo = ctrl_word(ws_me, L_TMO);

    const int d = (int)blockIdx.x;
    const bool xpusher = d < NPUSH;
    const int wu = d - NPUSH;
    const bool wpusher = !xpusher && wu < NWU;

    const int grp = d / WG_PER_GROUP;
    const int gwv = (d % WG_PER_GROUP) * 8 + wave;
    const int ro = gwv >> 1;
    const int kk0 = (gwv & 1) * 2;
    const int wp = (rank + 1 + wave) & 7;
    const int wtn = wu >> 2, wkk = wu & 3;
    v4i in[16];
    if (xpusher) {
        const int q = ro * 8 + (lane >> 3);
        const int grow0 = (q >> 5) * MS + (q & 31);
#pragma unroll
        for (int st = 0; st < 4; ++st) {
            const int tm = grp + XGROUPS * st;
            const unsigned char* src = xs + (size_t)(grow0 + RPO * tm) * KS + kk0 * BK + (lane & 7) * 16;
            in[2 * st + 0] = *reinterpret_cast<const v4i*>(src);
            in[2 * st + 1] = *reinterpret_cast<const v4i*>(src + BK);
        }
    } else if (wpusher) {
        const unsigned char* src = wsrc + (size_t)(wkk * BK + (lane & 7) * 16) * GN + wp * NS + wtn * BN + (lane >> 3) * 16;
#pragma unroll
        for (int j = 0; j < 16; ++j) in[j] = *reinterpret_cast<const v4i*>(src + (size_t)j * GN);
    }

    unsigned* s_bcast = reinterpret_cast<unsigned*>(smem + LDS_BYTES - 64);
    if (tid == 0) {
        unsigned t = __hip_atomic_fetch_add(ctrl_word(ws_me, L_TICKET), 1u, __ATOMIC_RELAXED,
                                            __HIP_MEMORY_SCOPE_AGENT);
        s_bcast[0] = t;
    }
    __syncthreads();
    const unsigned ticket = __builtin_amdgcn_readfirstlane(s_bcast[0]);
    __syncthreads();
    const unsigned call = ticket / NWG + 1u;
    const size_t stage_off = CTRL_BYTES + (size_t)(call & 1u) * STAGE_BYTES;
    TS(0);

    if (xpusher) {
        gu32* sig = ctrl_word(ws_all[(rank + 1 + lane) & 7], L_XCNT + grp * 8 + rank);
#pragma unroll
        for (int st = 0; st < 4; ++st) {
            const int tm = grp + XGROUPS * st;
            const size_t off = stage_off + (size_t)(tm * NKB + rank * KBS + kk0) * XIMG + ro * 1024 + lane * 16;
            if (st == 1) {
                WAIT_VM(0);
                __builtin_amdgcn_s_barrier();
                if (tid < NR) __hip_atomic_fetch_add(sig, 1u, __ATOMIC_RELAXED, __HIP_MEMORY_SCOPE_SYSTEM);
                TS(3);
            } else if (st > 1) {
                WAIT_VM(XLEAD);
            }
#pragma unroll
            for (int i = 0; i < NR; ++i) {
                const int p = (rank + 1 + ((i + wave) & 7)) & 7;
                gchar* dst = gptr(ws_all[p]) + off;
                st16_wt(dst, in[2 * st + 0]);
                st16_wt(dst + XIMG, in[2 * st + 1]);
            }
            if (st > 1) {
                WAIT_VM(16);
                __builtin_amdgcn_s_barrier();
                if (tid < NR)
                    __hip_atomic_fetch_add(sig + (st - 1) * XGROUPS * 8 * 32, 1u, __ATOMIC_RELAXED,
                                           __HIP_MEMORY_SCOPE_SYSTEM);
                TS(2 + st);
            }
        }
        WAIT_VM(0);
        __builtin_amdgcn_s_barrier();
        if (tid < NR)
            __hip_atomic_fetch_add(sig + 3 * XGROUPS * 8 * 32, 1u, __ATOMIC_RELAXED, __HIP_MEMORY_SCOPE_SYSTEM);
        if (tid == 0)
            __hip_atomic_fetch_add(ctrl_word(ws_me, L_PDONE), 1u, __ATOMIC_RELAXED, __HIP_MEMORY_SCOPE_AGENT);
        TS(6);
    } else if (wpusher) {
        v4i o[16];
#pragma unroll
        for (int q = 0; q < 4; ++q) {
#pragma unroll
            for (int c = 0; c < 4; ++c) {
                const unsigned a0 = in[4 * q + 0][c], a1 = in[4 * q + 1][c];
                const unsigned a2 = in[4 * q + 2][c], a3 = in[4 * q + 3][c];
                const unsigned t0 = __builtin_amdgcn_perm(a1, a0, 0x05010400u);
                const unsigned t1 = __builtin_amdgcn_perm(a1, a0, 0x07030602u);
                const unsigned t2 = __builtin_amdgcn_perm(a3, a2, 0x05010400u);
                const unsigned t3 = __builtin_amdgcn_perm(a3, a2, 0x07030602u);
                o[4 * c + 0][q] = (int)__builtin_amdgcn_perm(t2, t0, 0x05040100u);
                o[4 * c + 1][q] = (int)__builtin_amdgcn_perm(t2, t0, 0x07060302u);
                o[4 * c + 2][q] = (int)__builtin_amdgcn_perm(t3, t1, 0x05040100u);
                o[4 * c + 3][q] = (int)__builtin_amdgcn_perm(t3, t1, 0x07060302u);
            }
        }
#pragma unroll
        for (int i = 0; i < 16; ++i)
            *reinterpret_cast<v4i*>(smem + wave * (int)WIMG + i * 1024 + lane * 16) = o[i];
        __syncthreads();
        const size_t doff = stage_off + X_STAGE + (size_t)(wtn * NKB + rank * KBS + wkk) * WIMG +
                            wave * 2048 + lane * 16;
#pragma unroll
        for (int i = 0; i < 16; ++i) {
            const int e = (wave + i) & 7;
            const v4i v = *reinterpret_cast<const v4i*>(smem + e * (int)WIMG + wave * 2048 + (i >> 3) * 1024 + lane * 16);
            st16_wt(gptr(ws_all[(rank + 1 + e) & 7]) + doff + (i >> 3) * 1024, v);
        }
        WAIT_VM(0);
        __builtin_amdgcn_s_barrier();
        if (tid < NR)
            __hip_atomic_fetch_add(ctrl_word(ws_all[(rank + 1 + tid) & 7], L_WCNT + rank), 1u,
                                   __ATOMIC_RELAXED, __HIP_MEMORY_SCOPE_SYSTEM);
        TS(1);
    }
    int tm, tn;
    {
        const int xcd = d & 7, j = d >> 3;
        const int idx = j < NPUSH / 8 ? 32 - NPUSH / 8 + j : j - NPUSH / 8;
        tm = 4 * (idx >> 3) + 2 * (xcd & 1) + ((idx >> 2) & 1);
        tn = 4 * (xcd >> 1) + (idx & 3);
    }

    if (wave == 0) {
        const int s = lane & 7;
        gu32* addr = lane < 8 ? ctrl_word(ws_me, L_WCNT + s) : ctrl_word(ws_me, L_XCNT + tm * 8 + s);
        const unsigned target = (lane < 8 ? (unsigned)NWU : (unsigned)WG_PER_GROUP) * call;
        wave_wait(addr, target, lane < 16, tmo, call);
        __builtin_amdgcn_fence(__ATOMIC_ACQUIRE, "");
        const unsigned bad = __hip_atomic_load(tmo, __ATOMIC_RELAXED, __HIP_MEMORY_SCOPE_AGENT) == call;
        WAIT_VM(0);
        if (lane == 0) s_bcast[0] = bad;
    }
    __syncthreads();
    const bool dead = __builtin_amdgcn_readfirstlane(s_bcast[0]) != 0;
    __syncthreads();
    TS(7);

    const int wm = wave >> 1, wn = wave & 1;
    v16f acc[2][2];
#pragma unroll
    for (int a = 0; a < 2; ++a)
#pragma unroll
        for (int b = 0; b < 2; ++b)
#pragma unroll
            for (int i = 0; i < 16; ++i) acc[a][b][i] = 0.f;

    if (!dead) {
        const gchar* xg = gptr(ws_me) + stage_off + (size_t)tm * NKB * XIMG;
        const gchar* wg = gptr(ws_me) + stage_off + X_STAGE + (size_t)tn * NKB * WIMG;
        const int swz = ((tid & 7) ^ ((tid >> 4) & 7)) * 16;
        const int xoff = (tid >> 3) * 128 + swz;
        int woff[2];
#pragma unroll
        for (int i = 0; i < 2; ++i) {
            const int n = i * 64 + (tid >> 3);
            woff[i] = (((n & 15) * 8) + (n >> 4)) * 128 + swz;
        }
        auto piece = [&](int kb, int buf, int i) {
            char* la = smem + buf * STAGE_LDS + tid * 16;
            if (i < 4)
                __builtin_amdgcn_global_load_lds((const gu32*)(xg + (size_t)kb * XIMG + xoff + i * 8192),
                                                 (__attribute__((address_space(3))) unsigned*)(la + i * 8192), 16, 0, 0);
            else
                __builtin_amdgcn_global_load_lds((const gu32*)(wg + (size_t)kb * WIMG + woff[i - 4]),
                                                 (__attribute__((address_space(3))) unsigned*)(la + (int)XIMG + (i - 4) * 8192), 16, 0, 0);
        };
        const int r32 = lane & 31, h = lane >> 5;
        const int sw = (r32 >> 1) & 7;
        int foff[2][2];
#pragma unroll
        for (int ks = 0; ks < 2; ++ks)
#pragma unroll
            for (int e = 0; e < 2; ++e) foff[ks][e] = r32 * 128 + (((ks * 4 + h * 2 + e) ^ sw) << 4);
        auto frags = [&](int buf, int ks, v8i (&af)[2], v8i (&bf)[2]) {
            const char* la = smem + buf * STAGE_LDS + (wm * 64) * 128;
            const char* lb = smem + buf * STAGE_LDS + (int)XIMG + (wn * 64) * 128;
#pragma unroll
            for (int i = 0; i < 2; ++i) {
                const v4i a0 = *reinterpret_cast<const v4i*>(la + i * 4096 + foff[ks][0]);
                const v4i a1 = *reinterpret_cast<const v4i*>(la + i * 4096 + foff[ks][1]);
                const v4i b0 = *reinterpret_cast<const v4i*>(lb + i * 4096 + foff[ks][0]);
                const v4i b1 = *reinterpret_cast<const v4i*>(lb + i * 4096 + foff[ks][1]);
                af[i] = __builtin_shufflevector(a0, a1, 0, 1, 2, 3, 4, 5, 6, 7);
                bf[i] = __builtin_shufflevector(b0, b1, 0, 1, 2, 3, 4, 5, 6, 7);
            }
        };
#define MFMA(A, B, C) C = __builtin_amdgcn_mfma_scale_f32_32x32x64_f8f6f4(A, B, C, 0, 1, 0, 127, 0, 127)
#define SB() __builtin_amdgcn_sched_barrier(0)
#pragma unroll
        for (int b3 = 0; b3 < 3; ++b3)
#pragma unroll
            for (int i = 0; i < 6; ++i) piece(b3, b3, i);
        v8i a0[2], b0[2], a1[2], b1[2];
        WAIT_VM(12);
        __builtin_amdgcn_s_barrier();
        frags(0, 0, a0, b0);
        int buf = 0;
        auto body = [&](int kb, auto tail_c) {
            constexpr int TAIL = decltype(tail_c)::value;
            const int nbuf = buf == 2 ? 0 : buf + 1;
            MFMA(a0[0], b0[0], acc[0][0]);
            SB();
            frags(buf, 1, a1, b1);
            SB();
            MFMA(a0[0], b0[1], acc[0][1]);
            MFMA(a0[1], b0[0], acc[1][0]);
            MFMA(a0[1], b0[1], acc[1][1]);
            SB();
            if (TAIL < 3) {
                asm volatile("s_waitcnt lgkmcnt(0)" ::: "memory");
                if (TAIL < 2) { WAIT_VM(6); } else { WAIT_VM(0); }
                __builtin_amdgcn_s_barrier();
            }
            SB();
            MFMA(a1[0], b1[0], acc[0][0]);
            SB();
            if (TAIL < 3) frags(nbuf, 0, a0, b0);
            SB();
            MFMA(a1[0], b1[1], acc[0][1]);
            SB();
            if (TAIL == 0) { piece(kb + 3, buf, 0); piece(kb + 3, buf, 1); piece(kb + 3, buf, 2); }
            SB();
            MFMA(a1[1], b1[0], acc[1][0]);
            SB();
            if (TAIL == 0) { piece(kb + 3, buf, 3); piece(kb + 3, buf, 4); piece(kb + 3, buf, 5); }
            SB();
            MFMA(a1[1], b1[1], acc[1][1]);
            SB();
            buf = nbuf;
        };
#pragma unroll 1
        for (int kb = 0; kb < NKB - 3; ++kb) body(kb, std::integral_constant<int, 0>{});
        body(NKB - 3, std::integral_constant<int, 1>{});
        body(NKB - 2, std::integral_constant<int, 2>{});
        body(NKB - 1, std::integral_constant<int, 3>{});
    }
    __syncthreads();
    TS(8);

    const float scale = sx[0] * sw[0];
    {
        float* ct = reinterpret_cast<float*>(smem);
        const int r32 = lane & 31, h = lane >> 5;
        const float nanv = __builtin_nanf("");
#pragma unroll
        for (int a = 0; a < 2; ++a)
#pragma unroll
            for (int b = 0; b < 2; ++b)
#pragma unroll
                for (int i = 0; i < 16; ++i) {
                    const int row = wm * 64 + a * 32 + (i & 3) + 8 * (i >> 2) + 4 * h;
                    const int col = wn * 64 + b * 32 + r32;
                    ct[row * BN + col] = dead ? nanv : acc[a][b][i] * scale;
                }
    }
    __syncthreads();
    TS(12);
    {
        const float* ct = reinterpret_cast<const float*>(smem);
        const int r32 = lane & 31, h = lane >> 5;
        const size_t doff = (size_t)(RPO * tm + 4 * wave + h) * GN + rank * NS + tn * BN + r32 * 4;
        const float* src = ct + (4 * wave + h) * BN + r32 * 4;
#pragma unroll
        for (int i = 0; i < 16; ++i) {
            const int owner = (rank + 1 + wave + i) & 7;
            const v4f v = *reinterpret_cast<const v4f*>(src + (owner * RPO + (i >> 3) * 2) * BN);
            st16_wt((gchar*)((gf32*)out_all[owner] + doff + (size_t)(i >> 3) * 2 * GN), __builtin_bit_cast(v4i, v));
            if ((i % THR_N) == THR_N - 1) WAIT_VM(THR_K);
        }
    }
    WAIT_VM(0);
    __syncthreads();
    TS(9);
    if (tid == 0) {
#pragma unroll 1
        for (int i = 0; i < NR - 1; ++i) {
            const int p = (rank + 1 + i) & 7;
            __hip_atomic_fetch_add(ctrl_word(ws_all[p], L_OCNT + rank), 1u, __ATOMIC_RELAXED,
                                   __HIP_MEMORY_SCOPE_SYSTEM);
        }
    }
    TS(10);
    if (d == NPUSH - 1 && wave == 0) {
        const int s = lane & 7;
        const bool ok = wave_wait(ctrl_word(ws_me, L_OCNT + s), (unsigned)NWG * call,
                                  lane < 8 && s != rank, tmo, call);
        __builtin_amdgcn_fence(__ATOMIC_ACQUIRE, "");
        TS(11);
        if (!ok && lane == 0) {
            gf32* o = (gf32*)out_all[rank] + (size_t)(RPO * tm) * GN + rank * NS + tn * BN;
            *o = __builtin_nanf("");
        }
    }
}

}

extern "C" void kernel_launch(int rank, int nranks, ncclComm_t comm,
                              hipStream_t stream,
                              void* const* d_in, const int* in_sizes, int n_in,
                              void* d_out, int out_size,
                              const int* in_dtypes, int out_dtype,
                              void* const* d_out_all,
                              void* const* d_ws_all, size_t ws_bytes) {
    (void)comm; (void)in_dtypes; (void)out_dtype; (void)d_out;
    if (nranks != NR || (unsigned)rank >= (unsigned)NR || n_in < 4 || ws_bytes < WS_NEED ||
        in_sizes[0] != GM * KS || in_sizes[1] != KS * GN || out_size != MS * GN) {
        fprintf(stderr, "kernel_launch: unexpected problem geometry\n");
        return;
    }
    hipLaunchKernelGGL(fused_gemm_kernel, dim3(NWG), dim3(NTHREADS), 0, stream,
                       (const unsigned char*)d_in[0], (const unsigned char*)d_in[1],
                       (const float*)d_in[2], (const float*)d_in[3], d_out_all, d_ws_all, rank);
    HIP_CHECK(hipGetLastError());
}
```

```cpp
#include <hip/hip_runtime.h>
#include <rccl/rccl.h>
#include <cstdio>
#include <cstdint>
#include <type_traits>

#define HIP_CHECK(expr) do { hipError_t _e = (expr); if (_e != hipSuccess) { \
    fprintf(stderr, "HIP error %s at %s:%d\n", hipGetErrorString(_e), __FILE__, __LINE__); \
    return; } } while (0)

typedef int v8i __attribute__((ext_vector_type(8)));
typedef int v4i __attribute__((ext_vector_type(4)));
typedef float v16f __attribute__((ext_vector_type(16)));
typedef float v4f __attribute__((ext_vector_type(4)));
#define GAS __attribute__((address_space(1)))
typedef GAS char gchar;
typedef GAS unsigned gu32;
typedef GAS v4i gv4i;
typedef GAS v4f gv4f;
typedef GAS float gf32;

namespace {

constexpr int NR = 8;
constexpr int GM = 4096, GK = 4096, GN = 16384;
constexpr int KS = GK / NR;
constexpr int NS = GN / NR;
constexpr int MS = GM / NR;
constexpr int BM = 256, BN = 128, BK = 128;
constexpr int TM = GM / BM;
constexpr int TN = NS / BN;
constexpr int NKB = GK / BK;
constexpr int KBS = KS / BK;
constexpr int RPO = BM / NR;
constexpr int NWG = TM * TN;
constexpr int NPUSH = 16;
constexpr int NWP = 32;
constexpr int NTHREADS = 512;

constexpr size_t XIMG = (size_t)BM * BK;
constexpr size_t WIMG = (size_t)BN * BK;
constexpr size_t CTRL_BYTES = 1u << 20;
constexpr size_t SLOT = 4u << 20;
constexpr size_t STAGE_BYTES = NR * SLOT;
constexpr size_t WS_NEED = CTRL_BYTES + 2 * STAGE_BYTES;
constexpr size_t ITEM = 512u << 10;
__device__ __forceinline__ size_t x_off(int tm) {
    return (size_t)((tm >> 2) ? (tm >> 2) + 2 : 0) * ITEM + (size_t)(tm & 3) * KBS * XIMG;
}
__device__ __forceinline__ size_t w_off(int tn) {
    return (size_t)(1 + ((tn >> 1) & 1)) * ITEM + (size_t)((tn >> 2) * 2 + (tn & 1)) * KBS * WIMG;
}

constexpr int L_DONE = 0;
constexpr int L_TMO = 1;
constexpr int L_WISS = 2;
constexpr int L_STICK = 288;
constexpr int L_SREL = 296;
constexpr int L_OCNT = 16;
constexpr int L_XCNT = 32;
constexpr int L_WCNT = 160;
constexpr int WG_PER_XTILE = NPUSH / 4;
static_assert(TN * KBS == 2 * NWP && TM == 16, "two w units per pusher");
constexpr unsigned long long DEADLINE_TICKS = 100000000ull;

constexpr int STAGE_LDS = (int)(XIMG + WIMG);
constexpr int LDS_BYTES = 3 * STAGE_LDS;

__device__ __forceinline__ gchar* gptr(void* p) { return (gchar*)p; }
__device__ __forceinline__ gu32* ctrl_word(void* ws, int line) {
    return (gu32*)(gptr(ws) + (size_t)line * 128);
}

#define WAIT_VM_(n) asm volatile("s_waitcnt vmcnt(" #n ")" ::: "memory")
#define WAIT_VM(n) WAIT_VM_(n)
#define WGATE 24
#define NSLOT 2
#define KLEAD 4
#define THR_N 16
#define THR_K 16
#define TS(e) do { } while (0)

__device__ __forceinline__ void st16_wt(gchar* p, v4i v) {
    asm volatile("global_store_dwordx4 %0, %1, off sc0 sc1\n\ts_nop 1" :: "v"(p), "v"(v) : "memory");
}

__device__ __forceinline__ v4i ld16(const unsigned char* p) {
    v4i v;
    asm volatile("global_load_dwordx4 %0, %1, off" : "=v"(v) : "v"(p) : "memory");
    return v;
}

__device__ __forceinline__ bool wave_wait(gu32* addr, unsigned target, bool active,
                                          gu32* tmo, unsigned call) {
    const unsigned long long t0 = __builtin_amdgcn_s_memrealtime();
    for (;;) {
        bool ok = true;
        if (active)
            ok = __hip_atomic_load(addr, __ATOMIC_RELAXED, __HIP_MEMORY_SCOPE_SYSTEM) >= target;
        if (__all(ok)) return true;
        if (__hip_atomic_load(tmo, __ATOMIC_RELAXED, __HIP_MEMORY_SCOPE_AGENT) == call ||
            __builtin_amdgcn_s_memrealtime() - t0 > DEADLINE_TICKS) {
            __hip_atomic_store(tmo, call, __ATOMIC_RELAXED, __HIP_MEMORY_SCOPE_AGENT);
            return false;
        }
        __builtin_amdgcn_s_sleep(2);
    }
}

__global__ __launch_bounds__(NTHREADS, 2)
void fused_gemm_kernel(const unsigned char* __restrict__ xs,
                       const unsigned char* __restrict__ wsrc,
                       const float* __restrict__ sx, const float* __restrict__ sw,
                       void* const* __restrict__ out_all, void* const* __restrict__ ws_all,
                       int rank) {
    __shared__ __attribute__((aligned(16))) char smem[LDS_BYTES];
    asm volatile("" :: "s"(xs), "s"(wsrc), "s"(rank));
    const int tid = threadIdx.x;
    const int lane = tid & 63;
    const int wave = __builtin_amdgcn_readfirstlane(tid >> 6);

    const int d = (int)blockIdx.x;
    unsigned* s_bcast = reinterpret_cast<unsigned*>(smem + LDS_BYTES - 64);

    const int ph = (d + wave) % 7;
    auto rot = [&](int i) { int e = i + ph; return i == 7 ? 7 : (e >= 7 ? e - 7 : e); };

    void* ws_me = ws_all[rank];
    gu32* tmo = ctrl_word(ws_me, L_TMO);
    auto get_call = [&]() {
        unsigned v;
        static_assert(L_DONE == 0, "counter offset");
        asm volatile("s_load_dword %0, %1, 0x0\n\ts_waitcnt lgkmcnt(0)" : "=s"(v) : "s"(ws_me));
        return v / NWG + 1u;
    };
    unsigned call;

    const bool xpusher = d < NPUSH, wpusher = !xpusher && d < NPUSH + NWP;
    if (xpusher || wpusher) {
        gchar* dws[NR];
        gu32* sig;
        size_t slot_off;
        auto setup = [&]() {
            unsigned nd;
            asm volatile("s_load_dword %0, %1, 0x0" : "=s"(nd) : "s"(ws_me));
#pragma unroll
            for (int i = 0; i < NR; ++i) dws[i] = gptr(ws_all[(rank + 1 + rot(i)) & 7]);
            gchar* sigb = dws[0];
#pragma unroll
            for (int i = 1; i < NR; ++i) sigb = lane == i ? dws[i] : sigb;
            sig = (gu32*)(sigb + rank * 128);
            static_assert(NWG == 256, "shift");
            asm volatile("s_waitcnt lgkmcnt(0)\n\ts_lshr_b32 %0, %1, 8" : "=s"(call) : "s"(nd) : "scc");
            call += 1u;
            slot_off = CTRL_BYTES + (size_t)(call & 1u) * STAGE_BYTES + (size_t)rank * SLOT + lane * 16;
        };
        auto signal = [&](int line) {
            __builtin_amdgcn_s_barrier();
            if (tid < NR)
                __hip_atomic_fetch_add(sig + line * 32, 1u, __ATOMIC_RELAXED, __HIP_MEMORY_SCOPE_SYSTEM);
        };
#define TIE4(a, i) asm volatile("" : "+v"(a[i]), "+v"(a[i + 1]), "+v"(a[i + 2]), "+v"(a[i + 3]))
#define TIE16(a) do { TIE4(a, 0); TIE4(a, 4); TIE4(a, 8); TIE4(a, 12); } while (0)
#define LGKM0() asm volatile("s_waitcnt lgkmcnt(0)" ::: "memory")
        if (xpusher) {
            const int ro = (d & 3) * 8 + wave;
            const unsigned char* xp = xs + (size_t)((ro >> 2) * MS + (ro & 3) * 8 + (lane >> 3) + RPO * (d >> 2)) * KS +
                                      (lane & 7) * 16;
            v4i xin[16];
#pragma unroll
            for (int j = 0; j < 16; ++j) xin[j] = ld16(xp + (size_t)(RPO * 4 * (j >> 2)) * KS + (j & 3) * BK);
            setup();
            TS(0);
            auto push_x = [&](int g, int i0, int i1) {
                const size_t off = slot_off + x_off((d >> 2) + 4 * g) + ro * 1024;
#pragma unroll
                for (int i = i0; i < i1; ++i) st16_wt(dws[i & 7] + off + (i >> 3) * XIMG, xin[4 * g + (i >> 3)]);
            };
            const int lx = L_XCNT + (d >> 2) * 8;
            WAIT_VM(12);
            TS(13);
            push_x(0, 0, 32);
            WAIT_VM(0);
            signal(lx);
            TS(3);
            if (wave == 0)
                wave_wait(ctrl_word(ws_me, L_WISS), (unsigned)NWP * (call - 1u) + WGATE, lane == 0, tmo, call);
            __builtin_amdgcn_s_barrier();
            TS(14);
            push_x(1, 0, 32);
            push_x(2, 0, KLEAD);
            WAIT_VM(KLEAD);
            signal(lx + 32);
            TS(4);
            push_x(2, KLEAD, 32);
            push_x(3, 0, KLEAD);
            WAIT_VM(KLEAD);
            signal(lx + 64);
            TS(5);
            push_x(3, KLEAD, 32);
            __builtin_amdgcn_s_barrier();
            if (tid == 64)
                __hip_atomic_fetch_add(ctrl_word(ws_me, L_SREL + (d & 7)), 1u, __ATOMIC_RELAXED, __HIP_MEMORY_SCOPE_AGENT);
            WAIT_VM(0);
            signal(lx + 96);
            TS(6);
        } else {
            const int wd = d - NPUSH;
            const int wkk = wd & 3, wtn = (wd >> 3) * 4 + ((wd >> 2) & 1);
            const unsigned char* wp = wsrc + (size_t)(wkk * BK + (lane & 7) * 16) * GN +
                                      ((rank + 1 + wave) & 7) * NS + wtn * BN + (lane >> 3) * 16;
            v4i win[2][16];
#pragma unroll
            for (int u = 0; u < 2; ++u)
#pragma unroll
                for (int j = 0; j < 16; ++j) win[u][j] = ld16(wp + (size_t)j * GN + u * 2 * BN);
            setup();
            TS(0);
            auto push_w = [&](int u, int i0, int i1) {
                const size_t off = slot_off + w_off(wtn + 2 * u) + wkk * WIMG + wave * 2048;
#pragma unroll
                for (int i = i0; i < i1; ++i) st16_wt(dws[i & 7] + off + (i >> 3) * 1024, win[u & 1][i]);
            };
            auto transpose_w = [&](int u) {
#pragma unroll
                for (int c = 0; c < 4; ++c) {
                    v4i o[4];
#pragma unroll
                    for (int q = 0; q < 4; ++q) {
                        const unsigned a0 = win[u][4 * q + 0][c], a1 = win[u][4 * q + 1][c];
                        const unsigned a2 = win[u][4 * q + 2][c], a3 = win[u][4 * q + 3][c];
                        const unsigned t0 = __builtin_amdgcn_perm(a1, a0, 0x05010400u);
                        const unsigned t1 = __builtin_amdgcn_perm(a1, a0, 0x07030602u);
                        const unsigned t2 = __builtin_amdgcn_perm(a3, a2, 0x05010400u);
                        const unsigned t3 = __builtin_amdgcn_perm(a3, a2, 0x07030602u);
                        o[0][q] = (int)__builtin_amdgcn_perm(t2, t0, 0x05040100u);
                        o[1][q] = (int)__builtin_amdgcn_perm(t2, t0, 0x07060302u);
                        o[2][q] = (int)__builtin_amdgcn_perm(t3, t1, 0x05040100u);
                        o[3][q] = (int)__builtin_amdgcn_perm(t3, t1, 0x07060302u);
                    }
#pragma unroll
                    for (int j = 0; j < 4; ++j)
                        *reinterpret_cast<v4i*>(smem + wave * (int)WIMG + (4 * c + j) * 1024 + lane * 16) = o[j];
                }
                asm volatile("s_waitcnt lgkmcnt(0)" ::: "memory");
                __builtin_amdgcn_s_barrier();
                asm volatile("" ::: "memory");
#pragma unroll
                for (int i = 0; i < 16; ++i)
                    win[u][i] = *reinterpret_cast<const v4i*>(smem + rot(i & 7) * (int)WIMG + wave * 2048 +
                                                              (i >> 3) * 1024 + lane * 16);
            };
            const int lw = L_WCNT + wtn * 8;
            WAIT_VM(16);
            TIE16(win[0]);
            transpose_w(0);
            LGKM0();
            WAIT_VM(0);
            TIE16(win[0]); TIE16(win[1]);
            __builtin_amdgcn_s_barrier();
            TS(13);
            push_w(0, 0, 8);
            transpose_w(1);
            push_w(0, 8, 16);
            push_w(1, 0, KLEAD);
            WAIT_VM(KLEAD);
            signal(lw);
            TS(1);
            push_w(1, KLEAD, 16);
            __builtin_amdgcn_s_barrier();
            if (tid == 64)
                __hip_atomic_fetch_add(ctrl_word(ws_me, L_WISS), 1u, __ATOMIC_RELAXED, __HIP_MEMORY_SCOPE_AGENT);
            TS(14);
            WAIT_VM(0);
            signal(lw + 16);
            TS(2);
        }
    } else {
        call = get_call();
        TS(0);
    }
    call = __builtin_amdgcn_readfirstlane(call);
    const size_t stage_off = CTRL_BYTES + (size_t)(call & 1u) * STAGE_BYTES;
    int tm, tn;
    {
        const int xcd = d & 7, j = d >> 3;
        constexpr int JP = (NPUSH + NWP) / 8;
        const int idx = j < JP ? 31 - j : j - JP;
        tm = 4 * (idx >> 3) + 2 * (xcd & 1) + ((idx >> 2) & 1);
        tn = 4 * (xcd >> 1) + (idx & 3);
    }

    if (wave == 0) {
        const int s = lane & 7;
        gu32* addr = ctrl_word(ws_me, (lane < 8 ? L_WCNT + tn * 8 : L_XCNT + tm * 8) + s);
        const unsigned target = (lane < 8 ? (unsigned)KBS : (unsigned)WG_PER_XTILE) * call;
        wave_wait(addr, target, lane < 16, tmo, call);
        __builtin_amdgcn_fence(__ATOMIC_ACQUIRE, "");
        const unsigned bad = __hip_atomic_load(tmo, __ATOMIC_RELAXED, __HIP_MEMORY_SCOPE_AGENT) == call;
        WAIT_VM(0);
        if (lane == 0) s_bcast[0] = bad;
    }
    __syncthreads();
    const bool dead = __builtin_amdgcn_readfirstlane(s_bcast[0]) != 0;
    __syncthreads();
    TS(7);

    const int wm = wave >> 1, wn = wave & 1;
    v16f acc[2][2];
#pragma unroll
    for (int a = 0; a < 2; ++a)
#pragma unroll
        for (int b = 0; b < 2; ++b)
#pragma unroll
            for (int i = 0; i < 16; ++i) acc[a][b][i] = 0.f;

    if (!dead) {
        const gchar* xg = gptr(ws_me) + stage_off + x_off(tm);
        const gchar* wg = gptr(ws_me) + stage_off + w_off(tn);
        const int swz = ((tid & 7) ^ ((tid >> 4) & 7)) * 16;
        const int xoff = (tid >> 3) * 128 + swz;
        int woff[2];
#pragma unroll
        for (int i = 0; i < 2; ++i) {
            const int n = i * 64 + (tid >> 3);
            woff[i] = (((n & 15) * 8) + (n >> 4)) * 128 + swz;
        }
        auto piece = [&](int kb, int buf, int i) {
            char* la = smem + buf * STAGE_LDS + tid * 16;
            if (i < 4)
                __builtin_amdgcn_global_load_lds((const gu32*)(xg + (size_t)(kb >> 2) * SLOT + (kb & 3) * XIMG + xoff + i * 8192),
                                                 (__attribute__((address_space(3))) unsigned*)(la + i * 8192), 16, 0, 0);
            else
                __builtin_amdgcn_global_load_lds((const gu32*)(wg + (size_t)(kb >> 2) * SLOT + (kb & 3) * WIMG + woff[i - 4]),
                                                 (__attribute__((address_space(3))) unsigned*)(la + (int)XIMG + (i - 4) * 8192), 16, 0, 0);
        };
        const int r32 = lane & 31, h = lane >> 5;
        const int sw = (r32 >> 1) & 7;
        int foff[2][2];
#pragma unroll
        for (int ks = 0; ks < 2; ++ks)
#pragma unroll
            for (int e = 0; e < 2; ++e) foff[ks][e] = r32 * 128 + (((ks * 4 + h * 2 + e) ^ sw) << 4);
        auto frags = [&](int buf, int ks, v8i (&af)[2], v8i (&bf)[2]) {
            const char* la = smem + buf * STAGE_LDS + (wm * 64) * 128;
            const char* lb = smem + buf * STAGE_LDS + (int)XIMG + (wn * 64) * 128;
#pragma unroll
            for (int i = 0; i < 2; ++i) {
                const v4i a0 = *reinterpret_cast<const v4i*>(la + i * 4096 + foff[ks][0]);
                const v4i a1 = *reinterpret_cast<const v4i*>(la + i * 4096 + foff[ks][1]);
                const v4i b0 = *reinterpret_cast<const v4i*>(lb + i * 4096 + foff[ks][0]);
                const v4i b1 = *reinterpret_cast<const v4i*>(lb + i * 4096 + foff[ks][1]);
                af[i] = __builtin_shufflevector(a0, a1, 0, 1, 2, 3, 4, 5, 6, 7);
                bf[i] = __builtin_shufflevector(b0, b1, 0, 1, 2, 3, 4, 5, 6, 7);
            }
        };
#define MFMA(A, B, C) C = __builtin_amdgcn_mfma_scale_f32_32x32x64_f8f6f4(A, B, C, 0, 1, 0, 127, 0, 127)
#define SB() __builtin_amdgcn_sched_barrier(0)
#pragma unroll
        for (int b3 = 0; b3 < 3; ++b3)
#pragma unroll
            for (int i = 0; i < 6; ++i) piece(b3, b3, i);
        v8i a0[2], b0[2], a1[2], b1[2];
        WAIT_VM(12);
        __builtin_amdgcn_s_barrier();
        frags(0, 0, a0, b0);
        int buf = 0;
        auto body = [&](int kb, auto tail_c) {
            constexpr int TAIL = decltype(tail_c)::value;
            const int nbuf = buf == 2 ? 0 : buf + 1;
            MFMA(a0[0], b0[0], acc[0][0]);
            SB();
            frags(buf, 1, a1, b1);
            SB();
            MFMA(a0[0], b0[1], acc[0][1]);
            MFMA(a0[1], b0[0], acc[1][0]);
            MFMA(a0[1], b0[1], acc[1][1]);
            SB();
            if (TAIL < 3) {
                asm volatile("s_waitcnt lgkmcnt(0)" ::: "memory");
                if (TAIL < 2) { WAIT_VM(6); } else { WAIT_VM(0); }
                __builtin_amdgcn_s_barrier();
            }
            SB();
            MFMA(a1[0], b1[0], acc[0][0]);
            SB();
            if (TAIL < 3) frags(nbuf, 0, a0, b0);
            SB();
            MFMA(a1[0], b1[1], acc[0][1]);
            SB();
            if (TAIL == 0) { piece(kb + 3, buf, 0); piece(kb + 3, buf, 1); piece(kb + 3, buf, 2); }
            SB();
            MFMA(a1[1], b1[0], acc[1][0]);
            SB();
            if (TAIL == 0) { piece(kb + 3, buf, 3); piece(kb + 3, buf, 4); piece(kb + 3, buf, 5); }
            SB();
            MFMA(a1[1], b1[1], acc[1][1]);
            SB();
            buf = nbuf;
        };
#pragma unroll 1
        for (int kb = 0; kb < NKB - 3; ++kb) body(kb, std::integral_constant<int, 0>{});
        body(NKB - 3, std::integral_constant<int, 1>{});
        body(NKB - 2, std::integral_constant<int, 2>{});
        body(NKB - 1, std::integral_constant<int, 3>{});
    }
    __syncthreads();
    TS(8);

    const float scale = sx[0] * sw[0];
    gf32* dout[NR];
#pragma unroll
    for (int i = 0; i < NR; ++i) dout[i] = (gf32*)out_all[(rank + 1 + rot(i)) & 7];
    {
        float* ct = reinterpret_cast<float*>(smem);
        const int r32 = lane & 31, h = lane >> 5;
        const float nanv = __builtin_nanf("");
#pragma unroll
        for (int a = 0; a < 2; ++a)
#pragma unroll
            for (int b = 0; b < 2; ++b)
#pragma unroll
                for (int i = 0; i < 16; ++i) {
                    const int row = wm * 64 + a * 32 + (i & 3) + 8 * (i >> 2) + 4 * h;
                    const int col = wn * 64 + b * 32 + r32;
                    ct[row * BN + col] = dead ? nanv : acc[a][b][i] * scale;
                }
    }
    if (wave == 0) {
        constexpr unsigned GW = NWG / 8, GP = NPUSH / 8;
        unsigned t = 0;
        if (lane == 0)
            t = __hip_atomic_fetch_add(ctrl_word(ws_me, L_STICK + (d & 7)), 1u, __ATOMIC_RELAXED, __HIP_MEMORY_SCOPE_AGENT);
        t = __builtin_amdgcn_readfirstlane(t) - GW * (call - 1u);
        if (t + 1u + GP > NSLOT)
            wave_wait(ctrl_word(ws_me, L_SREL + (d & 7)), (GW + GP) * (call - 1u) + t + 1u + GP - NSLOT,
                      lane == 0, tmo, call);
    }
    __syncthreads();
    TS(12);
    {
        const float* ct = reinterpret_cast<const float*>(smem);
        const int r32 = lane & 31, h = lane >> 5;
        const size_t doff = (size_t)(RPO * tm + 4 * wave + h) * GN + rank * NS + tn * BN + r32 * 4;
        const float* src = ct + (4 * wave + h) * BN + r32 * 4;
#pragma unroll
        for (int i = 0; i < 16; ++i) {
            const int owner = (rank + 1 + rot(i & 7)) & 7;
            const v4f v = *reinterpret_cast<const v4f*>(src + (owner * RPO + (i >> 3) * 2) * BN);
            st16_wt((gchar*)(dout[i & 7] + doff + (size_t)(i >> 3) * 2 * GN), __builtin_bit_cast(v4i, v));
            if ((i % THR_N) == THR_N - 1) WAIT_VM(THR_K);
        }
    }
    __builtin_amdgcn_s_barrier();
    if (tid == 64)
        __hip_atomic_fetch_add(ctrl_word(ws_me, L_SREL + (d & 7)), 1u, __ATOMIC_RELAXED, __HIP_MEMORY_SCOPE_AGENT);
    WAIT_VM(0);
    __syncthreads();
    TS(9);
    if (tid < NR - 1)
        __hip_atomic_fetch_add(ctrl_word(ws_all[(rank + 1 + tid) & 7], L_OCNT + rank), 1u, __ATOMIC_RELAXED,
                               __HIP_MEMORY_SCOPE_SYSTEM);
    if (tid == 63)
        __hip_atomic_fetch_add(ctrl_word(ws_me, L_DONE), 1u, __ATOMIC_RELAXED, __HIP_MEMORY_SCOPE_AGENT);
    TS(10);
    if (d == NPUSH - 1 && wave == 0) {
        const int s = lane & 7;
        const bool ok = wave_wait(ctrl_word(ws_me, L_OCNT + s), (unsigned)NWG * call,
                                  lane < 8 && s != rank, tmo, call);
        __builtin_amdgcn_fence(__ATOMIC_ACQUIRE, "");
        TS(11);
        if (!ok && lane == 0) {
            gf32* o = (gf32*)out_all[rank] + (size_t)(RPO * tm) * GN + rank * NS + tn * BN;
            *o = __builtin_nanf("");
        }
    }
}

}

extern "C" void kernel_launch(int rank, int nranks, ncclComm_t comm,
                              hipStream_t stream,
                              void* const* d_in, const int* in_sizes, int n_in,
                              void* d_out, int out_size,
                              const int* in_dtypes, int out_dtype,
                              void* const* d_out_all,
                              void* const* d_ws_all, size_t ws_bytes) {
    (void)comm; (void)in_dtypes; (void)out_dtype; (void)d_out;
    if (nranks != NR || (unsigned)rank >= (unsigned)NR || n_in < 4 || ws_bytes < WS_NEED ||
        in_sizes[0] != GM * KS || in_sizes[1] != KS * GN || out_size != MS * GN) {
        fprintf(stderr, "kernel_launch: unexpected problem geometry\n");
        return;
    }
    hipLaunchKernelGGL(fused_gemm_kernel, dim3(NWG), dim3(NTHREADS), 0, stream,
                       (const unsigned char*)d_in[0], (const unsigned char*)d_in[1],
                       (const float*)d_in[2], (const float*)d_in[3], d_out_all, d_ws_all, rank);
    HIP_CHECK(hipGetLastError());
}
```

```cpp
#include <hip/hip_runtime.h>
#include <rccl/rccl.h>
#include <cstdio>
#include <cstdint>
#include <type_traits>

#define HIP_CHECK(expr) do { hipError_t _e = (expr); if (_e != hipSuccess) { \
    fprintf(stderr, "HIP error %s at %s:%d\n", hipGetErrorString(_e), __FILE__, __LINE__); \
    return; } } while (0)

typedef int v8i __attribute__((ext_vector_type(8)));
typedef int v4i __attribute__((ext_vector_type(4)));
typedef unsigned v2u __attribute__((ext_vector_type(2)));
typedef float v16f __attribute__((ext_vector_type(16)));
typedef float v4f __attribute__((ext_vector_type(4)));
#define GAS __attribute__((address_space(1)))
typedef GAS char gchar;
typedef GAS unsigned gu32;
typedef GAS v4i gv4i;
typedef GAS v4f gv4f;
typedef GAS float gf32;

namespace {

constexpr int NR = 8;
constexpr int GM = 4096, GK = 4096, GN = 16384;
constexpr int KS = GK / NR;
constexpr int NS = GN / NR;
constexpr int MS = GM / NR;
constexpr int BM = 256, BN = 128, BK = 128;
constexpr int TM = GM / BM;
constexpr int TN = NS / BN;
constexpr int NKB = GK / BK;
constexpr int KBS = KS / BK;
constexpr int RPO = BM / NR;
constexpr int NWG = TM * TN;
constexpr int NPUSH = 16;
constexpr int NWP = 32;
constexpr int NTHREADS = 512;

constexpr size_t XIMG = (size_t)BM * BK;
constexpr size_t WIMG = (size_t)BN * BK;
constexpr size_t CTRL_BYTES = 1u << 20;
constexpr size_t SLOT = 4u << 20;
constexpr size_t STAGE_BYTES = NR * SLOT;
constexpr size_t WS_NEED = CTRL_BYTES + 4 * STAGE_BYTES;
constexpr size_t ITEM = 512u << 10;
__host__ __device__ __forceinline__ size_t x_off(int tm) {
    return (size_t)((tm >> 2) ? (tm >> 2) + 2 : 0) * ITEM + (size_t)(tm & 3) * KBS * XIMG;
}
__host__ __device__ __forceinline__ size_t w_off(int tn) {
    return (size_t)(1 + ((tn >> 1) & 1)) * ITEM + (size_t)((tn >> 2) * 2 + (tn & 1)) * KBS * WIMG;
}
#define CXCAP 63
#define CWCAP 255
constexpr unsigned XWIN = 5u * 0x01010101u, WWIN = 35u * 0x01010101u;
constexpr int CXL = 128, CXB = 3584 + CXL;
constexpr int CWL = 512, CWB = CWL + 12288;
constexpr int CWP = CWB / 8;
constexpr int WREG = CWB + 4096;
static_assert(CXCAP < CXL / 2 && CWCAP < CWL / 2 && CWP % 64 == 0, "list sizes");
constexpr size_t CXCH = (size_t)4 * 32 * CXB, CWH = (size_t)32 * CWB;
static_assert(4 * CXCH + 2 * CWH <= SLOT, "wire slot");
__device__ __forceinline__ size_t cx_off(int tm) {
    return (size_t)(tm >> 2) * CXCH + ((tm >> 2) ? 2 * CWH : 0) + (size_t)(tm & 3) * 32 * CXB;
}
__device__ __forceinline__ size_t cw_off(int tn, int kk) {
    return CXCH + (size_t)((tn >> 1) & 1) * CWH + (size_t)(((tn >> 2) * 2 + (tn & 1)) * KBS + kk) * CWB;
}
__device__ __forceinline__ unsigned bfi(unsigned m, unsigned a, unsigned b) { return (a & m) | (b & ~m); }
__device__ __forceinline__ unsigned mvbit(unsigned v, int from, int to) {
    return (from >= to ? v >> (from - to) : v << (to - from)) & (0x01010101u << to);
}

constexpr int L_DONE = 0;
constexpr int L_TMO = 1;
constexpr int L_WISS = 2;
constexpr int L_STICK = 288;
constexpr int L_SREL = 296;
constexpr int L_OCNT = 16;
constexpr int L_XCNT = 32;
constexpr int L_WCNT = 160;
constexpr int L_DX = 320;
constexpr int L_DW = 340;
constexpr int WG_PER_XTILE = NPUSH / 4;
static_assert(TN * KBS == 2 * NWP && TM == 16, "two w units per pusher");
constexpr unsigned long long DEADLINE_TICKS = 100000000ull;

constexpr int STAGE_LDS = (int)(XIMG + WIMG);
constexpr int LDS_BYTES = 3 * STAGE_LDS;
static_assert(8 * WREG <= LDS_BYTES - 64, "LDS of the w pushers");

__device__ __forceinline__ gchar* gptr(void* p) { return (gchar*)p; }
__device__ __forceinline__ gu32* ctrl_word(void* ws, int line) {
    return (gu32*)(gptr(ws) + (size_t)line * 128);
}

#define WAIT_VM_(n) asm volatile("s_waitcnt vmcnt(" #n ")" ::: "memory")
#define WAIT_VM(n) WAIT_VM_(n)
#define WGATE 10
#define NSLOT 2
#define KLEAD 4
#define THR_N 16
#define THR_K 16
#define TS(e) do { } while (0)

__device__ __forceinline__ void st16_wt(gchar* p, v4i v) {
    asm volatile("global_store_dwordx4 %0, %1, off sc0 sc1\n\ts_nop 1" :: "v"(p), "v"(v) : "memory");
}

__device__ __forceinline__ v4i ld16(const unsigned char* p) {
    v4i v;
    asm volatile("global_load_dwordx4 %0, %1, off" : "=v"(v) : "v"(p) : "memory");
    return v;
}

__device__ __forceinline__ bool wave_wait(gu32* addr, unsigned target, bool active,
                                          gu32* tmo, unsigned call) {
    const unsigned long long t0 = __builtin_amdgcn_s_memrealtime();
    for (;;) {
        bool ok = true;
        if (active)
            ok = __hip_atomic_load(addr, __ATOMIC_RELAXED, __HIP_MEMORY_SCOPE_SYSTEM) >= target;
        if (__all(ok)) return true;
        if (__hip_atomic_load(tmo, __ATOMIC_RELAXED, __HIP_MEMORY_SCOPE_AGENT) == call ||
            __builtin_amdgcn_s_memrealtime() - t0 > DEADLINE_TICKS) {
            __hip_atomic_store(tmo, call, __ATOMIC_RELAXED, __HIP_MEMORY_SCOPE_AGENT);
            return false;
        }
        __builtin_amdgcn_s_sleep(2);
    }
}

__global__ __launch_bounds__(NTHREADS, 2)
void fused_gemm_kernel(const unsigned char* __restrict__ xs,
                       const unsigned char* __restrict__ wsrc,
                       const float* __restrict__ sx, const float* __restrict__ sw,
                       void* const* __restrict__ out_all, void* const* __restrict__ ws_all,
                       int rank) {
    __shared__ __attribute__((aligned(16))) char smem[LDS_BYTES];
    asm volatile("" :: "s"(xs), "s"(wsrc), "s"(rank));
    const int tid = threadIdx.x;
    const int lane = tid & 63;
    const int wave = __builtin_amdgcn_readfirstlane(tid >> 6);

    const int d = (int)blockIdx.x;
    unsigned* s_bcast = reinterpret_cast<unsigned*>(smem + LDS_BYTES - 64);

    const int ph = (d + wave) % 7;
    auto rot = [&](int i) { int e = i + ph; return i == 7 ? 7 : (e >= 7 ? e - 7 : e); };

    void* ws_me = ws_all[rank];
    gu32* tmo = ctrl_word(ws_me, L_TMO);
    auto get_call = [&]() {
        unsigned v;
        static_assert(L_DONE == 0, "counter offset");
        asm volatile("s_load_dword %0, %1, 0x0\n\ts_waitcnt lgkmcnt(0)" : "=s"(v) : "s"(ws_me));
        return v / NWG + 1u;
    };
    unsigned call;

    auto list_bits = [&](unsigned m, unsigned tag, unsigned short* list, unsigned& cnt, unsigned cap) {
        while (cnt <= cap) {
            const unsigned long long bal = __ballot(m != 0);
            if (!bal) break;
            if (m) {
                const unsigned bit = __builtin_ctz(m);
                m &= m - 1u;
                const unsigned slot = cnt + __builtin_amdgcn_mbcnt_hi((unsigned)(bal >> 32),
                                                __builtin_amdgcn_mbcnt_lo((unsigned)bal, 1u));
                if (slot <= cap)
                    list[slot] = (unsigned short)(tag + ((bit & 4u) << 8) + ((bit & 3u) << 2) + (bit >> 3));
            }
            cnt += (unsigned)__builtin_popcountll(bal);
        }
    };

    const bool xpusher = d < NPUSH, wpusher = !xpusher && d < NPUSH + NWP;
    if (xpusher || wpusher) {
        gchar* dws[NR];
        gu32* sig;
        size_t slot_off;
        auto setup = [&]() {
            unsigned nd;
            asm volatile("s_load_dword %0, %1, 0x0" : "=s"(nd) : "s"(ws_me));
#pragma unroll
            for (int i = 0; i < NR; ++i) dws[i] = gptr(ws_all[(rank + 1 + rot(i)) & 7]);
            gchar* sigb = dws[0];
#pragma unroll
            for (int i = 1; i < NR; ++i) sigb = lane == i ? dws[i] : sigb;
            sig = (gu32*)(sigb + rank * 128);
            static_assert(NWG == 256, "shift");
            asm volatile("s_waitcnt lgkmcnt(0)\n\ts_lshr_b32 %0, %1, 8" : "=s"(call) : "s"(nd) : "scc");
            call += 1u;
            slot_off = CTRL_BYTES + (size_t)(call & 1u) * STAGE_BYTES + (size_t)rank * SLOT + lane * 16;
        };
        auto signal = [&](int line) {
            __builtin_amdgcn_s_barrier();
            if (tid < NR)
                __hip_atomic_fetch_add(sig + line * 32, 1u, __ATOMIC_RELAXED, __HIP_MEMORY_SCOPE_SYSTEM);
        };
#define TIE4(a, i) asm volatile("" : "+v"(a[i]), "+v"(a[i + 1]), "+v"(a[i + 2]), "+v"(a[i + 3]))
#define TIE16(a) do { TIE4(a, 0); TIE4(a, 4); TIE4(a, 8); TIE4(a, 12); } while (0)
#define LGKM0() asm volatile("s_waitcnt lgkmcnt(0)" ::: "memory")
        if (xpusher) {
            const int ro = (d & 3) * 8 + wave;
            const unsigned char* xp = xs + (size_t)((ro >> 2) * MS + (ro & 3) * 8 + (lane >> 3) + RPO * (d >> 2)) * KS +
                                      (lane & 7) * 16;
            v4i xin[16], xo[16];
            bool raw[4];
#pragma unroll
            for (int j = 0; j < 16; ++j) xin[j] = ld16(xp + (size_t)(RPO * 4 * (j >> 2)) * KS + (j & 3) * BK);
            setup();
            TS(0);
            auto pack_x = [&](int g) {
                unsigned c[16], bm[2] = {0u, 0u};
#pragma unroll
                for (int k = 0; k < 16; ++k) {
                    const unsigned v = (unsigned)xin[4 * g + (k >> 2)][k & 3];
                    c[k] = bfi(0x80808080u, v, (v & 0x7f7f7f7fu) + (0x80808080u - XWIN));
                    bm[k >> 3] |= mvbit(c[k], 6, k & 7);
                }
                char* tl = smem + (wave * 4 + g) * 1024;
                unsigned short* list = reinterpret_cast<unsigned short*>(tl + 512);
                unsigned cnt = 0;
                list_bits(bm[0], lane * 16, list, cnt, CXCAP);
                list_bits(bm[1], lane * 16 + 2048, list, cnt, CXCAP);
                raw[g] = cnt > CXCAP;
                if (lane == 0) list[0] = (unsigned short)(raw[g] ? 0xffffu : cnt);
                unsigned pk[14];
#pragma unroll
                for (int h = 0; h < 2; ++h)
#pragma unroll
                    for (int i = 0; i < 7; ++i)
                        pk[7 * h + i] = bfi(0x40404040u, i < 6 ? c[8 * h + 7] << (6 - i) : c[8 * h + 7] >> 1, c[8 * h + i]);
                *reinterpret_cast<v2u*>(tl + lane * 8) = v2u{pk[12], pk[13]};
                const v4i t = *reinterpret_cast<const v4i*>(tl + lane * 16);
#pragma unroll
                for (int r = 0; r < 3; ++r)
#pragma unroll
                    for (int q = 0; q < 4; ++q) xo[4 * g + r][q] = raw[g] ? xin[4 * g + r][q] : (int)pk[4 * r + q];
                xo[4 * g + 3] = t;
            };
            auto push_x = [&](int g, int i0, int i1) {
                const size_t off = slot_off + x_off((d >> 2) + 4 * g) + ro * 1024;
                const size_t coff = slot_off + 2 * STAGE_BYTES + cx_off((d >> 2) + 4 * g) + ro * CXB;
                const size_t o0 = raw[g] ? off : coff, os = raw[g] ? XIMG : 1024;
                if (i0 == 0 && raw[g]) {
                    if (lane >= 32 && lane < 40)
#pragma unroll
                        for (int i = 0; i < 7; ++i) st16_wt(dws[i] + coff + 3072, xo[4 * g + 3]);
                    xo[4 * g + 3] = xin[4 * g + 3];
                }
#pragma unroll
                for (int i = i0; i < i1; ++i) {
                    const int r = i >> 3;
                    if ((i & 7) == 7) st16_wt(dws[7] + off + r * XIMG, xin[4 * g + r]);
                    else if (r < 3 || raw[g] || lane < 40) st16_wt(dws[i & 7] + o0 + r * os, xo[4 * g + r]);
                }
            };
            const int lx = L_XCNT + (d >> 2) * 8;
            WAIT_VM(12);
            TIE4(xin, 0);
            pack_x(0);
            TS(13);
            push_x(0, 0, 32);
            WAIT_VM(25);
            TIE4(xin, 4); TIE4(xin, 8); TIE4(xin, 12);
            pack_x(1); pack_x(2); pack_x(3);
            WAIT_VM(0);
            signal(lx);
            TS(3);
            if (wave == 0)
                wave_wait(ctrl_word(ws_me, L_WISS), (unsigned)NWP * (call - 1u) + WGATE, lane == 0, tmo, call);
            __builtin_amdgcn_s_barrier();
            TS(14);
            push_x(1, 0, 32);
            push_x(2, 0, KLEAD);
            WAIT_VM(KLEAD);
            signal(lx + 32);
            TS(4);
            push_x(2, KLEAD, 32);
            push_x(3, 0, KLEAD);
            WAIT_VM(KLEAD);
            signal(lx + 64);
            TS(5);
            push_x(3, KLEAD, 32);
            __builtin_amdgcn_s_barrier();
            if (tid == 64)
                __hip_atomic_fetch_add(ctrl_word(ws_me, L_SREL + (d & 7)), 1u, __ATOMIC_RELAXED, __HIP_MEMORY_SCOPE_AGENT);
            WAIT_VM(0);
            signal(lx + 96);
            TS(6);
        } else {
            const int wd = d - NPUSH;
            const int wkk = wd & 3, wtn = (wd >> 3) * 4 + ((wd >> 2) & 1);
            const unsigned char* wp = wsrc + (size_t)(wkk * BK + (lane & 7) * 16) * GN +
                                      ((rank + 1 + wave) & 7) * NS + wtn * BN + (lane >> 3) * 16;
            v4i win[2][16];
            unsigned rawm[2];
#pragma unroll
            for (int u = 0; u < 2; ++u)
#pragma unroll
                for (int j = 0; j < 16; ++j) win[u][j] = ld16(wp + (size_t)j * GN + u * 2 * BN);
            setup();
            TS(0);
            auto push_w = [&](int u, int i0, int i1) {
                const size_t off = slot_off + w_off(wtn + 2 * u) + wkk * WIMG + wave * 2048;
                const size_t coff = slot_off + 2 * STAGE_BYTES + cw_off(wtn + 2 * u, wkk);
                if (i0 == 0 && (rawm[u] & 127u) && wave == 0 && lane == 0) {
                    const v4i hd = {0xffff, 0, 0, 0};
#pragma unroll
                    for (int i = 0; i < 7; ++i)
                        if ((rawm[u] >> i) & 1u) st16_wt(dws[i] + coff, hd);
                }
#pragma unroll
                for (int i = i0; i < i1; ++i) {
                    const bool rw = (i & 7) == 7 || ((rawm[u] >> (i & 7)) & 1u);
                    if (i < 8 || rw || lane < (CWP - 1024) / 16)
                        st16_wt(dws[i & 7] + (rw ? off : coff + wave * CWP) + (i >> 3) * 1024, win[u][i]);
                }
            };
            auto tr4 = [&](int u, int c, v4i (&o)[4]) {
#pragma unroll
                for (int q = 0; q < 4; ++q) {
                    const unsigned a0 = win[u][4 * q + 0][c], a1 = win[u][4 * q + 1][c];
                    const unsigned a2 = win[u][4 * q + 2][c], a3 = win[u][4 * q + 3][c];
                    const unsigned t0 = __builtin_amdgcn_perm(a1, a0, 0x05010400u);
                    const unsigned t1 = __builtin_amdgcn_perm(a1, a0, 0x07030602u);
                    const unsigned t2 = __builtin_amdgcn_perm(a3, a2, 0x05010400u);
                    const unsigned t3 = __builtin_amdgcn_perm(a3, a2, 0x07030602u);
                    o[0][q] = (int)__builtin_amdgcn_perm(t2, t0, 0x05040100u);
                    o[1][q] = (int)__builtin_amdgcn_perm(t2, t0, 0x07060302u);
                    o[2][q] = (int)__builtin_amdgcn_perm(t3, t1, 0x05040100u);
                    o[3][q] = (int)__builtin_amdgcn_perm(t3, t1, 0x07060302u);
                }
            };
            auto transpose_w = [&](int u) {
                char* reg = smem + wave * WREG;
                bool rw = wave == 7;
                if (!rw) {
                    unsigned nz = 0;
#pragma unroll
                    for (int c = 0; c < 4; ++c) {
                        v4i o[4], mk;
                        tr4(u, c, o);
#pragma unroll
                        for (int j = 0; j < 4; ++j) {
                            unsigned m = 0;
#pragma unroll
                            for (int q = 0; q < 4; ++q) {
                                const unsigned t = ((unsigned)o[j][q] & 0x7f7f7f7fu) + (0x80808080u - WWIN);
                                m |= (q < 3 ? t >> (5 - 2 * q) : t << 1) & (0x03030303u << (2 * q));
                            }
                            mk[j] = (int)m;
                            nz |= (m < 1u ? m : 1u) << (4 * c + j);
                        }
                        *reinterpret_cast<v4i*>(reg + CWB + c * 1024 + lane * 16) = mk;
#pragma unroll
                        for (int j = 0; j < 3; ++j) {
                            v4i pk;
#pragma unroll
                            for (int q = 0; q < 4; ++q) {
                                const unsigned c3 = bfi(0x20202020u, (unsigned)o[3][q] >> 2, (unsigned)o[3][q]);
                                pk[q] = (int)bfi(0x60606060u, c3 << (5 - 2 * j), (unsigned)o[j][q]);
                            }
                            *reinterpret_cast<v4i*>(reg + CWL + (3 * c + j) * 1024 + lane * 16) = pk;
                        }
                    }
                    unsigned short* list = reinterpret_cast<unsigned short*>(reg);
                    unsigned cnt = 0, m = 0, wb = 0;
                    while (cnt <= CWCAP) {
                        if (m == 0 && nz != 0) {
                            const unsigned w = __builtin_ctz(nz);
                            nz &= nz - 1u;
                            m = *reinterpret_cast<const unsigned*>(reg + CWB + (w >> 2) * 1024 + lane * 16 + (w & 3u) * 4);
                            wb = w * 1024 + lane * 16;
                        }
                        const unsigned long long bal = __ballot(m != 0);
                        if (!bal) break;
                        if (m) {
                            const unsigned bit = __builtin_ctz(m) & 30u;
                            const unsigned slot = cnt + __builtin_amdgcn_mbcnt_hi((unsigned)(bal >> 32),
                                                            __builtin_amdgcn_mbcnt_lo((unsigned)bal, 1u));
                            if (slot <= CWCAP)
                                list[slot] = (unsigned short)(wb + ((bit & 6u) << 1) + (bit >> 3) + ((m >> bit) << 14));
                            m &= ~(3u << bit);
                        }
                        cnt += (unsigned)__builtin_popcountll(bal);
                    }
                    if (lane == 0) list[0] = (unsigned short)cnt;
                    rw = cnt > CWCAP;
                }
                if (rw) {
#pragma unroll
                    for (int c = 0; c < 4; ++c) {
                        v4i o[4];
                        tr4(u, c, o);
#pragma unroll
                        for (int j = 0; j < 4; ++j)
                            *reinterpret_cast<v4i*>(reg + (4 * c + j) * 1024 + lane * 16) = o[j];
                    }
                }
                if (lane == 0) s_bcast[wave] = rw;
                asm volatile("s_waitcnt lgkmcnt(0)" ::: "memory");
                __builtin_amdgcn_s_barrier();
                asm volatile("" ::: "memory");
                rawm[u] = (unsigned)__ballot(lane < 8 && s_bcast[rot(lane & 7)] != 0);
#pragma unroll
                for (int i = 0; i < 16; ++i) {
                    const bool rw2 = (i & 7) == 7 || ((rawm[u] >> (i & 7)) & 1u);
                    win[u][i] = *reinterpret_cast<const v4i*>(smem + rot(i & 7) * WREG + wave * (rw2 ? 2048 : CWP) +
                                                              (i >> 3) * 1024 + lane * 16);
                }
            };
            const int lw = L_WCNT + wtn * 8;
            WAIT_VM(16);
            TIE16(win[0]);
            transpose_w(0);
            LGKM0();
            WAIT_VM(0);
            TIE16(win[0]); TIE16(win[1]);
            __builtin_amdgcn_s_barrier();
            TS(13);
            push_w(0, 0, 8);
            transpose_w(1);
            push_w(0, 8, 16);
            push_w(1, 0, KLEAD);
            WAIT_VM(KLEAD);
            signal(lw);
            TS(1);
            push_w(1, KLEAD, 16);
            __builtin_amdgcn_s_barrier();
            if (tid == 64)
                __hip_atomic_fetch_add(ctrl_word(ws_me, L_WISS), 1u, __ATOMIC_RELAXED, __HIP_MEMORY_SCOPE_AGENT);
            TS(14);
            WAIT_VM(0);
            signal(lw + 16);
            TS(2);
        }
    } else {
        call = get_call();
        TS(0);
    }
    call = __builtin_amdgcn_readfirstlane(call);
    const size_t stage_off = CTRL_BYTES + (size_t)(call & 1u) * STAGE_BYTES;
    const int xcd = d & 7;
    constexpr int JP = (NPUSH + NWP) / 8;
    const int idx = (d >> 3) < JP ? 31 - (d >> 3) : (d >> 3) - JP;
    const int tm = 4 * (idx >> 3) + 2 * (xcd & 1) + ((idx >> 2) & 1);
    const int tn = 4 * (xcd >> 1) + (idx & 3);

    {
        const int o = idx * 8 + xcd;
#pragma unroll 1
        for (int ph2 = 0; ph2 < 7; ++ph2) {
            int t;
            if (ph2 == 0 || ph2 == 2) t = o < 208 ? o - 96 : -1;
            else if (ph2 == 1) t = o < 112 ? o : -1;
            else if (ph2 < 5) t = o < 240 ? o - 128 : -1;
            else if (ph2 == 5) t = o - 192;
            else t = o >= 192 && o < 240 ? o - 128 : -1;
            if (t < 0) continue;
            const bool isw = ph2 == 1 || ph2 == 2;
            const int src = (rank + 1 + t % 7) & 7, r = t / 7;
            const int tx = 4 * (ph2 == 0 ? 0 : (ph2 == 6 ? 3 : ph2 - 2)) + (r >> 2);
            const int tw = (r >> 2) * 4 + ((r >> 1) & 1) + 2 * (ph2 - 1);
            if (wave == 0) {
                wave_wait(ctrl_word(ws_me, (isw ? L_WCNT + tw * 8 : L_XCNT + tx * 8) + src), 4u * call, lane == 0, tmo, call);
                __builtin_amdgcn_fence(__ATOMIC_ACQUIRE, "");
            }
            __syncthreads();
            const gchar* wire = gptr(ws_me) + stage_off + 2 * STAGE_BYTES + (size_t)src * SLOT;
            gchar* img = gptr(ws_me) + stage_off + (size_t)src * SLOT + lane * 16;
            char* pt = smem + wave * 4096;
            v4i pin[3], res[4];
            size_t ostep;
            int lso;
            if (isw) {
                const int kk = (r & 1) * 2 + (wave >> 2), m = wave & 3;
                wire += cw_off(tw, kk) + CWL + m * 3072;
                lso = lane * 8 - (CWL + m * 3072);
                img += w_off(tw) + kk * WIMG + m * 4096;
                ostep = 1024;
            } else {
                const int ro = (r & 3) * 8 + wave;
                wire += cx_off(tx) + ro * CXB;
                lso = 3584 + (lane & 15) * 8;
                img += x_off(tx) + ro * 1024;
                ostep = XIMG;
            }
            v2u ls = *(const GAS v2u*)(wire + lso);
            v2u tl = *(const GAS v2u*)(wire + (isw ? 0 : 3072) + lane * 8);
#pragma unroll
            for (int j = 0; j < 3; ++j) pin[j] = *(const gv4i*)(wire + j * 1024 + lane * 16);
#pragma unroll
            for (int j = 0; j < 4; ++j) *reinterpret_cast<v4i*>(pt + j * 1024 + lane * 16) = v4i{0, 0, 0, 0};
            asm volatile("" : "+v"(ls), "+v"(tl), "+v"(pin[0]), "+v"(pin[1]), "+v"(pin[2]));
            const unsigned cnt = __builtin_amdgcn_readfirstlane(ls.x) & 0xffffu;
            if (cnt != 0xffffu) {
                if (isw) {
#pragma unroll
                    for (int e = 0; e < 4; ++e) {
                        const unsigned en = ((e & 2) ? ls.y : ls.x) >> ((e & 1) * 16);
                        const unsigned slot = lane * 4 + e;
                        if (slot >= 1u && slot <= cnt && ((en >> 12) & 3u) == (unsigned)(wave & 3))
                            pt[en & 0xfffu] = (char)((en >> 9) & 0x60u);
                    }
#pragma unroll
                    for (int j = 0; j < 4; ++j) res[j] = *reinterpret_cast<const v4i*>(pt + j * 1024 + lane * 16);
#pragma unroll
                    for (int q = 0; q < 4; ++q) {
                        const unsigned c3 = (((unsigned)pin[0][q] >> 5) & 0x03030303u) | (((unsigned)pin[1][q] >> 3) & 0x0c0c0c0cu) |
                                            (((unsigned)pin[2][q] >> 1) & 0x30303030u);
#pragma unroll
                        for (int j = 0; j < 4; ++j) {
                            const unsigned cc = j < 3 ? (unsigned)pin[j][q] : c3;
                            res[j][q] = (int)bfi(0x80808080u, j < 3 ? cc : cc << 2,
                                                 ((((cc & 0x1f1f1f1fu) + (0x40404040u - WWIN)) & 0x1f1f1f1fu) | (unsigned)res[j][q]) + WWIN);
                        }
                    }
                } else {
#pragma unroll
                    for (int e = 0; e < 4; ++e) {
                        const unsigned slot = lane * 4 + e;
                        if (slot >= 1u && slot <= cnt) pt[(((e & 2) ? ls.y : ls.x) >> ((e & 1) * 16)) & 0xfffu] = 0x40;
                    }
#pragma unroll
                    for (int j = 0; j < 4; ++j) res[j] = *reinterpret_cast<const v4i*>(pt + j * 1024 + lane * 16);
                    unsigned pk[14];
#pragma unroll
                    for (int k = 0; k < 12; ++k) pk[k] = (unsigned)pin[k >> 2][k & 3];
                    pk[12] = tl.x; pk[13] = tl.y;
#pragma unroll
                    for (int h = 0; h < 2; ++h) {
                        unsigned c7 = mvbit(pk[7 * h + 6], 6, 7);
#pragma unroll
                        for (int i = 0; i < 6; ++i) c7 |= mvbit(pk[7 * h + i], 6, i);
#pragma unroll
                        for (int i = 0; i < 8; ++i) {
                            const int k = 8 * h + i;
                            const unsigned cc = i < 7 ? pk[7 * h + i] : c7;
                            res[k >> 2][k & 3] = (int)bfi(0x80808080u, cc,
                                                          ((cc & 0x3f3f3f3fu) | (unsigned)res[k >> 2][k & 3]) + XWIN);
                        }
                    }
                }
#pragma unroll
                for (int j = 0; j < 4; ++j) st16_wt(img + j * ostep, res[j]);
            }
            WAIT_VM(0);
            __syncthreads();
            if (tid == 0)
                __hip_atomic_fetch_add(ctrl_word(ws_me, isw ? L_DW + tw : L_DX + tx), 1u, __ATOMIC_RELAXED,
                                       __HIP_MEMORY_SCOPE_AGENT);
        }
    }
    TS(15);

    if (wave == 0) {
        gu32* addr = ctrl_word(ws_me, lane == 0 ? L_DW + tn : lane == 1 ? L_DX + tm
                                      : lane == 2 ? L_WCNT + tn * 8 + rank : L_XCNT + tm * 8 + rank);
        const unsigned target = (lane == 0 ? 14u : lane == 1 ? 28u : 4u) * call;
        wave_wait(addr, target, lane < 4, tmo, call);
        __builtin_amdgcn_fence(__ATOMIC_ACQUIRE, "");
        const unsigned bad = __hip_atomic_load(tmo, __ATOMIC_RELAXED, __HIP_MEMORY_SCOPE_AGENT) == call;
        WAIT_VM(0);
        if (lane == 0) s_bcast[0] = bad;
    }
    __syncthreads();
    const bool dead = __builtin_amdgcn_readfirstlane(s_bcast[0]) != 0;
    __syncthreads();
    TS(7);

    const int wm = wave >> 1, wn = wave & 1;
    v16f acc[2][2];
#pragma unroll
    for (int a = 0; a < 2; ++a)
#pragma unroll
        for (int b = 0; b < 2; ++b)
#pragma unroll
            for (int i = 0; i < 16; ++i) acc[a][b][i] = 0.f;

    if (!dead) {
        const gchar* xg = gptr(ws_me) + stage_off + x_off(tm);
        const gchar* wg = gptr(ws_me) + stage_off + w_off(tn);
        const int swz = ((tid & 7) ^ ((tid >> 4) & 7)) * 16;
        const int xoff = (tid >> 3) * 128 + swz;
        int woff[2];
#pragma unroll
        for (int i = 0; i < 2; ++i) {
            const int n = i * 64 + (tid >> 3);
            woff[i] = (((n & 15) * 8) + (n >> 4)) * 128 + swz;
        }
        auto piece = [&](int kb, int buf, int i) {
            char* la = smem + buf * STAGE_LDS + tid * 16;
            if (i < 4)
                __builtin_amdgcn_global_load_lds((const gu32*)(xg + (size_t)(kb >> 2) * SLOT + (kb & 3) * XIMG + xoff + i * 8192),
                                                 (__attribute__((address_space(3))) unsigned*)(la + i * 8192), 16, 0, 0);
            else
                __builtin_amdgcn_global_load_lds((const gu32*)(wg + (size_t)(kb >> 2) * SLOT + (kb & 3) * WIMG + woff[i - 4]),
                                                 (__attribute__((address_space(3))) unsigned*)(la + (int)XIMG + (i - 4) * 8192), 16, 0, 0);
        };
        const int r32 = lane & 31, h = lane >> 5;
        const int sw = (r32 >> 1) & 7;
        int foff[2][2];
#pragma unroll
        for (int ks = 0; ks < 2; ++ks)
#pragma unroll
            for (int e = 0; e < 2; ++e) foff[ks][e] = r32 * 128 + (((ks * 4 + h * 2 + e) ^ sw) << 4);
        auto frags = [&](int buf, int ks, v8i (&af)[2], v8i (&bf)[2]) {
            const char* la = smem + buf * STAGE_LDS + (wm * 64) * 128;
            const char* lb = smem + buf * STAGE_LDS + (int)XIMG + (wn * 64) * 128;
#pragma unroll
            for (int i = 0; i < 2; ++i) {
                const v4i a0 = *reinterpret_cast<const v4i*>(la + i * 4096 + foff[ks][0]);
                const v4i a1 = *reinterpret_cast<const v4i*>(la + i * 4096 + foff[ks][1]);
                const v4i b0 = *reinterpret_cast<const v4i*>(lb + i * 4096 + foff[ks][0]);
                const v4i b1 = *reinterpret_cast<const v4i*>(lb + i * 4096 + foff[ks][1]);
                af[i] = __builtin_shufflevector(a0, a1, 0, 1, 2, 3, 4, 5, 6, 7);
                bf[i] = __builtin_shufflevector(b0, b1, 0, 1, 2, 3, 4, 5, 6, 7);
            }
        };
#define MFMA(A, B, C) C = __builtin_amdgcn_mfma_scale_f32_32x32x64_f8f6f4(A, B, C, 0, 1, 0, 127, 0, 127)
#define SB() __builtin_amdgcn_sched_barrier(0)
#pragma unroll
        for (int b3 = 0; b3 < 3; ++b3)
#pragma unroll
            for (int i = 0; i < 6; ++i) piece(b3, b3, i);
        v8i a0[2], b0[2], a1[2], b1[2];
        WAIT_VM(12);
        __builtin_amdgcn_s_barrier();
        frags(0, 0, a0, b0);
        int buf = 0;
        auto body = [&](int kb, auto tail_c) {
            constexpr int TAIL = decltype(tail_c)::value;
            const int nbuf = buf == 2 ? 0 : buf + 1;
            MFMA(a0[0], b0[0], acc[0][0]);
            SB();
            frags(buf, 1, a1, b1);
            SB();
            MFMA(a0[0], b0[1], acc[0][1]);
            MFMA(a0[1], b0[0], acc[1][0]);
            MFMA(a0[1], b0[1], acc[1][1]);
            SB();
            if (TAIL < 3) {
                asm volatile("s_waitcnt lgkmcnt(0)" ::: "memory");
                if (TAIL < 2) { WAIT_VM(6); } else { WAIT_VM(0); }
                __builtin_amdgcn_s_barrier();
            }
            SB();
            MFMA(a1[0], b1[0], acc[0][0]);
            SB();
            if (TAIL < 3) frags(nbuf, 0, a0, b0);
            SB();
            MFMA(a1[0], b1[1], acc[0][1]);
            SB();
            if (TAIL == 0) { piece(kb + 3, buf, 0); piece(kb + 3, buf, 1); piece(kb + 3, buf, 2); }
            SB();
            MFMA(a1[1], b1[0], acc[1][0]);
            SB();
            if (TAIL == 0) { piece(kb + 3, buf, 3); piece(kb + 3, buf, 4); piece(kb + 3, buf, 5); }
            SB();
            MFMA(a1[1], b1[1], acc[1][1]);
            SB();
            buf = nbuf;
        };
#pragma unroll 1
        for (int kb = 0; kb < NKB - 3; ++kb) body(kb, std::integral_constant<int, 0>{});
        body(NKB - 3, std::integral_constant<int, 1>{});
        body(NKB - 2, std::integral_constant<int, 2>{});
        body(NKB - 1, std::integral_constant<int, 3>{});
    }
    __syncthreads();
    TS(8);

    const float scale = sx[0] * sw[0];
    gf32* dout[NR];
#pragma unroll
    for (int i = 0; i < NR; ++i) dout[i] = (gf32*)out_all[(rank + 1 + rot(i)) & 7];
    {
        float* ct = reinterpret_cast<float*>(smem);
        const int r32 = lane & 31, h = lane >> 5;
        const float nanv = __builtin_nanf("");
#pragma unroll
        for (int a = 0; a < 2; ++a)
#pragma unroll
            for (int b = 0; b < 2; ++b)
#pragma unroll
                for (int i = 0; i < 16; ++i) {
                    const int row = wm * 64 + a * 32 + (i & 3) + 8 * (i >> 2) + 4 * h;
                    const int col = wn * 64 + b * 32 + r32;
                    ct[row * BN + col] = dead ? nanv : acc[a][b][i] * scale;
                }
    }
    if (wave == 0) {
        constexpr unsigned GW = NWG / 8, GP = NPUSH / 8;
        unsigned t = 0;
        if (lane == 0)
            t = __hip_atomic_fetch_add(ctrl_word(ws_me, L_STICK + (d & 7)), 1u, __ATOMIC_RELAXED, __HIP_MEMORY_SCOPE_AGENT);
        t = __builtin_amdgcn_readfirstlane(t) - GW * (call - 1u);
        if (t + 1u + GP > NSLOT)
            wave_wait(ctrl_word(ws_me, L_SREL + (d & 7)), (GW + GP) * (call - 1u) + t + 1u + GP - NSLOT,
                      lane == 0, tmo, call);
    }
    __syncthreads();
    TS(12);
    {
        const float* ct = reinterpret_cast<const float*>(smem);
        const int r32 = lane & 31, h = lane >> 5;
        const size_t doff = (size_t)(RPO * tm + 4 * wave + h) * GN + rank * NS + tn * BN + r32 * 4;
        const float* src = ct + (4 * wave + h) * BN + r32 * 4;
#pragma unroll
        for (int i = 0; i < 16; ++i) {
            const int owner = (rank + 1 + rot(i & 7)) & 7;
            const v4f v = *reinterpret_cast<const v4f*>(src + (owner * RPO + (i >> 3) * 2) * BN);
            st16_wt((gchar*)(dout[i & 7] + doff + (size_t)(i >> 3) * 2 * GN), __builtin_bit_cast(v4i, v));
            if ((i % THR_N) == THR_N - 1) WAIT_VM(THR_K);
        }
    }
    __builtin_amdgcn_s_barrier();
    if (tid == 64)
        __hip_atomic_fetch_add(ctrl_word(ws_me, L_SREL + (d & 7)), 1u, __ATOMIC_RELAXED, __HIP_MEMORY_SCOPE_AGENT);
    WAIT_VM(0);
    __syncthreads();
    TS(9);
    if (tid < NR - 1)
        __hip_atomic_fetch_add(ctrl_word(ws_all[(rank + 1 + tid) & 7], L_OCNT + rank), 1u, __ATOMIC_RELAXED,
                               __HIP_MEMORY_SCOPE_SYSTEM);
    if (tid == 63)
        __hip_atomic_fetch_add(ctrl_word(ws_me, L_DONE), 1u, __ATOMIC_RELAXED, __HIP_MEMORY_SCOPE_AGENT);
    TS(10);
    if (d == NPUSH - 1 && wave == 0) {
        const int s = lane & 7;
        const bool ok = wave_wait(ctrl_word(ws_me, L_OCNT + s), (unsigned)NWG * call,
                                  lane < 8 && s != rank, tmo, call);
        __builtin_amdgcn_fence(__ATOMIC_ACQUIRE, "");
        TS(11);
        if (!ok && lane == 0) {
            gf32* o = (gf32*)out_all[rank] + (size_t)(RPO * tm) * GN + rank * NS + tn * BN;
            *o = __builtin_nanf("");
        }
    }
}

}

extern "C" void kernel_launch(int rank, int nranks, ncclComm_t comm,
                              hipStream_t stream,
                              void* const* d_in, const int* in_sizes, int n_in,
                              void* d_out, int out_size,
                              const int* in_dtypes, int out_dtype,
                              void* const* d_out_all,
                              void* const* d_ws_all, size_t ws_bytes) {
    (void)comm; (void)in_dtypes; (void)out_dtype; (void)d_out;
    if (nranks != NR || (unsigned)rank >= (unsigned)NR || n_in < 4 || ws_bytes < WS_NEED ||
        in_sizes[0] != GM * KS || in_sizes[1] != KS * GN || out_size != MS * GN) {
        fprintf(stderr, "kernel_launch: unexpected problem geometry\n");
        return;
    }
    hipLaunchKernelGGL(fused_gemm_kernel, dim3(NWG), dim3(NTHREADS), 0, stream,
                       (const unsigned char*)d_in[0], (const unsigned char*)d_in[1],
                       (const float*)d_in[2], (const float*)d_in[3], d_out_all, d_ws_all, rank);
    HIP_CHECK(hipGetLastError());
}
```

```cpp
#include <hip/hip_runtime.h>
#include <rccl/rccl.h>
#include <cstdio>
#include <cstdint>

#define HIP_CHECK(expr) do { hipError_t _e = (expr); if (_e != hipSuccess) { \
    fprintf(stderr, "HIP error %s at %s:%d\n", hipGetErrorString(_e), __FILE__, __LINE__); \
    return; } } while (0)

typedef int v8i __attribute__((ext_vector_type(8)));
typedef int v4i __attribute__((ext_vector_type(4)));
typedef unsigned v4u __attribute__((ext_vector_type(4)));
typedef unsigned v2u __attribute__((ext_vector_type(2)));
typedef float v16f __attribute__((ext_vector_type(16)));
typedef unsigned __attribute__((aligned(1))) u32u;
#define GAS __attribute__((address_space(1)))
typedef GAS char gchar;
typedef GAS unsigned gu32;
typedef GAS v4i gv4i;

namespace {

constexpr int NR = 8;
constexpr int GM = 4096, GK = 4096, GN = 16384;
constexpr int KS = GK / NR;
constexpr int MS = GM / NR;
constexpr int BN = 64, BK = 128;
constexpr int KBS = KS / BK;
constexpr int NWG = GN / BN;
constexpr int NTHREADS = 704;
constexpr int GW = NWG / 8;
constexpr int XP0 = 16, XPN = 4;

constexpr size_t CTRL_BYTES = 1u << 20;
constexpr int XITEM = MS * BK;
constexpr int REC = BK * BN;
constexpr size_t X_BYTES = (size_t)NR * KBS * XITEM;
constexpr size_t PAR_BYTES = X_BYTES + (size_t)NWG * NR * KBS * REC;
constexpr size_t WS_NEED = CTRL_BYTES + 2 * PAR_BYTES;
constexpr int L_DONE = 0;
constexpr int L_TMO = 1;
constexpr int L_SREL = 2;
constexpr int L_XCNT = 16;
constexpr int L_FLAG = 512;
constexpr unsigned long long DEADLINE_TICKS = 100000000ull;

#define CREC 6144
constexpr int RVAR = 5120, RS1 = RVAR + 32;
static_assert(CREC <= 6144 && CREC >= RS1, "record size");

constexpr int S_RAW = 0;
constexpr int S_WIRE = S_RAW + KBS * REC;
constexpr int S_DEC = S_WIRE + KBS * 6144;
constexpr int NS = 8;
constexpr int S_SLOT = S_DEC + 2 * REC;
constexpr int S_SNAP = S_SLOT + NS * REC;
constexpr int S_MISC = S_SNAP + 256;
constexpr int S_PUB = S_MISC + 256;
constexpr int S_CNT = S_PUB + 128;
constexpr int S_GO = S_CNT + 4;
constexpr int LDS_BYTES = S_CNT + 16;
static_assert(LDS_BYTES > 80 * 1024, "LDS");

__device__ __forceinline__ gchar* gptr(void* p) { return (gchar*)p; }
__device__ __forceinline__ gu32* ctrl_word(void* ws, int line) {
    return (gu32*)(gptr(ws) + (size_t)line * 128);
}
__device__ __forceinline__ unsigned bfi(unsigned m, unsigned a, unsigned b) { return (a & m) | (b & ~m); }
__device__ __forceinline__ unsigned wave_scan(unsigned s) {
    s += __builtin_amdgcn_update_dpp(0u, s, 0x111, 0xf, 0xf, false);
    s += __builtin_amdgcn_update_dpp(0u, s, 0x112, 0xf, 0xf, false);
    s += __builtin_amdgcn_update_dpp(0u, s, 0x114, 0xf, 0xf, false);
    s += __builtin_amdgcn_update_dpp(0u, s, 0x118, 0xf, 0xf, false);
    s += __builtin_amdgcn_update_dpp(0u, s, 0x142, 0xa, 0xf, false);
    s += __builtin_amdgcn_update_dpp(0u, s, 0x143, 0xc, 0xf, false);
    return s;
}

#define WAIT_VM_(n) asm volatile("s_waitcnt vmcnt(" #n ")" ::: "memory")
#define WAIT_VM(n) WAIT_VM_(n)
#define NSLOT 20
#define ROT 4
#define TSC(c, e) do { } while (0)
#define TS(e) do { } while (0)
#define TF(e) do { } while (0)

__device__ __forceinline__ void st16_wt(gchar* base, unsigned off, v4i v) {
    asm volatile("s_nop 4\n\tglobal_store_dwordx4 %0, %1, %2 sc0 sc1\n\ts_nop 1" :: "v"(off), "v"(v), "s"(base) : "memory");
}
__device__ __forceinline__ void ld4k_lds(const gchar* base, unsigned off, unsigned lds) {
    asm volatile("s_mov_b32 m0, %2\n\ts_nop 4\n\t"
                 "global_load_lds_dwordx4 %0, %1 sc0 sc1\n\t"
                 "global_load_lds_dwordx4 %0, %1 offset:1024 sc0 sc1\n\t"
                 "global_load_lds_dwordx4 %0, %1 offset:2048 sc0 sc1\n\t"
                 "global_load_lds_dwordx4 %0, %1 offset:3072 sc0 sc1"
                 :: "v"(off), "s"(base), "s"(lds) : "memory");
}
__device__ __forceinline__ void ld2k_lds(const gchar* base, unsigned off, unsigned lds) {
    asm volatile("s_mov_b32 m0, %2\n\ts_nop 4\n\t"
                 "global_load_lds_dwordx4 %0, %1 sc0 sc1\n\t"
                 "global_load_lds_dwordx4 %0, %1 offset:1024 sc0 sc1"
                 :: "v"(off), "s"(base), "s"(lds) : "memory");
}
__device__ __forceinline__ void ld4_lds(const gchar* base, unsigned off, unsigned lds, unsigned long long lanes) {
    asm volatile("s_mov_b32 m0, %2\n\ts_mov_b64 exec, %3\n\ts_nop 3\n\t"
                 "global_load_lds_dword %0, %1 sc0 sc1\n\ts_mov_b64 exec, -1"
                 :: "v"(off), "s"(base), "s"(lds), "s"(lanes) : "memory");
}
__device__ __forceinline__ unsigned lds_get(unsigned addr) {
    unsigned v;
    asm volatile("ds_read_b32 %0, %1\n\ts_waitcnt lgkmcnt(0)" : "=v"(v) : "v"(addr) : "memory");
    return v;
}
__device__ __forceinline__ void lds_put(unsigned addr, unsigned v) {
    asm volatile("s_mov_b64 exec, 1\n\tds_write_b32 %0, %1\n\ts_mov_b64 exec, -1" :: "v"(addr), "v"(v) : "memory");
}
__device__ __forceinline__ void lds_inc(unsigned addr) {
    asm volatile("s_mov_b64 exec, 1\n\tds_add_u32 %0, %1\n\ts_mov_b64 exec, -1" :: "v"(addr), "v"(1u) : "memory");
}
constexpr unsigned SPIN_MAX = 1u << 25;
__device__ __forceinline__ void lds_barrier() {
    asm volatile("s_waitcnt lgkmcnt(0)\n\ts_barrier" ::: "memory");
}

enum : unsigned { C_ITEM = 0x100u, C_PLAIN = 0x20u, C_OWN = 0x40u, C_DEAD = 0x200u };

__global__ __launch_bounds__(NTHREADS)
void fused_gemm_kernel(const unsigned char* __restrict__ xs,
                       const unsigned char* __restrict__ wsrc,
                       const float* __restrict__ sx, const float* __restrict__ sw,
                       float* __restrict__ out, void* const* __restrict__ ws_all, int rank) {
    __shared__ __attribute__((aligned(16))) char smem[LDS_BYTES];
    const int tid = threadIdx.x;
    const int lane = tid & 63;
    const int wave = __builtin_amdgcn_readfirstlane(tid >> 6);
    const int r32 = lane & 31, h = lane >> 5;
    const unsigned long long t0 = __builtin_amdgcn_s_memrealtime();

    const int d = (int)blockIdx.x;
    const int grp = d & 7;
    const int strip = grp * GW + (d >> 3);
    const int pos = ((d >> 3) + ROT * rank) & (GW - 1);

    const int ph = (d + wave) % 7;
    auto rot = [&](int i) { int e = i + ph; return i == 7 ? 7 : (e >= 7 ? e - 7 : e); };

    const bool helper = wave >= 8;
    const bool xpusher = pos >= XP0 && pos < XP0 + XPN;
    const int pi = (grp * XPN + pos - XP0) * 8 + wave;
    v4i xin[NR];
    if (xpusher && !helper) {
        const unsigned char* xp = xs + (size_t)(((pi >> 2) & 15) * 32 + r32) * KS + (pi >> 6) * BK + ((pi & 3) * 2 + h) * 16;
#pragma unroll
        for (int i = 0; i < NR; ++i)
            xin[i] = *reinterpret_cast<const v4i*>(xp + (size_t)((rank + 1 + rot(i)) & 7) * MS * KS);
    }
    unsigned wv[16];
    if (!helper) {
        const unsigned char* wp = wsrc + (size_t)((tid >> 4) * 16) * GN + strip * BN + (tid & 15) * 4;
#pragma unroll
        for (int j = 0; j < 16; ++j) wv[j] = *reinterpret_cast<const unsigned*>(wp + (size_t)j * GN);
    }

    void* ws_me = ws_all[rank];
    gchar* dws[NR];
#pragma unroll
    for (int i = 0; i < NR; ++i) dws[i] = gptr(ws_all[(rank + 1 + rot(i)) & 7]);
    gchar* sigb = dws[0];
#pragma unroll
    for (int i = 1; i < NR; ++i) sigb = lane == i ? dws[i] : sigb;

    unsigned call;
    {
        unsigned v;
        static_assert(L_DONE == 0, "counter offset");
        asm volatile("s_load_dword %0, %1, 0x0\n\ts_waitcnt lgkmcnt(0)" : "=s"(v) : "s"(ws_me));
        call = v / NWG + 1u;
    }
    const unsigned par_off = (unsigned)CTRL_BYTES + (call & 1u) * (unsigned)PAR_BYTES;
    const unsigned lds0 = (unsigned)(size_t)(__attribute__((address_space(3))) char*)smem;
    const unsigned fline = (L_FLAG + (call & 1u) * NWG + strip) * 128;
    TS(0);

    if (helper) {
        __builtin_amdgcn_s_setprio(3);
        if (wave == 8 && lane < 36) *reinterpret_cast<unsigned*>(smem + S_PUB + lane * 4) = 0u;
        lds_barrier();
        lds_barrier();
        lds_barrier();
        if (wave == 8) {
            const unsigned woff = lane < 32 ? fline + lane * 4 : lane < 40 ? (L_XCNT + lane - 32) * 128 : L_TMO * 128;
            const unsigned thr = lane < 32 ? call << 1 : lane < 40 ? 8u * XPN * call : call;
            const gchar* wg = gptr(ws_me) + par_off + X_BYTES + (size_t)strip * NR * KBS * REC;
            unsigned long long watch = 0x1ffffffffffull;
            unsigned taken = 15u << (4 * rank), q = 0;
            ld4_lds(gptr(ws_me), woff, lds0 + S_SNAP, watch);
#pragma unroll 1
            while (q < 32u) {
                WAIT_VM(0);
                const unsigned pv = lds_get(lds0 + S_SNAP + lane * 4);
                const unsigned used = __builtin_amdgcn_readfirstlane(lds_get(lds0 + S_CNT)) >> 3;
                const unsigned long long ok = __ballot((int)(pv - thr) >= 0);
                const unsigned pl = (unsigned)__ballot(pv & 1u);
                const bool late = __builtin_amdgcn_s_memrealtime() - t0 > DEADLINE_TICKS;
                if (late || ((ok >> 40) & 1u)) {
                    if (late && lane == 0)
                        __hip_atomic_store(ctrl_word(ws_me, L_TMO), call, __ATOMIC_RELAXED, __HIP_MEMORY_SCOPE_SYSTEM);
                    lds_put(lds0 + S_PUB + q * 4, C_DEAD);
                    break;
                }
                if (q == 0u && ((ok >> 32) & 0xffu) == 0xffu) {
                    __builtin_amdgcn_fence(__ATOMIC_ACQUIRE, "");
                    WAIT_VM(0);
                    if (lane < KBS) *reinterpret_cast<volatile unsigned*>(smem + S_PUB + lane * 4) = C_ITEM | C_OWN | (unsigned)(4 * rank + lane);
                    q = KBS;
                    watch = 0x100ffffffffull;
                }
                unsigned rem = q ? (unsigned)ok & ~taken : 0u;
                unsigned n = 0, code[4] = {0u, 0u, 0u, 0u}, anyp = 0;
#pragma unroll
                for (int i = 0; i < 4; ++i) {
                    if (rem && q + i < used + NS) {
                        const unsigned it = __builtin_ctz(rem);
                        rem &= rem - 1u;
                        taken |= 1u << it;
                        const gchar* rp = wg + it * (unsigned)REC;
                        const unsigned sl = lds0 + S_SLOT + ((q + i) & (NS - 1)) * REC;
                        ld4k_lds(rp, lane * 16, sl);
                        const unsigned p = (pl >> it) & 1u;
                        if (p) ld4k_lds(rp + 4096, lane * 16, sl + 4096);
                        else ld2k_lds(rp + 4096, lane * 16, sl + 4096);
                        anyp |= p;
                        code[i] = C_ITEM | it | p * C_PLAIN;
                        n = i + 1;
                    } else {
                        rem = 0u;
                    }
                }
                ld4_lds(gptr(ws_me), woff, lds0 + S_SNAP, watch);
                if (anyp) WAIT_VM(1);
#pragma unroll
                for (int i = 0; i < 4; ++i) {
                    if (i < (int)n) {
                        const unsigned left = n - 1u - i;
                        if (left == 3u) { WAIT_VM(19); } else if (left == 2u) { WAIT_VM(13); } else if (left == 1u) { WAIT_VM(7); } else { WAIT_VM(1); }
                        lds_put(lds0 + S_PUB + (q + i) * 4, code[i]);
                    }
                }
                q += n;
            }
            WAIT_VM(0);
            return;
        }
        const int sw_ = wave - 9;
        if (pos * 8 + grp >= NSLOT) {
            if (sw_ == 0) {
                const unsigned thr = (unsigned)NWG * (call - 1u) + (unsigned)(pos * 8 + grp - NSLOT + 1);
                unsigned go = 1u;
#pragma unroll 1
                for (;;) {
                    const int dist = (int)(thr - __builtin_amdgcn_readfirstlane(
                        __hip_atomic_load(ctrl_word(ws_me, L_SREL), __ATOMIC_RELAXED, __HIP_MEMORY_SCOPE_SYSTEM)));
                    if (dist <= 0) break;
                    if (__builtin_amdgcn_s_memrealtime() - t0 > DEADLINE_TICKS) {
                        if (lane == 0)
                            __hip_atomic_store(ctrl_word(ws_me, L_TMO), call, __ATOMIC_RELAXED, __HIP_MEMORY_SCOPE_SYSTEM);
                        go = 2u;
                        break;
                    }
                    if (dist > 24) __builtin_amdgcn_s_sleep(100);
                    else if (dist > 6) __builtin_amdgcn_s_sleep(16);
                }
                lds_put(lds0 + S_GO, go);
                if (go != 1u) return;
            } else {
                unsigned go = 0u;
#pragma unroll 1
                for (unsigned spin = 0; spin < SPIN_MAX; ++spin) {
                    go = __builtin_amdgcn_readfirstlane(lds_get(lds0 + S_GO));
                    if (go) break;
                    __builtin_amdgcn_s_sleep(2);
                }
                if (go != 1u) return;
            }
        }
        TSC(tid == 576, 2);
        {
            const unsigned slot = par_off + (unsigned)X_BYTES + (strip * NR + rank) * KBS * REC + lane * 16;
            gu32* flag = (gu32*)(sigb + fline + rank * 16);
            const unsigned* s_len = reinterpret_cast<const unsigned*>(smem + S_MISC + 192);
            auto put = [&](int r, unsigned len, int c0, int c1) {
                const char* src = smem + (len == (unsigned)REC ? S_RAW + r * REC : S_WIRE + r * 6144) + lane * 16;
#pragma unroll 1
                for (int c = c0; c < c1; ++c) {
                    if ((unsigned)(c * 1024 + lane * 16) < len) {
                        const v4i v = *reinterpret_cast<const v4i*>(src + c * 1024);
#pragma unroll
                        for (int i = 0; i < 7; ++i) st16_wt(dws[i], slot + r * REC + c * 1024, v);
                    }
                }
            };
            const int ra = sw_, rb = sw_ + 2;
            const unsigned la = __builtin_amdgcn_readfirstlane(s_len[ra]), lb = __builtin_amdgcn_readfirstlane(s_len[rb]);
            put(ra, la, 0, la == (unsigned)REC ? 8 : 6);
            WAIT_VM(42);
            put(rb, lb, 0, 3);
            WAIT_VM(21);
            if (lane < NR - 1)
                __hip_atomic_store(flag + ra, (call << 1) | (la == (unsigned)REC), __ATOMIC_RELAXED, __HIP_MEMORY_SCOPE_SYSTEM);
            if (tid == 576)
                __hip_atomic_fetch_add(ctrl_word(ws_me, L_SREL), 1u, __ATOMIC_RELAXED, __HIP_MEMORY_SCOPE_AGENT);
            put(rb, lb, 3, lb == (unsigned)REC ? 8 : 6);
            WAIT_VM(0);
            if (lane < NR - 1)
                __hip_atomic_store(flag + rb, (call << 1) | (lb == (unsigned)REC), __ATOMIC_RELAXED, __HIP_MEMORY_SCOPE_SYSTEM);
        }
        TSC(tid == 576, 3);
        return;
    }

    asm volatile("" :: "v"(wv[15]), "v"(wv[14]), "v"(wv[13]), "v"(wv[12]), "v"(wv[11]), "v"(wv[10]), "v"(wv[9]), "v"(wv[8]),
                       "v"(wv[7]), "v"(wv[6]), "v"(wv[5]), "v"(wv[4]), "v"(wv[3]), "v"(wv[2]), "v"(wv[1]), "v"(wv[0]));
    if (xpusher) {
#pragma unroll
        for (int i = 0; i < NR; ++i)
            st16_wt(dws[i], par_off + rank * KBS * XITEM + pi * 1024 + lane * 16, xin[i]);
    }

    unsigned* s_sel = reinterpret_cast<unsigned*>(smem + S_MISC);
    unsigned* s_tot = s_sel + 16;
    {
        const int kc = tid >> 4, n0 = (tid & 15) * 4;
        char* dst = smem + S_RAW + (kc >> 3) * REC + (((n0 >> 5) * 4 + ((kc >> 1) & 3)) * 64 + (kc & 1) * 32 + (n0 & 31)) * 16;
        v4u o[4];
#pragma unroll
        for (int q = 0; q < 4; ++q) {
            const unsigned a0 = wv[4 * q], a1 = wv[4 * q + 1], a2 = wv[4 * q + 2], a3 = wv[4 * q + 3];
            const unsigned u0 = __builtin_amdgcn_perm(a1, a0, 0x05010400u);
            const unsigned u1 = __builtin_amdgcn_perm(a1, a0, 0x07030602u);
            const unsigned u2 = __builtin_amdgcn_perm(a3, a2, 0x05010400u);
            const unsigned u3 = __builtin_amdgcn_perm(a3, a2, 0x07030602u);
            o[0][q] = __builtin_amdgcn_perm(u2, u0, 0x05040100u);
            o[1][q] = __builtin_amdgcn_perm(u2, u0, 0x07060302u);
            o[2][q] = __builtin_amdgcn_perm(u3, u1, 0x05040100u);
            o[3][q] = __builtin_amdgcn_perm(u3, u1, 0x07060302u);
        }
#pragma unroll
        for (int i = 0; i < 4; ++i) *reinterpret_cast<v4u*>(dst + i * 16) = o[i];
        if (tid < 256) *reinterpret_cast<v4u*>(smem + S_WIRE + (tid >> 6) * 6144 + RVAR + lane * 16) = v4u{0u, 0u, 0u, 0u};
        if (tid < 16) {
            unsigned sel = 0x0c0c0c0cu, n = 0;
#pragma unroll
            for (int b = 0; b < 4; ++b)
                if ((tid >> b) & 1) { sel = bfi(0xffu << (8 * n), (unsigned)b << (8 * n), sel); ++n; }
            s_sel[tid] = sel;
        }
    }
    lds_barrier();
    {
        unsigned bits[KBS], m2[KBS], sc[KBS];
#pragma unroll
        for (int kb = 0; kb < KBS; ++kb) {
            const v4u c = *reinterpret_cast<const v4u*>(smem + S_RAW + kb * REC + tid * 16);
            unsigned S[4], G = 0, n1 = 0;
            m2[kb] = 0;
            bits[kb] = 0;
#pragma unroll
            for (int q = 0; q < 4; ++q) {
                unsigned t = ((c[q] & 0x7f7f7f7fu) + (128u - 41u) * 0x01010101u) & 0x7f7f7f7fu;
                const unsigned fl = (t + (128u - 27u) * 0x01010101u) & 0x80808080u;
                t = bfi((fl << 1) - (fl >> 7), 0x1b1b1b1bu, t);
                const unsigned e = __builtin_amdgcn_perm(0x008c0804u, 0x009cacbcu, (t >> 2) & 0x07070707u);
                S[q] = (t & 0x03030303u) | (e & 0x0c0c0c0cu);
                const unsigned F = (e >> 7) & 0x01010101u;
                const unsigned sel = s_sel[__builtin_amdgcn_udot4(F, 0x08040201u, 0u, false)];
                bits[kb] |= __builtin_amdgcn_udot4(__builtin_amdgcn_perm(0u, (e >> 4) & 0x03030303u, sel), 0x40100401u, 0u, false) << (2u * n1);
                n1 += __builtin_popcount(F);
                m2[kb] |= fl >> (7 - q);
                G |= (c[q] & 0x80808080u) >> (7 - q);
            }
            char* rec = smem + S_WIRE + kb * 6144;
            *reinterpret_cast<v2u*>(rec + tid * 8) = v2u{S[0] | (S[1] << 4), S[2] | (S[3] << 4)};
            *reinterpret_cast<unsigned short*>(rec + 4096 + tid * 2) =
                (unsigned short)__builtin_amdgcn_perm(0u, G | (G >> 4), 0x0c0c0200u);
            const unsigned cnt = n1 | ((unsigned)__builtin_popcount(m2[kb]) << 16);
            const unsigned in = wave_scan(cnt);
            if (lane == 63) s_tot[kb * 8 + wave] = in;
            sc[kb] = in - cnt;
        }
        lds_barrier();
#pragma unroll
        for (int kb = 0; kb < KBS; ++kb) {
            unsigned base = 0, tot = 0;
#pragma unroll
            for (int w = 0; w < 8; ++w) {
                const unsigned v = __builtin_amdgcn_readfirstlane(s_tot[kb * 8 + w]);
                tot += v;
                if (w < wave) base += v;
            }
            const unsigned o2 = 32u + (((tot & 0xffffu) + 15u) >> 4) * 4u;
            const unsigned rlen = RVAR + o2 + (tot >> 16);
            if (tid == 0) s_tot[32 + kb] = rlen > (unsigned)CREC ? (unsigned)REC : (rlen + 63u) & ~63u;
            if (rlen > (unsigned)CREC) continue;
            base += o2 << 16;
            char* rec = smem + S_WIRE + kb * 6144;
            if (lane == 0) *reinterpret_cast<unsigned*>(rec + RVAR + wave * 4) = base;
            const unsigned e0 = (base + sc[kb]) & 0xffffu;
            const unsigned long long pk = (unsigned long long)bits[kb] << ((e0 & 15u) * 2u);
            unsigned* p = reinterpret_cast<unsigned*>(rec + RS1 + (e0 >> 4) * 4u);
            __hip_atomic_fetch_or(p, (unsigned)pk, __ATOMIC_RELAXED, __HIP_MEMORY_SCOPE_WORKGROUP);
            __hip_atomic_fetch_or(p + 1, (unsigned)(pk >> 32), __ATOMIC_RELAXED, __HIP_MEMORY_SCOPE_WORKGROUP);
            unsigned m = m2[kb], i2 = RVAR + ((base + sc[kb]) >> 16);
            while (m) {
                const unsigned bit = __builtin_ctz(m);
                m &= m - 1u;
                rec[i2++] = smem[S_RAW + kb * REC + tid * 16 + (bit & 3u) * 4 + (bit >> 3)];
            }
        }
    }
    if (xpusher) WAIT_VM(0);
    lds_barrier();
    if (xpusher && tid < NR)
        __hip_atomic_fetch_add((gu32*)(sigb + (L_XCNT + rank) * 128), 1u, __ATOMIC_RELAXED, __HIP_MEMORY_SCOPE_SYSTEM);
    TS(1);

    v16f acc[2][2];
#pragma unroll
    for (int a = 0; a < 2; ++a)
#pragma unroll
        for (int b = 0; b < 2; ++b)
#pragma unroll
            for (int i = 0; i < 16; ++i) acc[a][b][i] = 0.f;

    const gchar* xg = gptr(ws_me) + par_off + wave * 8192 + lane * 16;
    bool dead = false;
#pragma unroll 1
    for (unsigned q = 0; q < 32u; ++q) {
        unsigned c = 0u, spin = 0u;
        TF(0);
#pragma unroll 1
        for (;;) {
            c = __builtin_amdgcn_readfirstlane(lds_get(lds0 + S_PUB + q * 4));
            if (c) break;
            if (++spin > SPIN_MAX) { c = C_DEAD; break; }
            __builtin_amdgcn_s_sleep(1);
        }
        if (c & C_DEAD) { dead = true; break; }
        TF(1);
        v4i xf[8];
        {
            const gchar* xp = xg + (c & 31u) * (unsigned)XITEM;
#pragma unroll
            for (int i = 0; i < 8; ++i) xf[i] = *(const gv4i*)(xp + i * 1024);
        }
        asm volatile("" ::: "memory");
        TF(2);
        if (!(c & C_OWN)) {
            const char* rec = smem + S_SLOT + (q & (NS - 1)) * REC;
            char* dec = smem + S_DEC + (q & 1u) * REC + tid * 16;
            if (c & C_PLAIN) {
                *reinterpret_cast<v4u*>(dec) = *reinterpret_cast<const v4u*>(rec + tid * 16);
            } else {
                const v2u nb = *reinterpret_cast<const v2u*>(rec + tid * 8);
                const unsigned wsg = *reinterpret_cast<const unsigned short*>(rec + 4096 + tid * 2);
                const unsigned hd = *reinterpret_cast<const unsigned*>(rec + RVAR + wave * 4);
                unsigned S[4], sel[4], cum[4], n1 = 0;
                S[0] = nb[0] & 0x0f0f0f0fu; S[1] = (nb[0] >> 4) & 0x0f0f0f0fu;
                S[2] = nb[1] & 0x0f0f0f0fu; S[3] = (nb[1] >> 4) & 0x0f0f0f0fu;
#pragma unroll
                for (int k = 0; k < 4; ++k) {
                    const unsigned F = (S[k] >> 3) & (S[k] >> 2) & 0x01010101u;
                    const unsigned p1 = F + (F << 8), p2 = p1 + (p1 << 16);
                    sel[k] = (p2 << 8) | ((F ^ 0x01010101u) << 2);
                    cum[k] = n1;
                    n1 += p2 >> 24;
                }
                const unsigned e0 = (hd + wave_scan(n1) - n1) & 4095u;
                const unsigned* sp = reinterpret_cast<const unsigned*>(rec + RS1 + (e0 >> 4) * 4u);
                const unsigned bits = __builtin_amdgcn_alignbit(sp[1], sp[0], (e0 & 15u) * 2u);
                const unsigned X = __builtin_amdgcn_perm(0u, wsg, 0x01010000u);
                const unsigned G = bfi(0x00ff00ffu, X, X >> 4);
                v4u o;
                unsigned m = 0;
#pragma unroll
                for (int k = 0; k < 4; ++k) {
                    unsigned b4 = __builtin_amdgcn_ubfe(bits, 2u * cum[k], 8u);
                    b4 = (b4 | (b4 << 12)) & 0x000f000fu;
                    b4 = (b4 | (b4 << 6)) & 0x03030303u;
                    const unsigned cc = S[k] + __builtin_amdgcn_perm(0u, 0x1d212535u, __builtin_amdgcn_perm(0u, b4, sel[k]));
                    m |= ((cc >> 6) & (cc >> 2) & 0x01010101u) << k;
                    o[k] = cc | ((G << (7 - k)) & 0x80808080u);
                }
                *reinterpret_cast<v4u*>(dec) = o;
                const unsigned n2 = __builtin_popcount(m);
                unsigned i2 = (hd >> 16) + wave_scan(n2) - n2;
                while (m) {
                    const unsigned bit = __builtin_ctz(m);
                    m &= m - 1u;
                    dec[(bit & 3u) * 4 + (bit >> 3)] = rec[RVAR + (i2++ & 1023u)];
                }
            }
        }
        TF(3);
        lds_inc(lds0 + S_CNT);
#pragma unroll 1
        for (;;) {
            if (__builtin_amdgcn_readfirstlane(lds_get(lds0 + S_CNT)) >= 8u * q + 8u) break;
            if (++spin > SPIN_MAX) { dead = true; break; }
        }
        if (dead) break;
        TF(4);
        {
            const char* lb = smem + ((c & C_OWN) ? S_RAW + (c & 3u) * REC : S_DEC + (q & 1u) * REC) + lane * 16;
#pragma unroll
            for (int ks = 0; ks < 2; ++ks) {
                v8i bf[2];
#pragma unroll
                for (int i = 0; i < 2; ++i) {
                    const v4i b0 = *reinterpret_cast<const v4i*>(lb + (i * 4 + ks * 2) * 1024);
                    const v4i b1 = *reinterpret_cast<const v4i*>(lb + (i * 4 + ks * 2 + 1) * 1024);
                    bf[i] = __builtin_shufflevector(b0, b1, 0, 1, 2, 3, 4, 5, 6, 7);
                }
#pragma unroll
                for (int a = 0; a < 2; ++a) {
                    const v8i xa = __builtin_shufflevector(xf[a * 4 + ks * 2], xf[a * 4 + ks * 2 + 1], 0, 1, 2, 3, 4, 5, 6, 7);
#pragma unroll
                    for (int b = 0; b < 2; ++b)
                        acc[a][b] = __builtin_amdgcn_mfma_scale_f32_32x32x64_f8f6f4(xa, bf[b], acc[a][b], 0, 1, 0, 127, 0, 127);
                }
            }
        }
        TF(5);
        TS(16 + q);
    }
    TS(4);

    {
        const float scale = dead ? __builtin_nanf("") : sx[0] * sw[0];
        float* op = out + (size_t)(wave * 64 + 4 * h) * GN + strip * BN + r32;
#pragma unroll
        for (int a = 0; a < 2; ++a)
#pragma unroll
            for (int i = 0; i < 16; ++i)
#pragma unroll
                for (int b = 0; b < 2; ++b)
                    op[(size_t)(a * 32 + (i & 3) + 8 * (i >> 2)) * GN + b * 32] = acc[a][b][i] * scale;
    }
    if (tid == 63)
        __hip_atomic_fetch_add(ctrl_word(ws_me, L_DONE), 1u, __ATOMIC_RELAXED, __HIP_MEMORY_SCOPE_AGENT);
    TS(5);
}

}

extern "C" void kernel_launch(int rank, int nranks, ncclComm_t comm,
                              hipStream_t stream,
                              void* const* d_in, const int* in_sizes, int n_in,
                              void* d_out, int out_size,
                              const int* in_dtypes, int out_dtype,
                              void* const* d_out_all,
                              void* const* d_ws_all, size_t ws_bytes) {
    (void)comm; (void)in_dtypes; (void)out_dtype; (void)d_out_all;
    if (nranks != NR || (unsigned)rank >= (unsigned)NR || n_in < 4 || ws_bytes < WS_NEED ||
        in_sizes[0] != GM * KS || in_sizes[1] != KS * GN || out_size != MS * GN) {
        fprintf(stderr, "kernel_launch: unexpected problem geometry\n");
        return;
    }
    hipLaunchKernelGGL(fused_gemm_kernel, dim3(NWG), dim3(NTHREADS), 0, stream,
                       (const unsigned char*)d_in[0], (const unsigned char*)d_in[1],
                       (const float*)d_in[2], (const float*)d_in[3], (float*)d_out, d_ws_all, rank);
    HIP_CHECK(hipGetLastError());
}
```

```cpp
#include <hip/hip_runtime.h>
#include <rccl/rccl.h>
#include <cstdio>
#include <cstdint>

#define HIP_CHECK(expr) do { hipError_t _e = (expr); if (_e != hipSuccess) { \
    fprintf(stderr, "HIP error %s at %s:%d\n", hipGetErrorString(_e), __FILE__, __LINE__); \
    return; } } while (0)

typedef int v8i __attribute__((ext_vector_type(8)));
typedef int v4i __attribute__((ext_vector_type(4)));
typedef unsigned v4u __attribute__((ext_vector_type(4)));
typedef unsigned v2u __attribute__((ext_vector_type(2)));
typedef float v16f __attribute__((ext_vector_type(16)));
typedef unsigned __attribute__((aligned(1))) u32u;
#define GAS __attribute__((address_space(1)))
typedef GAS char gchar;
typedef GAS unsigned gu32;
typedef GAS v4i gv4i;

namespace {

constexpr int NR = 8;
constexpr int GM = 4096, GK = 4096, GN = 16384;
constexpr int KS = GK / NR;
constexpr int MS = GM / NR;
constexpr int BN = 64, BK = 128;
constexpr int KBS = KS / BK;
constexpr int NWG = GN / BN;
constexpr int NTHREADS = 640;
constexpr int GW = NWG / 8;

constexpr size_t CTRL_BYTES = 1u << 20;
constexpr int XITEM = MS * BK;
constexpr int REC = BK * BN;
constexpr size_t X_BYTES = (size_t)NR * KBS * XITEM;
constexpr size_t PAR_BYTES = X_BYTES + (size_t)NWG * NR * KBS * REC;
constexpr size_t WS_NEED = CTRL_BYTES + 2 * PAR_BYTES;
constexpr int L_DONE = 0;
constexpr int L_TMO = 1;
constexpr int L_TICK = 4;
constexpr int L_XCNT = 16;
constexpr int L_GO = 256;
constexpr int L_FLAG = 512;
constexpr unsigned long long DEADLINE_TICKS = 100000000ull;

#define CREC 6144
constexpr int RVAR = 5120, RS1 = RVAR + 32;
static_assert(CREC <= 6144 && CREC >= RS1, "record size");

constexpr int S_RAW = 0;
constexpr int S_WIRE = S_RAW + KBS * REC;
constexpr int S_DEC = S_WIRE + KBS * 6144;
constexpr int NS = 8;
constexpr int S_SLOT = S_DEC + 2 * REC;
constexpr int S_SNAP = S_SLOT + NS * REC;
constexpr int S_MISC = S_SNAP + 256;
constexpr int S_PUB = S_MISC + 256;
constexpr int S_CNT = S_PUB + 128;
constexpr int S_GO = S_CNT + 4;
constexpr int LDS_BYTES = S_CNT + 16;
static_assert(LDS_BYTES > 80 * 1024, "LDS");

__device__ __forceinline__ gchar* gptr(void* p) { return (gchar*)p; }
__device__ __forceinline__ gu32* ctrl_word(void* ws, int line) {
    return (gu32*)(gptr(ws) + (size_t)line * 128);
}
__device__ __forceinline__ unsigned bfi(unsigned m, unsigned a, unsigned b) { return (a & m) | (b & ~m); }
__device__ __forceinline__ unsigned wave_scan(unsigned s) {
    s += __builtin_amdgcn_update_dpp(0u, s, 0x111, 0xf, 0xf, false);
    s += __builtin_amdgcn_update_dpp(0u, s, 0x112, 0xf, 0xf, false);
    s += __builtin_amdgcn_update_dpp(0u, s, 0x114, 0xf, 0xf, false);
    s += __builtin_amdgcn_update_dpp(0u, s, 0x118, 0xf, 0xf, false);
    s += __builtin_amdgcn_update_dpp(0u, s, 0x142, 0xa, 0xf, false);
    s += __builtin_amdgcn_update_dpp(0u, s, 0x143, 0xc, 0xf, false);
    return s;
}

#define WAIT_VM_(n) asm volatile("s_waitcnt vmcnt(" #n ")" ::: "memory")
#define WAIT_VM(n) WAIT_VM_(n)
#define NCH 65
#define LB 8
#define STAG 2
#define EARLY 0
#define PNEAR 3
#define PNAP 24
#define TG 2
#define NT 10
#define ABL 0
#define OST 2
#define ROTV 32
#define TAU 640u
#define OFF 570u
#define LEAD 200u
#define WIN 4
#define GAP 8
#define WEND 0
#define SWZ 0
#define LATE 21
#define LGRAN 64u
#define ORDB 4
#define PSL 2
#define TSC(c, e) do { } while (0)
#define TS(e) do { } while (0)
#define TM(v) do { } while (0)
#define TF(e) do { } while (0)

__device__ __forceinline__ void st16_wt(gchar* base, unsigned off, v4i v) {
    asm volatile("s_nop 4\n\tglobal_store_dwordx4 %0, %1, %2 sc0 sc1\n\ts_nop 1" :: "v"(off), "v"(v), "s"(base) : "memory");
}
__device__ __forceinline__ void ld4k_lds(const gchar* base, unsigned off, unsigned lds) {
    asm volatile("s_mov_b32 m0, %2\n\ts_nop 4\n\t"
                 "global_load_lds_dwordx4 %0, %1 sc0 sc1\n\t"
                 "global_load_lds_dwordx4 %0, %1 offset:1024 sc0 sc1\n\t"
                 "global_load_lds_dwordx4 %0, %1 offset:2048 sc0 sc1\n\t"
                 "global_load_lds_dwordx4 %0, %1 offset:3072 sc0 sc1"
                 :: "v"(off), "s"(base), "s"(lds) : "memory");
}
__device__ __forceinline__ void ld2k_lds(const gchar* base, unsigned off, unsigned lds) {
    asm volatile("s_mov_b32 m0, %2\n\ts_nop 4\n\t"
                 "global_load_lds_dwordx4 %0, %1 sc0 sc1\n\t"
                 "global_load_lds_dwordx4 %0, %1 offset:1024 sc0 sc1"
                 :: "v"(off), "s"(base), "s"(lds) : "memory");
}
__device__ __forceinline__ void ld4_lds(const gchar* base, unsigned off, unsigned lds, unsigned long long lanes) {
    asm volatile("s_mov_b32 m0, %2\n\ts_mov_b64 exec, %3\n\ts_nop 3\n\t"
                 "global_load_lds_dword %0, %1 sc0 sc1\n\ts_mov_b64 exec, -1"
                 :: "v"(off), "s"(base), "s"(lds), "s"(lanes) : "memory");
}
__device__ __forceinline__ unsigned lds_get(unsigned addr) {
    unsigned v;
    asm volatile("ds_read_b32 %0, %1\n\ts_waitcnt lgkmcnt(0)" : "=v"(v) : "v"(addr) : "memory");
    return v;
}
__device__ __forceinline__ void lds_put(unsigned addr, unsigned v) {
    asm volatile("s_mov_b64 exec, 1\n\tds_write_b32 %0, %1\n\ts_mov_b64 exec, -1" :: "v"(addr), "v"(v) : "memory");
}
__device__ __forceinline__ void lds_inc(unsigned addr) {
    asm volatile("s_mov_b64 exec, 1\n\tds_add_u32 %0, %1\n\ts_mov_b64 exec, -1" :: "v"(addr), "v"(1u) : "memory");
}
constexpr unsigned SPIN_MAX = 1u << 25;
__device__ __forceinline__ void lds_barrier() {
    asm volatile("s_waitcnt lgkmcnt(0)\n\ts_barrier" ::: "memory");
}

enum : unsigned { C_ITEM = 0x100u, C_PLAIN = 0x20u, C_OWN = 0x40u, C_DEAD = 0x200u, C_RDY = 0x400u };

__global__ __launch_bounds__(NTHREADS)
void fused_gemm_kernel(const unsigned char* __restrict__ xs,
                       const unsigned char* __restrict__ wsrc,
                       const float* __restrict__ sx, const float* __restrict__ sw,
                       float* __restrict__ out, void* const* __restrict__ ws_all, int rank) {
    __shared__ __attribute__((aligned(16))) char smem[LDS_BYTES];
    const int tid = threadIdx.x;
    const int lane = tid & 63;
    const int wave = __builtin_amdgcn_readfirstlane(tid >> 6);
    const int r32 = lane & 31, h = lane >> 5;
    const unsigned long long t0 = __builtin_amdgcn_s_memrealtime();

    const int d = (int)blockIdx.x;
    const int grp = d & 7;
    const int strip = grp * GW + (d >> 3);
    constexpr int BS = 1 << LB;
    const int dd = (d >> LB << LB) | ((d - ROTV * rank) & (BS - 1));
    auto slot_of = [&](int src, int r) { return (unsigned)((strip * NR + src) * KBS + ((SWZ & 1) ? (r + (d >> 1)) & 3 : r)) * (unsigned)REC; };
    const unsigned sx4 = (SWZ & 2) ? (unsigned)(d & 1) << 12 : 0u;
    const int ph = (d + wave) % 7;
    auto rot = [&](int i) { int e = i + ph; return e >= 7 ? e - 7 : e; };

    const unsigned lds0 = (unsigned)(size_t)(__attribute__((address_space(3))) char*)smem;
    void* ws_me;
    unsigned call, par_off;
    auto whoami = [&]() {
        ws_me = ws_all[rank];
        unsigned v;
        static_assert(L_DONE == 0, "counter offset");
        asm volatile("s_load_dword %0, %1, 0x0\n\ts_waitcnt lgkmcnt(0)" : "=s"(v) : "s"(ws_me));
        call = v / NWG + 1u;
        par_off = (unsigned)CTRL_BYTES + (call & 1u) * (unsigned)PAR_BYTES;
    };

    if (wave >= 8) {
        __builtin_amdgcn_s_setprio(3);
        const bool xpusher = dd >= NWG / 2;
        const int xb = ((dd & 7) << 2 | (dd >> 6 & 1) << 1 | (wave & 1)) * 2;
        const int xdst = (rank + ((dd >> 3) ^ (wave << 2))) & 7;
        v4i xin[8];
        if (xpusher) {
            const unsigned char* xp = xs + (xdst * MS + (xb & 15) * 32 + r32) * KS + (xb >> 4) * BK + h * 16;
#pragma unroll
            for (int i = 0; i < 8; ++i) xin[i] = *reinterpret_cast<const v4i*>(xp + (i >> 2) * 32 * KS + (i & 3) * 32);
        }
        gchar* xws = gptr(ws_all[xdst]);
        whoami();
        TM(tB);
        if (xpusher) {
            asm volatile("" :: "v"(xin[0]), "v"(xin[1]), "v"(xin[2]), "v"(xin[3]),
                               "v"(xin[4]), "v"(xin[5]), "v"(xin[6]), "v"(xin[7]));
#pragma unroll
            for (int i = 0; i < 8; ++i)
                st16_wt(xws, par_off + rank * KBS * XITEM + xb * 4096 + i * 1024 + lane * 16, xin[i]);
            WAIT_VM(0);
            if (lane == 0)
                __hip_atomic_fetch_add((gu32*)(xws + (L_XCNT + rank) * 128), 1u, __ATOMIC_RELAXED, __HIP_MEMORY_SCOPE_SYSTEM);
        }
        TM(tC);
        if (wave == 8) {
            const unsigned fline = (L_FLAG + (call & 1u) * NWG + strip) * 128;
            if (lane < 36) *reinterpret_cast<unsigned*>(smem + S_PUB + lane * 4) = 0u;
            lds_barrier();
            lds_barrier();
            lds_barrier();
            lds_barrier();
            lds_barrier();
            const unsigned woff = lane < 32 ? fline + lane * 4 : lane < 40 ? (L_XCNT + lane - 32) * 128 : L_TMO * 128;
            const unsigned thr = lane < 32 ? call << 1 : lane < 40 ? 32u * call : call;
            const gchar* wg = gptr(ws_me) + par_off + X_BYTES;
            const unsigned myslot = slot_of((lane >> 2) & 7, lane & 3);
            unsigned long long watch = 0x1ffffffffffull;
            unsigned taken = 15u << (4 * rank), q = 0;
            ld4_lds(gptr(ws_me), woff, lds0 + S_SNAP, watch);
#pragma unroll 1
            while (q < 32u) {
                WAIT_VM(0);
                const unsigned pv = lds_get(lds0 + S_SNAP + lane * 4);
                const unsigned used = __builtin_amdgcn_readfirstlane(lds_get(lds0 + S_CNT)) >> 3;
                const unsigned long long ok = __ballot((int)(pv - thr) >= 0);
                const unsigned pl = (unsigned)__ballot(pv & 1u);
                const bool late = __builtin_amdgcn_s_memrealtime() - t0 > DEADLINE_TICKS;
                if (late || ((ok >> 40) & 1u)) {
                    if (late && lane == 0)
                        __hip_atomic_store(ctrl_word(ws_me, L_TMO), call, __ATOMIC_RELAXED, __HIP_MEMORY_SCOPE_SYSTEM);
                    lds_put(lds0 + S_PUB + q * 4, C_DEAD);
                    break;
                }
                if (q == 0u && ((ok >> 32) & 0xffu) == 0xffu) {
                    __builtin_amdgcn_fence(__ATOMIC_ACQUIRE, "");
                    WAIT_VM(0);
                    if (lane < KBS) *reinterpret_cast<volatile unsigned*>(smem + S_PUB + lane * 4) = C_ITEM | C_RDY | C_OWN | (unsigned)(4 * rank + lane);
                    q = KBS;
                    TSC(lane == 0, 6);
                    WAIT_VM(0);
                    watch = 0x100ffffffffull;
                }
                unsigned rem = q ? (unsigned)ok & ~taken : 0u;
                unsigned n = 0, code[4] = {0u, 0u, 0u, 0u}, anyp = 0;
#pragma unroll
                for (int i = 0; i < 4; ++i) {
                    if (rem && q + i < used + NS) {
                        const unsigned it = __builtin_ctz(rem);
                        rem &= rem - 1u;
                        taken |= 1u << it;
                        const gchar* rp = wg + (unsigned)__builtin_amdgcn_readlane(myslot, (ABL & 2) ? 4 * rank : it);
                        const unsigned sl = lds0 + S_SLOT + ((q + i) & (NS - 1)) * REC;
                        if (!(ABL & 16)) ld4k_lds(rp + sx4, lane * 16, sl);
                        const unsigned p = (pl >> it) & 1u;
                        if (p) ld4k_lds(rp + (4096u ^ sx4), lane * 16, sl + 4096);
                        else if (!(ABL & 16)) ld2k_lds(rp + (4096u ^ sx4), lane * 16, sl + 4096);
                        anyp |= p;
                        code[i] = C_ITEM | it | p * C_PLAIN;
                        lds_put(lds0 + S_PUB + (q + i) * 4, code[i]);
                        n = i + 1;
                    } else {
                        rem = 0u;
                    }
                }
                ld4_lds(gptr(ws_me), woff, lds0 + S_SNAP, watch);
                if (anyp) WAIT_VM(1);
#pragma unroll
                for (int i = 0; i < 4; ++i) {
                    if (i < (int)n) {
                        const unsigned left = n - 1u - i;
                        if (left == 3u) { WAIT_VM(19); } else if (left == 2u) { WAIT_VM(13); } else if (left == 1u) { WAIT_VM(7); } else { WAIT_VM(1); }
                        lds_put(lds0 + S_PUB + (q + i) * 4, code[i] | C_RDY);
                    }
                }
                q += n;
            }
            WAIT_VM(0);
            return;
        }
        gchar* dws[NR - 1];
#pragma unroll
        for (int i = 0; i < NR - 1; ++i) dws[i] = gptr(ws_all[(rank + 1 + rot(i)) & 7]);
        gchar* sigb = dws[0];
#pragma unroll
        for (int i = 1; i < NR - 1; ++i) sigb = lane == i ? dws[i] : sigb;
        sigb += (L_FLAG + (call & 1u) * NWG + strip) * 128 + rank * KBS * 4;
        __builtin_amdgcn_s_waitcnt(0);
        lds_barrier();
        lds_barrier();
        lds_barrier();
#pragma unroll 1
        for (int r = 0; r < KBS; ++r) {
            const int seq = (dd >> LB << (LB + 2)) | (r << LB) | (dd & (BS - 1));
            if (r == (dd < (NT << TG) ? 1 : 0)) {
                lds_barrier();
                lds_barrier();
            }
            auto due_of = [&](int c) {
                int n = 0;
#pragma unroll
                for (int e = 0; e < 6; ++e) n += min(max(seq + (c - e) * GAP + (e < c ? 1 : 0), 0), NWG * KBS);
                return OFF + (unsigned)n * TAU / 384u;
            };
            const unsigned due = due_of(0);
            if (seq >= NCH) {
#pragma unroll 1
                while ((unsigned)(__builtin_amdgcn_s_memrealtime() - t0) + LEAD < due) __builtin_amdgcn_s_sleep(8);
#pragma unroll 1
                for (;;) {
                    const unsigned cv = __hip_atomic_load(ctrl_word(ws_me, L_GO + dd) + r, __ATOMIC_RELAXED, __HIP_MEMORY_SCOPE_SYSTEM);
                    if (__builtin_amdgcn_readfirstlane(cv) == call) break;
                    if (__builtin_amdgcn_s_memrealtime() - t0 > DEADLINE_TICKS) {
                        if (lane == 0)
                            __hip_atomic_store(ctrl_word(ws_me, L_TMO), call, __ATOMIC_RELAXED, __HIP_MEMORY_SCOPE_SYSTEM);
                        return;
                    }
                    __builtin_amdgcn_s_sleep(PSL);
                }
            }
            const unsigned slot = par_off + (unsigned)X_BYTES + slot_of(rank, r) + lane * 16;
            const unsigned len = __builtin_amdgcn_readfirstlane(*reinterpret_cast<const unsigned*>(smem + S_MISC + 192 + r * 4));
            const char* src = smem + (len == (unsigned)REC ? S_RAW + r * REC : S_WIRE + r * 6144) + lane * 16;
#pragma unroll 1
            for (unsigned c = 0; c < len; c += 1024u) {
                const unsigned duc = due_of(min((int)(c >> 10), 5));
#pragma unroll 1
                while ((unsigned)(__builtin_amdgcn_s_memrealtime() - t0) < duc) __builtin_amdgcn_s_sleep(1);
                if (seq + (int)(c >> 10) * GAP < NWG * KBS + WEND) WAIT_VM(WIN);
                if (c + lane * 16 < len) {
                    const v4i v = *reinterpret_cast<const v4i*>(src + c);
#pragma unroll
                    for (int i = 0; i < 7; ++i) st16_wt(dws[i], slot + (c ^ sx4), v);
                }
            }
            TM(tE);
            if (EARLY < 0) WAIT_VM(LATE);
            if (seq + NCH < NWG * KBS && lane == 0)
                __hip_atomic_store(ctrl_word(ws_me, L_GO + ((seq + NCH) & (NWG - 1))) + ((seq + NCH) >> 8), call, __ATOMIC_RELAXED, __HIP_MEMORY_SCOPE_SYSTEM);
            WAIT_VM(0);
            TM(tF);
            if (lane < NR - 1)
                __hip_atomic_store((gu32*)(sigb + r * 4), (call << 1) | (len == (unsigned)REC),
                                   __ATOMIC_RELAXED, __HIP_MEMORY_SCOPE_SYSTEM);
        }
        return;
    }

    unsigned wv[16];
    {
        const unsigned char* wp = wsrc + (size_t)((tid >> 4) * 16) * GN + strip * BN + (tid & 15) * 4;
#pragma unroll
        for (int j = 0; j < 16; ++j) wv[j] = *reinterpret_cast<const unsigned*>(wp + (size_t)j * GN);
    }
    whoami();
    TS(0);

    unsigned* s_sel = reinterpret_cast<unsigned*>(smem + S_MISC);
    unsigned* s_tot = s_sel + 16;
    {
        const int kc = tid >> 4, n0 = (tid & 15) * 4;
        char* dst = smem + S_RAW + (kc >> 3) * REC + (((n0 >> 5) * 4 + ((kc >> 1) & 3)) * 64 + (kc & 1) * 32 + (n0 & 31)) * 16;
        v4u o[4];
#pragma unroll
        for (int q = 0; q < 4; ++q) {
            const unsigned a0 = wv[4 * q], a1 = wv[4 * q + 1], a2 = wv[4 * q + 2], a3 = wv[4 * q + 3];
            const unsigned u0 = __builtin_amdgcn_perm(a1, a0, 0x05010400u);
            const unsigned u1 = __builtin_amdgcn_perm(a1, a0, 0x07030602u);
            const unsigned u2 = __builtin_amdgcn_perm(a3, a2, 0x05010400u);
            const unsigned u3 = __builtin_amdgcn_perm(a3, a2, 0x07030602u);
            o[0][q] = __builtin_amdgcn_perm(u2, u0, 0x05040100u);
            o[1][q] = __builtin_amdgcn_perm(u2, u0, 0x07060302u);
            o[2][q] = __builtin_amdgcn_perm(u3, u1, 0x05040100u);
            o[3][q] = __builtin_amdgcn_perm(u3, u1, 0x07060302u);
        }
#pragma unroll
        for (int i = 0; i < 4; ++i) *reinterpret_cast<v4u*>(dst + i * 16) = o[i];
        if (tid < 256) *reinterpret_cast<v4u*>(smem + S_WIRE + (tid >> 6) * 6144 + RVAR + lane * 16) = v4u{0u, 0u, 0u, 0u};
        if (tid < 16) {
            unsigned sel = 0x0c0c0c0cu, n = 0;
#pragma unroll
            for (int b = 0; b < 4; ++b)
                if ((tid >> b) & 1) { sel = bfi(0xffu << (8 * n), (unsigned)b << (8 * n), sel); ++n; }
            s_sel[tid] = sel;
            if (tid < KBS) s_tot[36 + tid] = 0u;
        }
    }
    lds_barrier();
#pragma unroll
    for (int part = 0; part < 2; ++part) {
        const int kb0 = part, kb1 = part ? KBS : 1;
        unsigned bits[KBS], m2[KBS], sc[KBS];
#pragma unroll
        for (int kb = kb0; kb < kb1; ++kb) {
            const v4u c = *reinterpret_cast<const v4u*>(smem + S_RAW + kb * REC + tid * 16);
            unsigned S[4], n1 = 0, mp = 0;
            m2[kb] = 0;
            bits[kb] = 0;
#pragma unroll
            for (int q = 0; q < 4; ++q) {
                unsigned t = ((c[q] & 0x7f7f7f7fu) + (128u - 41u) * 0x01010101u) & 0x7f7f7f7fu;
                const unsigned fl = (t + (128u - 28u) * 0x01010101u) & 0x80808080u;
                const unsigned l3 = (t + (128u - 98u) * 0x01010101u) & 0x80808080u;
                t = bfi((fl << 1) - (fl >> 7), 0x12121212u | (t & 0x01010101u) | (l3 >> 4), t);
                const unsigned e = __builtin_amdgcn_perm(0x00b00c08u, 0x048090a0u, (t >> 2) & 0x07070707u);
                S[q] = (t & 0x03030303u) | (e & 0x0c0c0c0cu);
                const unsigned F = (e >> 7) & 0x01010101u;
                const unsigned sel = s_sel[__builtin_amdgcn_udot4(F, 0x08040201u, 0u, false)];
                bits[kb] |= __builtin_amdgcn_udot4(__builtin_amdgcn_perm(0u, (e >> 4) & 0x03030303u, sel), 0x40100401u, 0u, false) << (2u * n1);
                n1 += __builtin_popcount(F);
                m2[kb] |= ((t + (128u - 26u) * 0x01010101u) & 0x80808080u) >> (4 - q);
                mp |= (fl & ~l3) >> (4 - q);
            }
            char* rec = smem + S_WIRE + kb * 6144;
            *reinterpret_cast<v2u*>(rec + tid * 8) = v2u{S[3] | (S[2] << 4), S[1] | (S[0] << 4)};
            const unsigned Y = (c[0] & 0x80808080u) | ((c[1] & 0x80808080u) >> 4) | ((c[2] & 0x80808080u) >> 2) | ((c[3] & 0x80808080u) >> 6);
            *reinterpret_cast<unsigned short*>(rec + 4096 + tid * 2) = (unsigned short)(Y | (Y >> 17));
            const unsigned n2 = __builtin_popcount(m2[kb]);
            if (n2 > 8u) __hip_atomic_fetch_add(s_tot + 36 + kb, 16u, __ATOMIC_RELAXED, __HIP_MEMORY_SCOPE_WORKGROUP);
            while (mp) {
                const unsigned bit = __builtin_ctz(mp);
                mp &= mp - 1u;
                const unsigned pos = tid * 16 + ((bit & 7u) - 3u) * 4 + (bit >> 3);
                const unsigned ix = __hip_atomic_fetch_add(s_tot + 36 + kb, 1u, __ATOMIC_RELAXED, __HIP_MEMORY_SCOPE_WORKGROUP);
                if (ix < 15u)
                    *reinterpret_cast<unsigned*>(rec + RS1 + ix * 4) = pos << 8 | (unsigned char)smem[S_RAW + kb * REC + pos];
            }
            const unsigned cnt = n1 | (n2 << 16);
            const unsigned in = wave_scan(cnt);
            if (lane == 63) s_tot[kb * 8 + wave] = in;
            sc[kb] = in - cnt;
        }
        lds_barrier();
#pragma unroll
        for (int kb = kb0; kb < kb1; ++kb) {
            unsigned base = 0, tot = 0;
#pragma unroll
            for (int w = 0; w < 8; ++w) {
                const unsigned v = __builtin_amdgcn_readfirstlane(s_tot[kb * 8 + w]);
                tot += v;
                if (w < wave) base += v;
            }
            const unsigned np = __builtin_amdgcn_readfirstlane(s_tot[36 + kb]);
            const unsigned o2 = 32u + np * 4u + (((tot & 0xffffu) + 15u) >> 4) * 4u;
            const unsigned rlen = np > 15u ? (unsigned)REC : RVAR + o2 + (((tot >> 16) + 1u) >> 1);
            if (tid == 0) s_tot[32 + kb] = rlen > (unsigned)CREC ? (unsigned)REC : (rlen + LGRAN - 1u) & ~(LGRAN - 1u);
            if (rlen > (unsigned)CREC) continue;
            base += (o2 << 17) + np * 16u;
            char* rec = smem + S_WIRE + kb * 6144;
            if (lane == 0) *reinterpret_cast<unsigned*>(rec + RVAR + wave * 4) = base | np << 12;
            const unsigned e0 = (base + sc[kb]) & 0xffffu;
            const unsigned long long pk = (unsigned long long)bits[kb] << ((e0 & 15u) * 2u);
            unsigned* p = reinterpret_cast<unsigned*>(rec + RS1 + (e0 >> 4) * 4u);
            __hip_atomic_fetch_or(p, (unsigned)pk, __ATOMIC_RELAXED, __HIP_MEMORY_SCOPE_WORKGROUP);
            __hip_atomic_fetch_or(p + 1, (unsigned)(pk >> 32), __ATOMIC_RELAXED, __HIP_MEMORY_SCOPE_WORKGROUP);
            unsigned m = m2[kb], i2 = (base + sc[kb]) >> 16;
            while (m) {
                const unsigned bit = __builtin_ctz(m);
                m &= m - 1u;
                const unsigned cb = (unsigned char)smem[S_RAW + kb * REC + tid * 16 + ((bit & 7u) - 3u) * 4 + (bit >> 3)] & 0x7fu;
                const unsigned v = cb > 42u ? 0u : (42u - cb) >> 1;
                __hip_atomic_fetch_or(reinterpret_cast<unsigned*>(rec + RVAR + (i2 >> 3) * 4u), v << ((i2 & 7u) * 4u),
                                      __ATOMIC_RELAXED, __HIP_MEMORY_SCOPE_WORKGROUP);
                ++i2;
            }
        }
        lds_barrier();
        if (part == 0) TS(1);
    }

    v16f acc[2][2];
#pragma unroll
    for (int a = 0; a < 2; ++a)
#pragma unroll
        for (int b = 0; b < 2; ++b)
#pragma unroll
            for (int i = 0; i < 16; ++i) acc[a][b][i] = 0.f;

    const gchar* xg = gptr(ws_me) + par_off + wave * 8192 + lane * 16;
    unsigned spin = 0u;
    auto wait_pub = [&](unsigned q, unsigned need) {
        unsigned c;
#pragma unroll 1
        for (;;) {
            c = __builtin_amdgcn_readfirstlane(lds_get(lds0 + S_PUB + q * 4));
            if (c & need) break;
            if ((c & C_DEAD) || ++spin > SPIN_MAX) {
                spin = SPIN_MAX;
                c = C_ITEM | C_RDY | C_OWN;
                break;
            }
            __builtin_amdgcn_s_sleep(1);
        }
        return c;
    };
    auto load_x = [&](v4i (&xf)[8], unsigned c) {
        const gchar* xp = xg + (c & 31u) * (unsigned)XITEM;
#pragma unroll
        for (int i = 0; i < 8; ++i) { if (ABL & 32) asm volatile("" : "=v"(xf[i])); else xf[i] = *(const gv4i*)((ABL & 1) ? xg + i * 1024 : xp + i * 1024); }
        asm volatile("" ::: "memory");
    };
    auto unpack = [&](unsigned c, unsigned q) {
        if ((c & C_OWN) | (ABL & 4)) return;
        const char* rec = smem + S_SLOT + (q & (NS - 1)) * REC;
        char* dec = smem + S_DEC + (q & 1u) * REC + tid * 16;
        if (c & C_PLAIN) {
            *reinterpret_cast<v4u*>(dec) = *reinterpret_cast<const v4u*>(rec + tid * 16);
            return;
        }
        const v2u nb = *reinterpret_cast<const v2u*>(rec + tid * 8);
        const unsigned wsg = *reinterpret_cast<const unsigned short*>(rec + 4096 + tid * 2);
        const unsigned hd = *reinterpret_cast<const unsigned*>(rec + RVAR + wave * 4);
        const unsigned N[4] = {nb[1] >> 4, nb[1], nb[0] >> 4, nb[0]};
        unsigned sel[4], cum[4], n1 = 0;
#pragma unroll
        for (int k = 0; k < 4; ++k) {
            const unsigned F = ~(N[k] | (N[k] >> 1)) & 0x04040404u;
            asm("v_mad_u32_u24 %0, %1, %2, %1" : "=v"(sel[k]) : "v"(F), "s"(0x404040u));
            cum[k] = n1;
            n1 += __builtin_popcount(F);
        }
        const unsigned e0 = (hd + wave_scan(n1) - n1) & 4095u;
        const unsigned* sp = reinterpret_cast<const unsigned*>(rec + RS1 + (e0 >> 4) * 4u);
        const unsigned bits = __builtin_amdgcn_alignbit(sp[1], sp[0], (e0 & 15u) * 2u);
        const unsigned X = wsg | (wsg << 17);
        const unsigned Ga = X & 0x88888888u, Gb = (X << 2) & 0x88888888u;
        const unsigned G[4] = {Ga, Ga << 4, Gb, Gb << 4};
        v4u o;
        unsigned f[4];
#pragma unroll
        for (int k = 0; k < 4; ++k) {
            unsigned b4 = __builtin_amdgcn_ubfe(bits, 2u * cum[k], 8u);
            b4 = (b4 | (b4 << 12)) & 0x000f000fu;
            b4 = (b4 | (b4 << 6)) & 0x03030303u;
            const unsigned tb = __builtin_amdgcn_perm(0u, 0x41292d31u, __builtin_amdgcn_perm(b4, 0u, sel[k]));
            o[k] = bfi(0x0f0f0f0fu, N[k], G[k]) + tb;
            f[k] = (k & 1) ? tb & (N[k] << 5) & 0x40404040u : (tb >> 1) & N[k + 1] & 0x20202020u;
        }
        *reinterpret_cast<v4u*>(dec) = o;
        unsigned m = f[3] | f[2] | ((f[1] | f[0]) >> 2);
        const unsigned n2 = __builtin_popcount(m);
        const unsigned i2 = ((hd >> 16) + wave_scan(n2) - n2) & 2047u;
        if (m) {
            const unsigned* s2 = reinterpret_cast<const unsigned*>(rec + RVAR + (i2 >> 3) * 4u);
            unsigned nib = __builtin_amdgcn_alignbit(s2[1], s2[0], (i2 & 7u) * 4u);
            do {
                const unsigned bit = __builtin_ctz(m);
                m &= m - 1u;
                const unsigned v = nib & 15u;
                nib >>= 4;
                __hip_atomic_fetch_sub(reinterpret_cast<unsigned*>(dec - 12 + (bit & 7u) * 4u), (v ? 26u + 2u * v : 0u) << (bit & 24u),
                                       __ATOMIC_RELAXED, __HIP_MEMORY_SCOPE_WORKGROUP);
            } while (m);
        }
        if ((unsigned)lane < ((hd >> 12) & 15u)) {
            const unsigned pe = *reinterpret_cast<const unsigned*>(rec + RS1 + lane * 4);
            if ((pe >> 18) == (unsigned)wave) smem[S_DEC + (q & 1u) * REC + (pe >> 8)] = (char)pe;
        }
    };
    auto step = [&](v4i (&xc)[8], v4i (&xn)[8], unsigned q, unsigned& c) {
        TF(0);
#pragma unroll 1
        for (;;) {
            if (__builtin_amdgcn_readfirstlane(lds_get(lds0 + S_CNT)) >= 8u * q + 8u) break;
            if (++spin > SPIN_MAX) break;
        }
        TF(1);
        unsigned cn = q < 31u ? __builtin_amdgcn_readfirstlane(lds_get(lds0 + S_PUB + q * 4 + 4)) : 0u;
        const bool ufirst = (wave & ORDB) && (cn & C_RDY);
        if (ufirst) unpack(cn, q + 1u);
        TF(2);
        asm volatile("" :: "v"(xc[0]), "v"(xc[1]), "v"(xc[2]), "v"(xc[3]), "v"(xc[4]), "v"(xc[5]), "v"(xc[6]), "v"(xc[7]));
        const bool early = cn & C_ITEM;
        if (early) load_x(xn, cn);
        else asm volatile("" : "=v"(xn[0]), "=v"(xn[1]), "=v"(xn[2]), "=v"(xn[3]), "=v"(xn[4]), "=v"(xn[5]), "=v"(xn[6]), "=v"(xn[7]));
        TF(3);
        {
            const char* lb = smem + ((c & C_OWN) ? S_RAW + (c & 3u) * REC : S_DEC + (q & 1u) * REC) + lane * 16;
#pragma unroll
            for (int ks = 0; ks < 2; ++ks) {
                v8i bf[2];
#pragma unroll
                for (int i = 0; i < 2; ++i) {
                    const v4i b0 = *reinterpret_cast<const v4i*>(lb + (i * 4 + ks * 2) * 1024);
                    const v4i b1 = *reinterpret_cast<const v4i*>(lb + (i * 4 + ks * 2 + 1) * 1024);
                    bf[i] = __builtin_shufflevector(b0, b1, 0, 1, 2, 3, 4, 5, 6, 7);
                }
                if (ks == 1 && ufirst) lds_inc(lds0 + S_CNT);
#pragma unroll
                for (int a = 0; a < ((ABL & 8) ? 0 : 2); ++a) {
                    const v8i xa = __builtin_shufflevector(xc[a * 4 + ks * 2], xc[a * 4 + ks * 2 + 1], 0, 1, 2, 3, 4, 5, 6, 7);
#pragma unroll
                    for (int b = 0; b < 2; ++b)
                        acc[a][b] = __builtin_amdgcn_mfma_scale_f32_32x32x64_f8f6f4(xa, bf[b], acc[a][b], 0, 1, 0, 127, 0, 127);
                }
                asm volatile("" : "+v"(acc[0][0]), "+v"(acc[0][1]), "+v"(acc[1][0]), "+v"(acc[1][1]));
            }
        }
        TF(4);
        if (!ufirst && q < 31u) {
            if (!early) {
                cn = wait_pub(q + 1u, C_ITEM);
                load_x(xn, cn);
            }
            if (!(cn & C_RDY)) cn = wait_pub(q + 1u, C_RDY);
            TF(5);
            unpack(cn, q + 1u);
            lds_inc(lds0 + S_CNT);
        }
        TF(6);
        c = cn;
    };
    {
        v4i xa[8], xb[8];
        unsigned c = wait_pub(0u, C_RDY);
        load_x(xa, c);
        lds_inc(lds0 + S_CNT);
#pragma unroll 1
        for (unsigned q = 0; q < 32u; q += 2u) {
            step(xa, xb, q, c);
            step(xb, xa, q + 1u, c);
        }
    }
    const bool dead = spin >= SPIN_MAX;
    TS(4);

    {
        const float scale = dead ? __builtin_nanf("") : sx[0] * sw[0];
        float* op = out + (size_t)(wave * 64 + 4 * h) * GN + strip * BN + r32;
#pragma unroll
        for (int a = 0; a < 2; ++a)
#pragma unroll
            for (int i = 0; i < 16; ++i)
#pragma unroll
                for (int b = 0; b < 2; ++b)
                    __hip_atomic_store(op + (size_t)(a * 32 + (i & 3) + 8 * (i >> 2)) * GN + b * 32, acc[a][b][i] * scale, __ATOMIC_RELAXED, __HIP_MEMORY_SCOPE_SYSTEM);
    }
    if (tid == 63)
        __hip_atomic_fetch_add(ctrl_word(ws_me, L_DONE), 1u, __ATOMIC_RELAXED, __HIP_MEMORY_SCOPE_AGENT);
    TS(5);
}

}

extern "C" void kernel_launch(int rank, int nranks, ncclComm_t comm,
                              hipStream_t stream,
                              void* const* d_in, const int* in_sizes, int n_in,
                              void* d_out, int out_size,
                              const int* in_dtypes, int out_dtype,
                              void* const* d_out_all,
                              void* const* d_ws_all, size_t ws_bytes) {
    (void)comm; (void)in_dtypes; (void)out_dtype; (void)d_out_all;
    if (nranks != NR || (unsigned)rank >= (unsigned)NR || n_in < 4 || ws_bytes < WS_NEED ||
        in_sizes[0] != GM * KS || in_sizes[1] != KS * GN || out_size != MS * GN) {
        fprintf(stderr, "kernel_launch: unexpected problem geometry\n");
        return;
    }
    hipLaunchKernelGGL(fused_gemm_kernel, dim3(NWG), dim3(NTHREADS), 0, stream,
                       (const unsigned char*)d_in[0], (const unsigned char*)d_in[1],
                       (const float*)d_in[2], (const float*)d_in[3], (float*)d_out, d_ws_all, rank);
    HIP_CHECK(hipGetLastError());
}
```

```cpp
#include <hip/hip_runtime.h>
#include <rccl/rccl.h>
#include <cstdio>
#include <cstdint>

#define HIP_CHECK(expr) do { hipError_t _e = (expr); if (_e != hipSuccess) { \
    fprintf(stderr, "HIP error %s at %s:%d\n", hipGetErrorString(_e), __FILE__, __LINE__); \
    return; } } while (0)

typedef int v8i __attribute__((ext_vector_type(8)));
typedef int v4i __attribute__((ext_vector_type(4)));
typedef unsigned v4u __attribute__((ext_vector_type(4)));
typedef unsigned v2u __attribute__((ext_vector_type(2)));
typedef float v16f __attribute__((ext_vector_type(16)));
typedef unsigned __attribute__((aligned(1))) u32u;
#define GAS __attribute__((address_space(1)))
typedef GAS char gchar;
typedef GAS unsigned gu32;
typedef GAS v4i gv4i;

namespace {

constexpr int NR = 8;
constexpr int GM = 4096, GK = 4096, GN = 16384;
constexpr int KS = GK / NR;
constexpr int MS = GM / NR;
constexpr int BN = 64, BK = 128;
constexpr int KBS = KS / BK;
constexpr int NWG = GN / BN;
constexpr int NTHREADS = 640;
constexpr int GW = NWG / 8;

constexpr size_t CTRL_BYTES = 1u << 20;
constexpr int XITEM = MS * BK;
constexpr int REC = BK * BN;
constexpr size_t X_BYTES = (size_t)NR * KBS * XITEM;
constexpr size_t PAR_BYTES = X_BYTES + (size_t)NWG * NR * KBS * REC;
constexpr size_t WS_NEED = CTRL_BYTES + 2 * PAR_BYTES;
constexpr int L_DONE = 0;
constexpr int L_TMO = 1;
constexpr int L_TICK = 4;
constexpr int L_XCNT = 16;
constexpr int L_GO = 256;
constexpr int L_FLAG = 512;
constexpr unsigned long long DEADLINE_TICKS = 100000000ull;

#define CREC 6144
constexpr int RVAR = 5120, RS1 = RVAR + 32;
static_assert(CREC <= 6144 && CREC >= RS1, "record size");

constexpr int S_RAW = 0;
constexpr int S_WIRE = S_RAW + KBS * REC;
constexpr int S_DEC = S_WIRE + KBS * 6144;
constexpr int NS = 8;
constexpr int S_SLOT = S_DEC + 2 * REC;
constexpr int S_SNAP = S_SLOT + NS * REC;
constexpr int S_MISC = S_SNAP + 256;
constexpr int S_PUB = S_MISC + 256;
constexpr int S_CNT = S_PUB + 128;
constexpr int S_GO = S_CNT + 4;
constexpr int LDS_BYTES = S_CNT + 16;
static_assert(LDS_BYTES > 80 * 1024, "LDS");

__device__ __forceinline__ gchar* gptr(void* p) { return (gchar*)p; }
__device__ __forceinline__ gu32* ctrl_word(void* ws, int line) {
    return (gu32*)(gptr(ws) + (size_t)line * 128);
}
__device__ __forceinline__ unsigned bfi(unsigned m, unsigned a, unsigned b) { return (a & m) | (b & ~m); }
__device__ __forceinline__ unsigned wave_scan(unsigned s) {
    s += __builtin_amdgcn_update_dpp(0u, s, 0x111, 0xf, 0xf, false);
    s += __builtin_amdgcn_update_dpp(0u, s, 0x112, 0xf, 0xf, false);
    s += __builtin_amdgcn_update_dpp(0u, s, 0x114, 0xf, 0xf, false);
    s += __builtin_amdgcn_update_dpp(0u, s, 0x118, 0xf, 0xf, false);
    s += __builtin_amdgcn_update_dpp(0u, s, 0x142, 0xa, 0xf, false);
    s += __builtin_amdgcn_update_dpp(0u, s, 0x143, 0xc, 0xf, false);
    return s;
}

#define WAIT_VM_(n) asm volatile("s_waitcnt vmcnt(" #n ")" ::: "memory")
#define WAIT_VM(n) WAIT_VM_(n)
#define NCH 65
#define LB 8
#define STAG 2
#define EARLY 0
#define PNEAR 3
#define PNAP 24
#define TG 2
#define NT 10
#define ABL 0
#define OST 2
#define ROTV 32
#define TAU 644u
#define OFF 450u
#define LEAD 200u
#define WIN 4
#define GAP 8
#define WEND 0
#define SWZ 0
#define NOCH 0
#define XWIN 2
#define TMOP 63u
#define LATE 21
#define LGRAN 32u
#define ORDB 4
#define PSL 2
#define TSC(c, e) do { } while (0)
#define TS(e) do { } while (0)
#define TM(v) do { } while (0)
#define TF(e) do { } while (0)

__device__ __forceinline__ void st16_wt(gchar* base, unsigned off, v4i v) {
    asm volatile("s_nop 4\n\tglobal_store_dwordx4 %0, %1, %2 sc0 sc1\n\ts_nop 1" :: "v"(off), "v"(v), "s"(base) : "memory");
}
__device__ __forceinline__ void ld4k_lds(const gchar* base, unsigned off, unsigned lds) {
    asm volatile("s_mov_b32 m0, %2\n\ts_nop 4\n\t"
                 "global_load_lds_dwordx4 %0, %1 sc0 sc1\n\t"
                 "global_load_lds_dwordx4 %0, %1 offset:1024 sc0 sc1\n\t"
                 "global_load_lds_dwordx4 %0, %1 offset:2048 sc0 sc1\n\t"
                 "global_load_lds_dwordx4 %0, %1 offset:3072 sc0 sc1"
                 :: "v"(off), "s"(base), "s"(lds) : "memory");
}
__device__ __forceinline__ void ld2k_lds(const gchar* base, unsigned off, unsigned lds) {
    asm volatile("s_mov_b32 m0, %2\n\ts_nop 4\n\t"
                 "global_load_lds_dwordx4 %0, %1 sc0 sc1\n\t"
                 "global_load_lds_dwordx4 %0, %1 offset:1024 sc0 sc1"
                 :: "v"(off), "s"(base), "s"(lds) : "memory");
}
__device__ __forceinline__ void ld4_lds(const gchar* base, unsigned off, unsigned lds, unsigned long long lanes) {
    asm volatile("s_mov_b32 m0, %2\n\ts_mov_b64 exec, %3\n\ts_nop 3\n\t"
                 "global_load_lds_dword %0, %1 sc0 sc1\n\ts_mov_b64 exec, -1"
                 :: "v"(off), "s"(base), "s"(lds), "s"(lanes) : "memory");
}
__device__ __forceinline__ unsigned lds_get(unsigned addr) {
    unsigned v;
    asm volatile("ds_read_b32 %0, %1\n\ts_waitcnt lgkmcnt(0)" : "=v"(v) : "v"(addr) : "memory");
    return v;
}
__device__ __forceinline__ void lds_put(unsigned addr, unsigned v) {
    asm volatile("s_mov_b64 exec, 1\n\tds_write_b32 %0, %1\n\ts_mov_b64 exec, -1" :: "v"(addr), "v"(v) : "memory");
}
__device__ __forceinline__ void lds_inc(unsigned addr) {
    asm volatile("s_mov_b64 exec, 1\n\tds_add_u32 %0, %1\n\ts_mov_b64 exec, -1" :: "v"(addr), "v"(1u) : "memory");
}
constexpr unsigned SPIN_MAX = 1u << 25;
__device__ __forceinline__ void lds_barrier() {
    asm volatile("s_waitcnt lgkmcnt(0)\n\ts_barrier" ::: "memory");
}

enum : unsigned { C_ITEM = 0x100u, C_PLAIN = 0x20u, C_OWN = 0x40u, C_DEAD = 0x200u, C_RDY = 0x400u };

__global__ __launch_bounds__(NTHREADS)
void fused_gemm_kernel(const unsigned char* __restrict__ xs,
                       const unsigned char* __restrict__ wsrc,
                       const float* __restrict__ sx, const float* __restrict__ sw,
                       float* __restrict__ out, void* const* __restrict__ ws_all, int rank) {
    __shared__ __attribute__((aligned(16))) char smem[LDS_BYTES];
    const int tid = threadIdx.x;
    const int lane = tid & 63;
    const int wave = __builtin_amdgcn_readfirstlane(tid >> 6);
    const int r32 = lane & 31, h = lane >> 5;
    const unsigned long long t0 = __builtin_amdgcn_s_memrealtime();

    const int d = (int)blockIdx.x;
    const int grp = d & 7;
    const int strip = grp * GW + (d >> 3);
    constexpr int BS = 1 << LB;
    const int dd = (d >> LB << LB) | ((d - ROTV * rank) & (BS - 1));
    auto slot_of = [&](int src, int r) { return (unsigned)((strip * NR + src) * KBS + ((SWZ & 1) ? (r + (d >> 1)) & 3 : r)) * (unsigned)REC; };
    const unsigned sx4 = (SWZ & 2) ? (unsigned)(d & 1) << 12 : 0u;
    const int ph = (d + wave) % 7;
    auto rot = [&](int i) { int e = i + ph; return e >= 7 ? e - 7 : e; };

    const unsigned lds0 = (unsigned)(size_t)(__attribute__((address_space(3))) char*)smem;
    void* ws_me;
    unsigned call, par_off;
    auto whoami = [&]() {
        ws_me = ws_all[rank];
        unsigned v;
        static_assert(L_DONE == 0, "counter offset");
        asm volatile("s_load_dword %0, %1, 0x0\n\ts_waitcnt lgkmcnt(0)" : "=s"(v) : "s"(ws_me));
        call = v / NWG + 1u;
        par_off = (unsigned)CTRL_BYTES + (call & 1u) * (unsigned)PAR_BYTES;
    };

    if (wave >= 8) {
        __builtin_amdgcn_s_setprio(3);
        const bool xpusher = dd >= NWG / 2;
        const int xb = ((dd & 7) << 2 | (dd >> 6 & 1) << 1 | (wave & 1)) * 2;
        const int xdst = (rank + ((dd >> 3) ^ (wave << 2))) & 7;
        v4i xin[8];
        if (xpusher) {
            const unsigned char* xp = xs + (xdst * MS + (xb & 15) * 32 + r32) * KS + (xb >> 4) * BK + h * 16;
#pragma unroll
            for (int i = 0; i < 8; ++i) xin[i] = *reinterpret_cast<const v4i*>(xp + (i >> 2) * 32 * KS + (i & 3) * 32);
        }
        gchar* xws = gptr(ws_all[xdst]);
        whoami();
        TM(tB);
        if (xpusher) {
            asm volatile("" :: "v"(xin[0]), "v"(xin[1]), "v"(xin[2]), "v"(xin[3]),
                               "v"(xin[4]), "v"(xin[5]), "v"(xin[6]), "v"(xin[7]));
#pragma unroll
            for (int i = 0; i < 8; ++i) {
                WAIT_VM(XWIN);
                st16_wt(xws, par_off + rank * KBS * XITEM + xb * 4096 + i * 1024 + lane * 16, xin[i]);
            }
            WAIT_VM(0);
            if (lane == 0)
                __hip_atomic_fetch_add((gu32*)(xws + (L_XCNT + rank) * 128), 1u, __ATOMIC_RELAXED, __HIP_MEMORY_SCOPE_SYSTEM);
        }
        TM(tC);
        if (wave == 8) {
            const unsigned fline = (L_FLAG + (call & 1u) * NWG + strip) * 128;
            if (lane < 36) *reinterpret_cast<unsigned*>(smem + S_PUB + lane * 4) = 0u;
            lds_barrier();
            lds_barrier();
            lds_barrier();
            lds_barrier();
            lds_barrier();
            const unsigned woff = lane < 32 ? fline + lane * 4 : lane < 40 ? (L_XCNT + lane - 32) * 128 : L_TMO * 128;
            const unsigned thr = lane < 32 ? call << 1 : lane < 40 ? 32u * call : call;
            const gchar* wg = gptr(ws_me) + par_off + X_BYTES;
            const unsigned myslot = slot_of((lane >> 2) & 7, lane & 3);
            unsigned long long watch = 0x1ffffffffffull;
            unsigned taken = 15u << (4 * rank), q = 0, polls = 0;
            ld4_lds(gptr(ws_me), woff, lds0 + S_SNAP, watch);
#pragma unroll 1
            while (q < 32u) {
                WAIT_VM(0);
                const unsigned pv = lds_get(lds0 + S_SNAP + lane * 4);
                const unsigned used = __builtin_amdgcn_readfirstlane(lds_get(lds0 + S_CNT)) >> 3;
                const unsigned long long ok = __ballot((int)(pv - thr) >= 0);
                const unsigned pl = (unsigned)__ballot(pv & 1u);
                const bool late = __builtin_amdgcn_s_memrealtime() - t0 > DEADLINE_TICKS;
                if (late || ((ok >> 40) & 1u)) {
                    if (late && lane == 0)
                        __hip_atomic_store(ctrl_word(ws_me, L_TMO), call, __ATOMIC_RELAXED, __HIP_MEMORY_SCOPE_SYSTEM);
                    lds_put(lds0 + S_PUB + q * 4, C_DEAD);
                    break;
                }
                if (q == 0u && ((ok >> 32) & 0xffu) == 0xffu) {
                    __builtin_amdgcn_fence(__ATOMIC_ACQUIRE, "");
                    WAIT_VM(0);
                    if (lane < KBS) *reinterpret_cast<volatile unsigned*>(smem + S_PUB + lane * 4) = C_ITEM | C_RDY | C_OWN | (unsigned)(4 * rank + lane);
                    q = KBS;
                    TSC(lane == 0, 6);
                    WAIT_VM(0);
                    watch = 0x100ffffffffull;
                }
                unsigned rem = q ? (unsigned)ok & ~taken : 0u;
                unsigned n = 0, code[4] = {0u, 0u, 0u, 0u}, anyp = 0;
#pragma unroll
                for (int i = 0; i < 4; ++i) {
                    if (rem && q + i < used + NS) {
                        const unsigned it = __builtin_ctz(rem);
                        rem &= rem - 1u;
                        taken |= 1u << it;
                        const gchar* rp = wg + (unsigned)__builtin_amdgcn_readlane(myslot, (ABL & 2) ? 4 * rank : it);
                        const unsigned sl = lds0 + S_SLOT + ((q + i) & (NS - 1)) * REC;
                        if (!(ABL & 16)) ld4k_lds(rp + sx4, lane * 16, sl);
                        const unsigned p = (pl >> it) & 1u;
                        if (p) ld4k_lds(rp + (4096u ^ sx4), lane * 16, sl + 4096);
                        else if (!(ABL & 16)) ld2k_lds(rp + (4096u ^ sx4), lane * 16, sl + 4096);
                        anyp |= p;
                        code[i] = C_ITEM | it | p * C_PLAIN;
                        lds_put(lds0 + S_PUB + (q + i) * 4, code[i]);
                        n = i + 1;
                    } else {
                        rem = 0u;
                    }
                }
                ld4_lds(gptr(ws_me), woff, lds0 + S_SNAP, (++polls & TMOP) ? watch & ~(1ull << 40) : watch);
                if (anyp) WAIT_VM(1);
#pragma unroll
                for (int i = 0; i < 4; ++i) {
                    if (i < (int)n) {
                        const unsigned left = n - 1u - i;
                        if (left == 3u) { WAIT_VM(19); } else if (left == 2u) { WAIT_VM(13); } else if (left == 1u) { WAIT_VM(7); } else { WAIT_VM(1); }
                        lds_put(lds0 + S_PUB + (q + i) * 4, code[i] | C_RDY);
                    }
                }
                q += n;
            }
            WAIT_VM(0);
            return;
        }
        gchar* dws[NR - 1];
#pragma unroll
        for (int i = 0; i < NR - 1; ++i) dws[i] = gptr(ws_all[(rank + 1 + rot(i)) & 7]);
        gchar* sigb = dws[0];
#pragma unroll
        for (int i = 1; i < NR - 1; ++i) sigb = lane == i ? dws[i] : sigb;
        sigb += (L_FLAG + (call & 1u) * NWG + strip) * 128 + rank * KBS * 4;
        __builtin_amdgcn_s_waitcnt(0);
        lds_barrier();
        lds_barrier();
        lds_barrier();
#pragma unroll 1
        for (int r = 0; r < KBS; ++r) {
            const int seq = (dd >> LB << (LB + 2)) | (r << LB) | (dd & (BS - 1));
            if (r == (dd < (NT << TG) ? 1 : 0)) {
                lds_barrier();
                lds_barrier();
            }
            auto due_of = [&](int c) {
                int n = 0;
#pragma unroll
                for (int e = 0; e < 6; ++e) n += min(max(seq + (c - e) * GAP + (e < c ? 1 : 0), 0), NWG * KBS);
                return OFF + (unsigned)n * TAU / 384u;
            };
            const unsigned due = due_of(0);
            if (seq >= NCH && seq < NWG * KBS - NOCH) {
#pragma unroll 1
                while ((unsigned)(__builtin_amdgcn_s_memrealtime() - t0) + LEAD < due) __builtin_amdgcn_s_sleep(8);
#pragma unroll 1
                for (;;) {
                    const unsigned cv = __hip_atomic_load(ctrl_word(ws_me, L_GO + dd) + r, __ATOMIC_RELAXED, __HIP_MEMORY_SCOPE_SYSTEM);
                    if (__builtin_amdgcn_readfirstlane(cv) == call) break;
                    if (__builtin_amdgcn_s_memrealtime() - t0 > DEADLINE_TICKS) {
                        if (lane == 0)
                            __hip_atomic_store(ctrl_word(ws_me, L_TMO), call, __ATOMIC_RELAXED, __HIP_MEMORY_SCOPE_SYSTEM);
                        return;
                    }
                    __builtin_amdgcn_s_sleep(PSL);
                }
            }
            const unsigned slot = par_off + (unsigned)X_BYTES + slot_of(rank, r) + lane * 16;
            const unsigned len = __builtin_amdgcn_readfirstlane(*reinterpret_cast<const unsigned*>(smem + S_MISC + 192 + r * 4));
            const char* src = smem + (len == (unsigned)REC ? S_RAW + r * REC : S_WIRE + r * 6144) + lane * 16;
#pragma unroll 1
            for (unsigned c = 0; c < len; c += 1024u) {
                const unsigned duc = due_of(min((int)(c >> 10), 5));
#pragma unroll 1
                while ((unsigned)(__builtin_amdgcn_s_memrealtime() - t0) < duc) __builtin_amdgcn_s_sleep(1);
                if (seq + (int)(c >> 10) * GAP < NWG * KBS + WEND) WAIT_VM(WIN);
                if (c + lane * 16 < len) {
                    const v4i v = *reinterpret_cast<const v4i*>(src + c);
#pragma unroll
                    for (int i = 0; i < 7; ++i) st16_wt(dws[i], slot + (c ^ sx4), v);
                }
            }
            TM(tE);
            if (EARLY < 0) WAIT_VM(LATE);
            if (seq + NCH < NWG * KBS && lane == 0)
                __hip_atomic_store(ctrl_word(ws_me, L_GO + ((seq + NCH) & (NWG - 1))) + ((seq + NCH) >> 8), call, __ATOMIC_RELAXED, __HIP_MEMORY_SCOPE_SYSTEM);
            WAIT_VM(0);
            TM(tF);
            if (lane < NR - 1)
                __hip_atomic_store((gu32*)(sigb + r * 4), (call << 1) | (len == (unsigned)REC),
                                   __ATOMIC_RELAXED, __HIP_MEMORY_SCOPE_SYSTEM);
        }
        return;
    }

    unsigned wv[16];
    {
        const unsigned char* wp = wsrc + (size_t)((tid >> 4) * 16) * GN + strip * BN + (tid & 15) * 4;
#pragma unroll
        for (int j = 0; j < 16; ++j) wv[j] = *reinterpret_cast<const unsigned*>(wp + (size_t)j * GN);
    }
    whoami();
    TS(0);

    unsigned* s_sel = reinterpret_cast<unsigned*>(smem + S_MISC);
    unsigned* s_tot = s_sel + 16;
    {
        const int kc = tid >> 4, n0 = (tid & 15) * 4;
        char* dst = smem + S_RAW + (kc >> 3) * REC + (((n0 >> 5) * 4 + ((kc >> 1) & 3)) * 64 + (kc & 1) * 32 + (n0 & 31)) * 16;
        v4u o[4];
#pragma unroll
        for (int q = 0; q < 4; ++q) {
            const unsigned a0 = wv[4 * q], a1 = wv[4 * q + 1], a2 = wv[4 * q + 2], a3 = wv[4 * q + 3];
            const unsigned u0 = __builtin_amdgcn_perm(a1, a0, 0x05010400u);
            const unsigned u1 = __builtin_amdgcn_perm(a1, a0, 0x07030602u);
            const unsigned u2 = __builtin_amdgcn_perm(a3, a2, 0x05010400u);
            const unsigned u3 = __builtin_amdgcn_perm(a3, a2, 0x07030602u);
            o[0][q] = __builtin_amdgcn_perm(u2, u0, 0x05040100u);
            o[1][q] = __builtin_amdgcn_perm(u2, u0, 0x07060302u);
            o[2][q] = __builtin_amdgcn_perm(u3, u1, 0x05040100u);
            o[3][q] = __builtin_amdgcn_perm(u3, u1, 0x07060302u);
        }
#pragma unroll
        for (int i = 0; i < 4; ++i) *reinterpret_cast<v4u*>(dst + i * 16) = o[i];
        if (tid < 256) *reinterpret_cast<v4u*>(smem + S_WIRE + (tid >> 6) * 6144 + RVAR + lane * 16) = v4u{0u, 0u, 0u, 0u};
        if (tid < 16) {
            unsigned sel = 0x0c0c0c0cu, n = 0;
#pragma unroll
            for (int b = 0; b < 4; ++b)
                if ((tid >> b) & 1) { sel = bfi(0xffu << (8 * n), (unsigned)b << (8 * n), sel); ++n; }
            s_sel[tid] = sel;
            if (tid < KBS) s_tot[36 + tid] = 0u;
        }
    }
    lds_barrier();
#pragma unroll
    for (int part = 0; part < 2; ++part) {
        const int kb0 = part, kb1 = part ? KBS : 1;
        unsigned bits[KBS], m2[KBS], sc[KBS];
#pragma unroll
        for (int kb = kb0; kb < kb1; ++kb) {
            const v4u c = *reinterpret_cast<const v4u*>(smem + S_RAW + kb * REC + tid * 16);
            unsigned S[4], n1 = 0, mp = 0;
            m2[kb] = 0;
            bits[kb] = 0;
#pragma unroll
            for (int q = 0; q < 4; ++q) {
                unsigned t = ((c[q] & 0x7f7f7f7fu) + (128u - 41u) * 0x01010101u) & 0x7f7f7f7fu;
                const unsigned fl = (t + (128u - 28u) * 0x01010101u) & 0x80808080u;
                const unsigned l3 = (t + (128u - 98u) * 0x01010101u) & 0x80808080u;
                t = bfi((fl << 1) - (fl >> 7), 0x12121212u | (t & 0x01010101u) | (l3 >> 4), t);
                const unsigned e = __builtin_amdgcn_perm(0x00b00c08u, 0x048090a0u, (t >> 2) & 0x07070707u);
                S[q] = (t & 0x03030303u) | (e & 0x0c0c0c0cu);
                const unsigned F = (e >> 7) & 0x01010101u;
                const unsigned sel = s_sel[__builtin_amdgcn_udot4(F, 0x08040201u, 0u, false)];
                bits[kb] |= __builtin_amdgcn_udot4(__builtin_amdgcn_perm(0u, (e >> 4) & 0x03030303u, sel), 0x40100401u, 0u, false) << (2u * n1);
                n1 += __builtin_popcount(F);
                m2[kb] |= ((t + (128u - 26u) * 0x01010101u) & 0x80808080u) >> (4 - q);
                mp |= (fl & ~l3) >> (4 - q);
            }
            char* rec = smem + S_WIRE + kb * 6144;
            *reinterpret_cast<v2u*>(rec + tid * 8) = v2u{S[3] | (S[2] << 4), S[1] | (S[0] << 4)};
            const unsigned Y = (c[0] & 0x80808080u) | ((c[1] & 0x80808080u) >> 4) | ((c[2] & 0x80808080u) >> 2) | ((c[3] & 0x80808080u) >> 6);
            *reinterpret_cast<unsigned short*>(rec + 4096 + tid * 2) = (unsigned short)(Y | (Y >> 17));
            const unsigned n2 = __builtin_popcount(m2[kb]);
            if (n2 > 8u) __hip_atomic_fetch_add(s_tot + 36 + kb, 16u, __ATOMIC_RELAXED, __HIP_MEMORY_SCOPE_WORKGROUP);
            while (mp) {
                const unsigned bit = __builtin_ctz(mp);
                mp &= mp - 1u;
                const unsigned pos = tid * 16 + ((bit & 7u) - 3u) * 4 + (bit >> 3);
                const unsigned ix = __hip_atomic_fetch_add(s_tot + 36 + kb, 1u, __ATOMIC_RELAXED, __HIP_MEMORY_SCOPE_WORKGROUP);
                if (ix < 15u)
                    *reinterpret_cast<unsigned*>(rec + RS1 + ix * 4) = pos << 8 | (unsigned char)smem[S_RAW + kb * REC + pos];
            }
            const unsigned cnt = n1 | (n2 << 16);
            const unsigned in = wave_scan(cnt);
            if (lane == 63) s_tot[kb * 8 + wave] = in;
            sc[kb] = in - cnt;
        }
        lds_barrier();
#pragma unroll
        for (int kb = kb0; kb < kb1; ++kb) {
            unsigned base = 0, tot = 0;
#pragma unroll
            for (int w = 0; w < 8; ++w) {
                const unsigned v = __builtin_amdgcn_readfirstlane(s_tot[kb * 8 + w]);
                tot += v;
                if (w < wave) base += v;
            }
            const unsigned np = __builtin_amdgcn_readfirstlane(s_tot[36 + kb]);
            const unsigned o2 = 32u + np * 4u + (((tot & 0xffffu) + 15u) >> 4) * 4u;
            const unsigned rlen = np > 15u ? (unsigned)REC : RVAR + o2 + (((tot >> 16) + 1u) >> 1);
            if (tid == 0) s_tot[32 + kb] = rlen > (unsigned)CREC ? (unsigned)REC : (rlen + LGRAN - 1u) & ~(LGRAN - 1u);
            if (rlen > (unsigned)CREC) continue;
            base += (o2 << 17) + np * 16u;
            char* rec = smem + S_WIRE + kb * 6144;
            if (lane == 0) *reinterpret_cast<unsigned*>(rec + RVAR + wave * 4) = base | np << 12;
            const unsigned e0 = (base + sc[kb]) & 0xffffu;
            const unsigned long long pk = (unsigned long long)bits[kb] << ((e0 & 15u) * 2u);
            unsigned* p = reinterpret_cast<unsigned*>(rec + RS1 + (e0 >> 4) * 4u);
            __hip_atomic_fetch_or(p, (unsigned)pk, __ATOMIC_RELAXED, __HIP_MEMORY_SCOPE_WORKGROUP);
            __hip_atomic_fetch_or(p + 1, (unsigned)(pk >> 32), __ATOMIC_RELAXED, __HIP_MEMORY_SCOPE_WORKGROUP);
            unsigned m = m2[kb], i2 = (base + sc[kb]) >> 16;
            while (m) {
                const unsigned bit = __builtin_ctz(m);
                m &= m - 1u;
                const unsigned cb = (unsigned char)smem[S_RAW + kb * REC + tid * 16 + ((bit & 7u) - 3u) * 4 + (bit >> 3)] & 0x7fu;
                const unsigned v = cb > 42u ? 0u : (42u - cb) >> 1;
                __hip_atomic_fetch_or(reinterpret_cast<unsigned*>(rec + RVAR + (i2 >> 3) * 4u), v << ((i2 & 7u) * 4u),
                                      __ATOMIC_RELAXED, __HIP_MEMORY_SCOPE_WORKGROUP);
                ++i2;
            }
        }
        lds_barrier();
        if (part == 0) TS(1);
    }

    v16f acc[2][2];
#pragma unroll
    for (int a = 0; a < 2; ++a)
#pragma unroll
        for (int b = 0; b < 2; ++b)
#pragma unroll
            for (int i = 0; i < 16; ++i) acc[a][b][i] = 0.f;

    const gchar* xg = gptr(ws_me) + par_off + wave * 8192 + lane * 16;
    unsigned spin = 0u;
    auto wait_pub = [&](unsigned q, unsigned need) {
        unsigned c;
#pragma unroll 1
        for (;;) {
            c = __builtin_amdgcn_readfirstlane(lds_get(lds0 + S_PUB + q * 4));
            if (c & need) break;
            if ((c & C_DEAD) || ++spin > SPIN_MAX) {
                spin = SPIN_MAX;
                c = C_ITEM | C_RDY | C_OWN;
                break;
            }
            __builtin_amdgcn_s_sleep(1);
        }
        return c;
    };
    auto load_x = [&](v4i (&xf)[8], unsigned c) {
        const gchar* xp = xg + (c & 31u) * (unsigned)XITEM;
#pragma unroll
        for (int i = 0; i < 8; ++i) { if (ABL & 32) asm volatile("" : "=v"(xf[i])); else xf[i] = *(const gv4i*)((ABL & 1) ? xg + i * 1024 : xp + i * 1024); }
        asm volatile("" ::: "memory");
    };
    auto unpack = [&](unsigned c, unsigned q) {
        if ((c & C_OWN) | (ABL & 4)) return;
        const char* rec = smem + S_SLOT + (q & (NS - 1)) * REC;
        char* dec = smem + S_DEC + (q & 1u) * REC + tid * 16;
        if (c & C_PLAIN) {
            *reinterpret_cast<v4u*>(dec) = *reinterpret_cast<const v4u*>(rec + tid * 16);
            return;
        }
        const v2u nb = *reinterpret_cast<const v2u*>(rec + tid * 8);
        const unsigned wsg = *reinterpret_cast<const unsigned short*>(rec + 4096 + tid * 2);
        const unsigned hd = *reinterpret_cast<const unsigned*>(rec + RVAR + wave * 4);
        const unsigned N[4] = {nb[1] >> 4, nb[1], nb[0] >> 4, nb[0]};
        unsigned sel[4], cum[4], n1 = 0;
#pragma unroll
        for (int k = 0; k < 4; ++k) {
            const unsigned F = ~(N[k] | (N[k] >> 1)) & 0x04040404u;
            asm("v_mad_u32_u24 %0, %1, %2, %1" : "=v"(sel[k]) : "v"(F), "s"(0x404040u));
            cum[k] = n1;
            n1 += __builtin_popcount(F);
        }
        const unsigned e0 = (hd + wave_scan(n1) - n1) & 4095u;
        const unsigned* sp = reinterpret_cast<const unsigned*>(rec + RS1 + (e0 >> 4) * 4u);
        const unsigned bits = __builtin_amdgcn_alignbit(sp[1], sp[0], (e0 & 15u) * 2u);
        const unsigned X = wsg | (wsg << 17);
        const unsigned Ga = X & 0x88888888u, Gb = (X << 2) & 0x88888888u;
        const unsigned G[4] = {Ga, Ga << 4, Gb, Gb << 4};
        v4u o;
        unsigned f[4];
#pragma unroll
        for (int k = 0; k < 4; ++k) {
            unsigned b4 = __builtin_amdgcn_ubfe(bits, 2u * cum[k], 8u);
            b4 = (b4 | (b4 << 12)) & 0x000f000fu;
            b4 = (b4 | (b4 << 6)) & 0x03030303u;
            const unsigned tb = __builtin_amdgcn_perm(0u, 0x41292d31u, __builtin_amdgcn_perm(b4, 0u, sel[k]));
            o[k] = bfi(0x0f0f0f0fu, N[k], G[k]) + tb;
            f[k] = (k & 1) ? tb & (N[k] << 5) & 0x40404040u : (tb >> 1) & N[k + 1] & 0x20202020u;
        }
        *reinterpret_cast<v4u*>(dec) = o;
        unsigned m = f[3] | f[2] | ((f[1] | f[0]) >> 2);
        const unsigned n2 = __builtin_popcount(m);
        const unsigned i2 = ((hd >> 16) + wave_scan(n2) - n2) & 2047u;
        if (m) {
            const unsigned* s2 = reinterpret_cast<const unsigned*>(rec + RVAR + (i2 >> 3) * 4u);
            unsigned nib = __builtin_amdgcn_alignbit(s2[1], s2[0], (i2 & 7u) * 4u);
            do {
                const unsigned bit = __builtin_ctz(m);
                m &= m - 1u;
                const unsigned v = nib & 15u;
                nib >>= 4;
                __hip_atomic_fetch_sub(reinterpret_cast<unsigned*>(dec - 12 + (bit & 7u) * 4u), (v ? 26u + 2u * v : 0u) << (bit & 24u),
                                       __ATOMIC_RELAXED, __HIP_MEMORY_SCOPE_WORKGROUP);
            } while (m);
        }
        if ((unsigned)lane < ((hd >> 12) & 15u)) {
            const unsigned pe = *reinterpret_cast<const unsigned*>(rec + RS1 + lane * 4);
            if ((pe >> 18) == (unsigned)wave) smem[S_DEC + (q & 1u) * REC + (pe >> 8)] = (char)pe;
        }
    };
    auto step = [&](v4i (&xc)[8], v4i (&xn)[8], unsigned q, unsigned& c) {
        TF(0);
#pragma unroll 1
        for (;;) {
            if (__builtin_amdgcn_readfirstlane(lds_get(lds0 + S_CNT)) >= 8u * q + 8u) break;
            if (++spin > SPIN_MAX) break;
        }
        TF(1);
        unsigned cn = q < 31u ? __builtin_amdgcn_readfirstlane(lds_get(lds0 + S_PUB + q * 4 + 4)) : 0u;
        const bool ufirst = (wave & ORDB) && (cn & C_RDY);
        if (ufirst) unpack(cn, q + 1u);
        TF(2);
        asm volatile("" :: "v"(xc[0]), "v"(xc[1]), "v"(xc[2]), "v"(xc[3]), "v"(xc[4]), "v"(xc[5]), "v"(xc[6]), "v"(xc[7]));
        const bool early = cn & C_ITEM;
        if (early) load_x(xn, cn);
        else asm volatile("" : "=v"(xn[0]), "=v"(xn[1]), "=v"(xn[2]), "=v"(xn[3]), "=v"(xn[4]), "=v"(xn[5]), "=v"(xn[6]), "=v"(xn[7]));
        TF(3);
        {
            const char* lb = smem + ((c & C_OWN) ? S_RAW + (c & 3u) * REC : S_DEC + (q & 1u) * REC) + lane * 16;
#pragma unroll
            for (int ks = 0; ks < 2; ++ks) {
                v8i bf[2];
#pragma unroll
                for (int i = 0; i < 2; ++i) {
                    const v4i b0 = *reinterpret_cast<const v4i*>(lb + (i * 4 + ks * 2) * 1024);
                    const v4i b1 = *reinterpret_cast<const v4i*>(lb + (i * 4 + ks * 2 + 1) * 1024);
                    bf[i] = __builtin_shufflevector(b0, b1, 0, 1, 2, 3, 4, 5, 6, 7);
                }
                if (ks == 1 && ufirst) lds_inc(lds0 + S_CNT);
#pragma unroll
                for (int a = 0; a < ((ABL & 8) ? 0 : 2); ++a) {
                    const v8i xa = __builtin_shufflevector(xc[a * 4 + ks * 2], xc[a * 4 + ks * 2 + 1], 0, 1, 2, 3, 4, 5, 6, 7);
#pragma unroll
                    for (int b = 0; b < 2; ++b)
                        acc[a][b] = __builtin_amdgcn_mfma_scale_f32_32x32x64_f8f6f4(xa, bf[b], acc[a][b], 0, 1, 0, 127, 0, 127);
                }
                asm volatile("" : "+v"(acc[0][0]), "+v"(acc[0][1]), "+v"(acc[1][0]), "+v"(acc[1][1]));
            }
        }
        TF(4);
        if (!ufirst && q < 31u) {
            if (!early) {
                cn = wait_pub(q + 1u, C_ITEM);
                load_x(xn, cn);
            }
            if (!(cn & C_RDY)) cn = wait_pub(q + 1u, C_RDY);
            TF(5);
            unpack(cn, q + 1u);
            lds_inc(lds0 + S_CNT);
        }
        TF(6);
        c = cn;
    };
    {
        v4i xa[8], xb[8];
        unsigned c = wait_pub(0u, C_RDY);
        load_x(xa, c);
        lds_inc(lds0 + S_CNT);
#pragma unroll 1
        for (unsigned q = 0; q < 32u; q += 2u) {
            step(xa, xb, q, c);
            step(xb, xa, q + 1u, c);
        }
    }
    const bool dead = spin >= SPIN_MAX;
    TS(4);

    {
        const float scale = dead ? __builtin_nanf("") : sx[0] * sw[0];
        float* op = out + (size_t)(wave * 64 + 4 * h) * GN + strip * BN + r32;
#pragma unroll
        for (int a = 0; a < 2; ++a)
#pragma unroll
            for (int i = 0; i < 16; ++i)
#pragma unroll
                for (int b = 0; b < 2; ++b)
                    __hip_atomic_store(op + (size_t)(a * 32 + (i & 3) + 8 * (i >> 2)) * GN + b * 32, acc[a][b][i] * scale, __ATOMIC_RELAXED, __HIP_MEMORY_SCOPE_SYSTEM);
    }
    if (tid == 63)
        __hip_atomic_fetch_add(ctrl_word(ws_me, L_DONE), 1u, __ATOMIC_RELAXED, __HIP_MEMORY_SCOPE_AGENT);
    TS(5);
}

}

extern "C" void kernel_launch(int rank, int nranks, ncclComm_t comm,
                              hipStream_t stream,
                              void* const* d_in, const int* in_sizes, int n_in,
                              void* d_out, int out_size,
                              const int* in_dtypes, int out_dtype,
                              void* const* d_out_all,
                              void* const* d_ws_all, size_t ws_bytes) {
    (void)comm; (void)in_dtypes; (void)out_dtype; (void)d_out_all;
    if (nranks != NR || (unsigned)rank >= (unsigned)NR || n_in < 4 || ws_bytes < WS_NEED ||
        in_sizes[0] != GM * KS || in_sizes[1] != KS * GN || out_size != MS * GN) {
        fprintf(stderr, "kernel_launch: unexpected problem geometry\n");
        return;
    }
    hipLaunchKernelGGL(fused_gemm_kernel, dim3(NWG), dim3(NTHREADS), 0, stream,
                       (const unsigned char*)d_in[0], (const unsigned char*)d_in[1],
                       (const float*)d_in[2], (const float*)d_in[3], (float*)d_out, d_ws_all, rank);
    HIP_CHECK(hipGetLastError());
}
```

```cpp
#include <hip/hip_runtime.h>
#include <rccl/rccl.h>
#include <cstdio>
#include <cstdint>

#define HIP_CHECK(expr) do { hipError_t _e = (expr); if (_e != hipSuccess) { \
    fprintf(stderr, "HIP error %s at %s:%d\n", hipGetErrorString(_e), __FILE__, __LINE__); \
    return; } } while (0)

typedef int v8i __attribute__((ext_vector_type(8)));
typedef int v4i __attribute__((ext_vector_type(4)));
typedef unsigned v4u __attribute__((ext_vector_type(4)));
typedef unsigned v2u __attribute__((ext_vector_type(2)));
typedef float v16f __attribute__((ext_vector_type(16)));
typedef unsigned __attribute__((aligned(1))) u32u;
#define GAS __attribute__((address_space(1)))
typedef GAS char gchar;
typedef GAS unsigned gu32;
typedef GAS v4i gv4i;

namespace {

constexpr int NR = 8;
constexpr int GM = 4096, GK = 4096, GN = 16384;
constexpr int KS = GK / NR;
constexpr int MS = GM / NR;
constexpr int BN = 64, BK = 128;
constexpr int KBS = KS / BK;
constexpr int NWG = GN / BN;
constexpr int NTHREADS = 640;
constexpr int GW = NWG / 8;

constexpr size_t CTRL_BYTES = 1u << 20;
constexpr int XITEM = MS * BK;
constexpr int REC = BK * BN;
constexpr size_t X_BYTES = (size_t)NR * KBS * XITEM;
constexpr size_t PAR_BYTES = X_BYTES + (size_t)NWG * NR * KBS * REC;
constexpr size_t WS_NEED = CTRL_BYTES + 2 * PAR_BYTES;
constexpr int L_DONE = 0;
constexpr int L_TMO = 1;
constexpr int L_TICK = 4;
constexpr int L_XCNT = 16;
constexpr int L_GO = 256;
constexpr int L_FLAG = 512;
constexpr unsigned long long DEADLINE_TICKS = 100000000ull;

#define CREC 6144
constexpr int RVAR = 5120, RS1 = RVAR + 16;
static_assert(CREC <= 6144 && CREC >= RS1, "record size");

constexpr int S_RAW = 0;
constexpr int S_WIRE = S_RAW + KBS * REC;
constexpr int S_DEC = S_WIRE + KBS * 6144;
constexpr int NS = 8;
constexpr int S_SLOT = S_DEC + 2 * REC;
constexpr int S_SNAP = S_SLOT + NS * REC;
constexpr int S_MISC = S_SNAP + 256;
constexpr int S_PUB = S_MISC + 256;
constexpr int S_CNT = S_PUB + 128;
constexpr int S_GO = S_CNT + 4;
constexpr int LDS_BYTES = S_CNT + 16;
static_assert(LDS_BYTES > 80 * 1024, "LDS");

__device__ __forceinline__ gchar* gptr(void* p) { return (gchar*)p; }
__device__ __forceinline__ gu32* ctrl_word(void* ws, int line) {
    return (gu32*)(gptr(ws) + (size_t)line * 128);
}
__device__ __forceinline__ unsigned bfi(unsigned m, unsigned a, unsigned b) { return (a & m) | (b & ~m); }
__device__ __forceinline__ unsigned wave_scan(unsigned s) {
    s += __builtin_amdgcn_update_dpp(0u, s, 0x111, 0xf, 0xf, false);
    s += __builtin_amdgcn_update_dpp(0u, s, 0x112, 0xf, 0xf, false);
    s += __builtin_amdgcn_update_dpp(0u, s, 0x114, 0xf, 0xf, false);
    s += __builtin_amdgcn_update_dpp(0u, s, 0x118, 0xf, 0xf, false);
    s += __builtin_amdgcn_update_dpp(0u, s, 0x142, 0xa, 0xf, false);
    s += __builtin_amdgcn_update_dpp(0u, s, 0x143, 0xc, 0xf, false);
    return s;
}

#define WAIT_VM_(n) asm volatile("s_waitcnt vmcnt(" #n ")" ::: "memory")
#define WAIT_VM(n) WAIT_VM_(n)
#define NCH 80
#define LB 8
#define STAG 2
#define EARLY 0
#define PNEAR 3
#define PNAP 24
#define TG 2
#define NT 10
#define ABL 0
#define OST 2
#define ROTV 32
#define TAU 640u
#define OFF 450u
#define LEAD 200u
#define WIN 4
#define GAP 8
#define WEND 0
#define SWZ 0
#define NOCH 0
#define XWIN 2
#define TMOP 63u
#define LATE 21
#define LGRAN 32u
#define ORDB 4
#define PSL 2
#define TSC(c, e) do { } while (0)
#define TS(e) do { } while (0)
#define TM(v) do { } while (0)
#define TF(e) do { } while (0)

__device__ __forceinline__ void st16_wt(gchar* base, unsigned off, v4i v) {
    asm volatile("s_nop 4\n\tglobal_store_dwordx4 %0, %1, %2 sc0 sc1\n\ts_nop 1" :: "v"(off), "v"(v), "s"(base) : "memory");
}
__device__ __forceinline__ void ld4k_lds(const gchar* base, unsigned off, unsigned lds) {
    asm volatile("s_mov_b32 m0, %2\n\ts_nop 4\n\t"
                 "global_load_lds_dwordx4 %0, %1 sc0 sc1\n\t"
                 "global_load_lds_dwordx4 %0, %1 offset:1024 sc0 sc1\n\t"
                 "global_load_lds_dwordx4 %0, %1 offset:2048 sc0 sc1\n\t"
                 "global_load_lds_dwordx4 %0, %1 offset:3072 sc0 sc1"
                 :: "v"(off), "s"(base), "s"(lds) : "memory");
}
__device__ __forceinline__ void ld2k_lds(const gchar* base, unsigned off, unsigned lds) {
    asm volatile("s_mov_b32 m0, %2\n\ts_nop 4\n\t"
                 "global_load_lds_dwordx4 %0, %1 sc0 sc1\n\t"
                 "global_load_lds_dwordx4 %0, %1 offset:1024 sc0 sc1"
                 :: "v"(off), "s"(base), "s"(lds) : "memory");
}
__device__ __forceinline__ void ld4_lds(const gchar* base, unsigned off, unsigned lds, unsigned long long lanes) {
    asm volatile("s_mov_b32 m0, %2\n\ts_mov_b64 exec, %3\n\ts_nop 3\n\t"
                 "global_load_lds_dword %0, %1 sc0 sc1\n\ts_mov_b64 exec, -1"
                 :: "v"(off), "s"(base), "s"(lds), "s"(lanes) : "memory");
}
__device__ __forceinline__ unsigned lds_get(unsigned addr) {
    unsigned v;
    asm volatile("ds_read_b32 %0, %1\n\ts_waitcnt lgkmcnt(0)" : "=v"(v) : "v"(addr) : "memory");
    return v;
}
__device__ __forceinline__ void lds_put(unsigned addr, unsigned v) {
    asm volatile("s_mov_b64 exec, 1\n\tds_write_b32 %0, %1\n\ts_mov_b64 exec, -1" :: "v"(addr), "v"(v) : "memory");
}
__device__ __forceinline__ void lds_inc(unsigned addr) {
    asm volatile("s_mov_b64 exec, 1\n\tds_add_u32 %0, %1\n\ts_mov_b64 exec, -1" :: "v"(addr), "v"(1u) : "memory");
}
constexpr unsigned SPIN_MAX = 1u << 25;
__device__ __forceinline__ void lds_barrier() {
    asm volatile("s_waitcnt lgkmcnt(0)\n\ts_barrier" ::: "memory");
}

enum : unsigned { C_ITEM = 0x100u, C_PLAIN = 0x20u, C_OWN = 0x40u, C_DEAD = 0x200u, C_RDY = 0x400u };

__global__ __launch_bounds__(NTHREADS)
void fused_gemm_kernel(const unsigned char* __restrict__ xs,
                       const unsigned char* __restrict__ wsrc,
                       const float* __restrict__ sx, const float* __restrict__ sw,
                       float* __restrict__ out, void* const* __restrict__ ws_all, int rank) {
    __shared__ __attribute__((aligned(16))) char smem[LDS_BYTES];
    const int tid = threadIdx.x;
    const int lane = tid & 63;
    const int wave = __builtin_amdgcn_readfirstlane(tid >> 6);
    const int r32 = lane & 31, h = lane >> 5;
    const unsigned long long t0 = __builtin_amdgcn_s_memrealtime();

    const int d = (int)blockIdx.x;
    const int grp = d & 7;
    const int strip = grp * GW + (d >> 3);
    constexpr int BS = 1 << LB;
    const int dd = (d >> LB << LB) | ((d - ROTV * rank) & (BS - 1));
    auto slot_of = [&](int src, int r) { return (unsigned)((strip * NR + src) * KBS + ((SWZ & 1) ? (r + (d >> 1)) & 3 : r)) * (unsigned)REC; };
    const unsigned sx4 = (SWZ & 2) ? (unsigned)(d & 1) << 12 : 0u;
    const int ph = (d + wave) % 7;
    auto rot = [&](int i) { int e = i + ph; return e >= 7 ? e - 7 : e; };

    const unsigned lds0 = (unsigned)(size_t)(__attribute__((address_space(3))) char*)smem;
    void* ws_me;
    unsigned call, par_off;
    auto whoami = [&]() {
        ws_me = ws_all[rank];
        unsigned v;
        static_assert(L_DONE == 0, "counter offset");
        asm volatile("s_load_dword %0, %1, 0x0\n\ts_waitcnt lgkmcnt(0)" : "=s"(v) : "s"(ws_me));
        call = v / NWG + 1u;
        par_off = (unsigned)CTRL_BYTES + (call & 1u) * (unsigned)PAR_BYTES;
    };

    if (wave >= 8) {
        __builtin_amdgcn_s_setprio(3);
        const bool xpusher = dd >= NWG / 2;
        const int xb = ((dd & 7) << 2 | (dd >> 6 & 1) << 1 | (wave & 1)) * 2;
        const int xdst = (rank + ((dd >> 3) ^ (wave << 2))) & 7;
        v4i xin[8];
        if (xpusher) {
            const unsigned char* xp = xs + (xdst * MS + (xb & 15) * 32 + r32) * KS + (xb >> 4) * BK + h * 16;
#pragma unroll
            for (int i = 0; i < 8; ++i) xin[i] = *reinterpret_cast<const v4i*>(xp + (i >> 2) * 32 * KS + (i & 3) * 32);
        }
        gchar* xws = gptr(ws_all[xdst]);
        whoami();
        TM(tB);
        if (xpusher) {
            asm volatile("" :: "v"(xin[0]), "v"(xin[1]), "v"(xin[2]), "v"(xin[3]),
                               "v"(xin[4]), "v"(xin[5]), "v"(xin[6]), "v"(xin[7]));
#pragma unroll
            for (int i = 0; i < 8; ++i) {
                WAIT_VM(XWIN);
                st16_wt(xws, par_off + rank * KBS * XITEM + xb * 4096 + i * 1024 + lane * 16, xin[i]);
            }
            WAIT_VM(0);
            if (lane == 0)
                __hip_atomic_fetch_add((gu32*)(xws + (L_XCNT + rank) * 128), 1u, __ATOMIC_RELAXED, __HIP_MEMORY_SCOPE_SYSTEM);
        }
        TM(tC);
        if (wave == 8) {
            const unsigned fline = (L_FLAG + (call & 1u) * NWG + strip) * 128;
            if (lane < 36) *reinterpret_cast<unsigned*>(smem + S_PUB + lane * 4) = 0u;
            lds_barrier();
            lds_barrier();
            lds_barrier();
            lds_barrier();
            lds_barrier();
            const unsigned woff = lane < 32 ? fline + lane * 4 : lane < 40 ? (L_XCNT + lane - 32) * 128 : L_TMO * 128;
            const unsigned thr = lane < 32 ? call << 1 : lane < 40 ? 32u * call : call;
            const gchar* wg = gptr(ws_me) + par_off + X_BYTES;
            const unsigned myslot = slot_of((lane >> 2) & 7, lane & 3);
            unsigned long long watch = 0x1ffffffffffull;
            unsigned taken = 15u << (4 * rank), q = 0, polls = 0;
            ld4_lds(gptr(ws_me), woff, lds0 + S_SNAP, watch);
#pragma unroll 1
            while (q < 32u) {
                WAIT_VM(0);
                const unsigned pv = lds_get(lds0 + S_SNAP + lane * 4);
                const unsigned used = __builtin_amdgcn_readfirstlane(lds_get(lds0 + S_CNT)) >> 3;
                const unsigned long long ok = __ballot((int)(pv - thr) >= 0);
                const unsigned pl = (unsigned)__ballot(pv & 1u);
                const bool late = __builtin_amdgcn_s_memrealtime() - t0 > DEADLINE_TICKS;
                if (late || ((ok >> 40) & 1u)) {
                    if (late && lane == 0)
                        __hip_atomic_store(ctrl_word(ws_me, L_TMO), call, __ATOMIC_RELAXED, __HIP_MEMORY_SCOPE_SYSTEM);
                    lds_put(lds0 + S_PUB + q * 4, C_DEAD);
                    break;
                }
                if (q == 0u && ((ok >> 32) & 0xffu) == 0xffu) {
                    __builtin_amdgcn_fence(__ATOMIC_ACQUIRE, "");
                    WAIT_VM(0);
                    if (lane < KBS) *reinterpret_cast<volatile unsigned*>(smem + S_PUB + lane * 4) = C_ITEM | C_RDY | C_OWN | (unsigned)(4 * rank + lane);
                    q = KBS;
                    TSC(lane == 0, 6);
                    WAIT_VM(0);
                    watch = 0x100ffffffffull;
                }
                unsigned rem = q ? (unsigned)ok & ~taken : 0u;
                unsigned n = 0, code[4] = {0u, 0u, 0u, 0u}, anyp = 0;
#pragma unroll
                for (int i = 0; i < 4; ++i) {
                    if (rem && q + i < used + NS) {
                        const unsigned it = __builtin_ctz(rem);
                        rem &= rem - 1u;
                        taken |= 1u << it;
                        const gchar* rp = wg + (unsigned)__builtin_amdgcn_readlane(myslot, (ABL & 2) ? 4 * rank : it);
                        const unsigned sl = lds0 + S_SLOT + ((q + i) & (NS - 1)) * REC;
                        if (!(ABL & 16)) ld4k_lds(rp + sx4, lane * 16, sl);
                        const unsigned p = (pl >> it) & 1u;
                        if (p) ld4k_lds(rp + (4096u ^ sx4), lane * 16, sl + 4096);
                        else if (!(ABL & 16)) ld2k_lds(rp + (4096u ^ sx4), lane * 16, sl + 4096);
                        anyp |= p;
                        code[i] = C_ITEM | it | p * C_PLAIN;
                        lds_put(lds0 + S_PUB + (q + i) * 4, code[i]);
                        n = i + 1;
                    } else {
                        rem = 0u;
                    }
                }
                ld4_lds(gptr(ws_me), woff, lds0 + S_SNAP, (++polls & TMOP) ? watch & ~(1ull << 40) : watch);
                if (anyp) WAIT_VM(1);
#pragma unroll
                for (int i = 0; i < 4; ++i) {
                    if (i < (int)n) {
                        const unsigned left = n - 1u - i;
                        if (left == 3u) { WAIT_VM(19); } else if (left == 2u) { WAIT_VM(13); } else if (left == 1u) { WAIT_VM(7); } else { WAIT_VM(1); }
                        lds_put(lds0 + S_PUB + (q + i) * 4, code[i] | C_RDY);
                    }
                }
                q += n;
            }
            WAIT_VM(0);
            return;
        }
        gchar* dws[NR - 1];
#pragma unroll
        for (int i = 0; i < NR - 1; ++i) dws[i] = gptr(ws_all[(rank + 1 + rot(i)) & 7]);
        gchar* sigb = dws[0];
#pragma unroll
        for (int i = 1; i < NR - 1; ++i) sigb = lane == i ? dws[i] : sigb;
        sigb += (L_FLAG + (call & 1u) * NWG + strip) * 128 + rank * KBS * 4;
        __builtin_amdgcn_s_waitcnt(0);
        lds_barrier();
        lds_barrier();
        lds_barrier();
#pragma unroll 1
        for (int r = 0; r < KBS; ++r) {
            const int seq = (dd >> LB << (LB + 2)) | (r << LB) | (dd & (BS - 1));
            if (r == (dd < (NT << TG) ? 1 : 0)) {
                lds_barrier();
                lds_barrier();
            }
            auto due_of = [&](int c) {
                int n = 0;
#pragma unroll
                for (int e = 0; e < 6; ++e) n += min(max(seq + (c - e) * GAP + (e < c ? 1 : 0), 0), NWG * KBS);
                return OFF + (unsigned)n * TAU / 384u;
            };
            const unsigned due = due_of(0);
            if (seq >= NCH && seq < NWG * KBS - NOCH) {
#pragma unroll 1
                while ((unsigned)(__builtin_amdgcn_s_memrealtime() - t0) + LEAD < due) __builtin_amdgcn_s_sleep(8);
#pragma unroll 1
                for (;;) {
                    const unsigned cv = __hip_atomic_load(ctrl_word(ws_me, L_GO + dd) + r, __ATOMIC_RELAXED, __HIP_MEMORY_SCOPE_SYSTEM);
                    if (__builtin_amdgcn_readfirstlane(cv) == call) break;
                    if (__builtin_amdgcn_s_memrealtime() - t0 > DEADLINE_TICKS) {
                        if (lane == 0)
                            __hip_atomic_store(ctrl_word(ws_me, L_TMO), call, __ATOMIC_RELAXED, __HIP_MEMORY_SCOPE_SYSTEM);
                        return;
                    }
                    __builtin_amdgcn_s_sleep(PSL);
                }
            }
            const unsigned slot = par_off + (unsigned)X_BYTES + slot_of(rank, r) + lane * 16;
            const unsigned len = __builtin_amdgcn_readfirstlane(*reinterpret_cast<const unsigned*>(smem + S_MISC + 192 + r * 4));
            const char* src = smem + (len == (unsigned)REC ? S_RAW + r * REC : S_WIRE + r * 6144) + lane * 16;
#pragma unroll 1
            for (unsigned c = 0; c < len; c += 1024u) {
                const unsigned duc = due_of(min((int)(c >> 10), 5));
#pragma unroll 1
                while ((unsigned)(__builtin_amdgcn_s_memrealtime() - t0) < duc) __builtin_amdgcn_s_sleep(1);
                if (seq + (int)(c >> 10) * GAP < NWG * KBS + WEND) WAIT_VM(WIN);
                if (c + lane * 16 < len) {
                    const v4i v = *reinterpret_cast<const v4i*>(src + c);
#pragma unroll
                    for (int i = 0; i < 7; ++i) st16_wt(dws[i], slot + (c ^ sx4), v);
                }
            }
            TM(tE);
            if (EARLY < 0) WAIT_VM(LATE);
            if (seq + NCH < NWG * KBS && lane == 0)
                __hip_atomic_store(ctrl_word(ws_me, L_GO + ((seq + NCH) & (NWG - 1))) + ((seq + NCH) >> 8), call, __ATOMIC_RELAXED, __HIP_MEMORY_SCOPE_SYSTEM);
            WAIT_VM(0);
            TM(tF);
            if (lane < NR - 1)
                __hip_atomic_store((gu32*)(sigb + r * 4), (call << 1) | (len == (unsigned)REC),
                                   __ATOMIC_RELAXED, __HIP_MEMORY_SCOPE_SYSTEM);
        }
        return;
    }

    unsigned wv[16];
    {
        const unsigned char* wp = wsrc + (size_t)((tid >> 4) * 16) * GN + strip * BN + (tid & 15) * 4;
#pragma unroll
        for (int j = 0; j < 16; ++j) wv[j] = *reinterpret_cast<const unsigned*>(wp + (size_t)j * GN);
    }
    whoami();
    TS(0);

    unsigned* s_sel = reinterpret_cast<unsigned*>(smem + S_MISC);
    unsigned* s_tot = s_sel + 16;
    {
        const int kc = tid >> 4, n0 = (tid & 15) * 4;
        char* dst = smem + S_RAW + (kc >> 3) * REC + (((n0 >> 5) * 4 + ((kc >> 1) & 3)) * 64 + (kc & 1) * 32 + (n0 & 31)) * 16;
        v4u o[4];
#pragma unroll
        for (int q = 0; q < 4; ++q) {
            const unsigned a0 = wv[4 * q], a1 = wv[4 * q + 1], a2 = wv[4 * q + 2], a3 = wv[4 * q + 3];
            const unsigned u0 = __builtin_amdgcn_perm(a1, a0, 0x05010400u);
            const unsigned u1 = __builtin_amdgcn_perm(a1, a0, 0x07030602u);
            const unsigned u2 = __builtin_amdgcn_perm(a3, a2, 0x05010400u);
            const unsigned u3 = __builtin_amdgcn_perm(a3, a2, 0x07030602u);
            o[0][q] = __builtin_amdgcn_perm(u2, u0, 0x05040100u);
            o[1][q] = __builtin_amdgcn_perm(u2, u0, 0x07060302u);
            o[2][q] = __builtin_amdgcn_perm(u3, u1, 0x05040100u);
            o[3][q] = __builtin_amdgcn_perm(u3, u1, 0x07060302u);
        }
#pragma unroll
        for (int i = 0; i < 4; ++i) *reinterpret_cast<v4u*>(dst + i * 16) = o[i];
        if (tid < 256) *reinterpret_cast<v4u*>(smem + S_WIRE + (tid >> 6) * 6144 + RVAR + lane * 16) = v4u{0u, 0u, 0u, 0u};
        if (tid < 16) {
            unsigned sel = 0x0c0c0c0cu, n = 0;
#pragma unroll
            for (int b = 0; b < 4; ++b)
                if ((tid >> b) & 1) { sel = bfi(0xffu << (8 * n), (unsigned)b << (8 * n), sel); ++n; }
            s_sel[tid] = sel;
            if (tid < KBS) s_tot[36 + tid] = 0u;
        }
    }
    lds_barrier();
#pragma unroll
    for (int part = 0; part < 2; ++part) {
        const int kb0 = part, kb1 = part ? KBS : 1;
        unsigned bits[KBS], m2[KBS], sc[KBS];
#pragma unroll
        for (int kb = kb0; kb < kb1; ++kb) {
            const v4u c = *reinterpret_cast<const v4u*>(smem + S_RAW + kb * REC + tid * 16);
            unsigned S[4], n1 = 0, n2 = 0, mp = 0, bb = 0, b = 0;
            m2[kb] = 0;
#pragma unroll
            for (int q = 0; q < 4; ++q) {
                constexpr unsigned U = 0x01010101u;
                unsigned t = ((c[q] & 0x7f7f7f7fu) + (128u - 38u) * U) & 0x7f7f7f7fu;
                const unsigned fl = (t + (128u - 31u) * U) & 0x80808080u;
                const unsigned l3 = (t + (128u - 98u) * U) & 0x80808080u;
                t = bfi((fl << 1) - (fl >> 7), (21u * U + (t & U)) | (l3 >> 4), t);
                const unsigned u = t + (128u - 15u) * U, g = u & 0x80808080u;
                const unsigned h27 = ((t + (128u - 27u) * U) >> 7) & U;
                S[q] = bfi((g >> 3) - (g >> 7), u | (h27 * 3u), __builtin_amdgcn_perm(0x0b0c0e0eu, 0x0d0d0d0du, (t >> 1) & 0x07070707u));
                const unsigned F2 = (S[q] + 3u * U) & 0x10101010u;
                const unsigned F1 = ((S[q] + 5u * U) ^ (S[q] + 2u * U)) & 0x10101010u;
                const unsigned sel2 = s_sel[__builtin_amdgcn_udot4(F2, 0x08040201u, 0u, false) >> 4];
                const unsigned sel1 = s_sel[__builtin_amdgcn_udot4(F1, 0x08040201u, 0u, false) >> 4];
                bb |= __builtin_amdgcn_udot4(__builtin_amdgcn_perm(0u, (t + h27) & 0x03030303u, sel2), 0x40100401u, 0u, false) << (2u * n2);
                b |= __builtin_amdgcn_udot4(__builtin_amdgcn_perm(0u, bfi(t >> 3, t ^ (t >> 4), t >> 2) & U, sel1), 0x08040201u, 0u, false) << n1;
                n2 += __builtin_popcount(F2);
                n1 += __builtin_popcount(F1);
                m2[kb] |= ((t + (128u - 29u) * U) & 0x80808080u) >> (4 - q);
                mp |= (fl & ~l3) >> (4 - q);
            }
            bits[kb] = bb | (b << ((2u * n2) & 31u));
            n1 += 2u * n2;
            char* rec = smem + S_WIRE + kb * 6144;
            *reinterpret_cast<v2u*>(rec + tid * 8) = v2u{S[3] | (S[2] << 4), S[1] | (S[0] << 4)};
            const unsigned Y = (c[0] & 0x80808080u) | ((c[1] & 0x80808080u) >> 4) | ((c[2] & 0x80808080u) >> 2) | ((c[3] & 0x80808080u) >> 6);
            *reinterpret_cast<unsigned short*>(rec + 4096 + tid * 2) = (unsigned short)(Y | (Y >> 17));
            n2 = __builtin_popcount(m2[kb]);
            if (n2 > 8u || n1 > 32u) __hip_atomic_fetch_add(s_tot + 36 + kb, 16u, __ATOMIC_RELAXED, __HIP_MEMORY_SCOPE_WORKGROUP);
            while (mp) {
                const unsigned bit = __builtin_ctz(mp);
                mp &= mp - 1u;
                const unsigned pos = tid * 16 + ((bit & 7u) - 3u) * 4 + (bit >> 3);
                const unsigned ix = __hip_atomic_fetch_add(s_tot + 36 + kb, 1u, __ATOMIC_RELAXED, __HIP_MEMORY_SCOPE_WORKGROUP);
                if (ix < 7u)
                    *reinterpret_cast<unsigned*>(rec + RS1 + ix * 4) = pos << 8 | (unsigned char)smem[S_RAW + kb * REC + pos];
            }
            const unsigned cnt = n1 | (n2 << 16);
            const unsigned in = wave_scan(cnt);
            if (lane == 63) s_tot[kb * 8 + wave] = in;
            sc[kb] = in - cnt;
        }
        lds_barrier();
#pragma unroll
        for (int kb = kb0; kb < kb1; ++kb) {
            const unsigned np = __builtin_amdgcn_readfirstlane(s_tot[36 + kb]);
            unsigned base = np * 32u, tot = base, mine = 0;
#pragma unroll
            for (int w = 0; w < 8; ++w) {
                const unsigned v = __builtin_amdgcn_readfirstlane(s_tot[kb * 8 + w]);
                const unsigned z = (v & 0xffffu) + (v >> 16) * 4u;
                tot += z;
                if (w < wave) base += z;
                if (w == wave) mine = v & 0xffffu;
            }
            const unsigned rlen = np > 7u ? (unsigned)REC : RS1 + ((tot + 7u) >> 3);
            if (tid == 0) s_tot[32 + kb] = rlen > (unsigned)CREC ? (unsigned)REC : (rlen + LGRAN - 1u) & ~(LGRAN - 1u);
            if (rlen > (unsigned)CREC) continue;
            char* rec = smem + S_WIRE + kb * 6144;
            if (lane == 0) *reinterpret_cast<unsigned short*>(rec + RVAR + wave * 2) = (unsigned short)(base | np << 13);
            auto put = [&](unsigned v, unsigned at) {
                const unsigned long long pk = (unsigned long long)v << (at & 31u);
                unsigned* p = reinterpret_cast<unsigned*>(rec + RS1 + (at >> 5) * 4u);
                __hip_atomic_fetch_or(p, (unsigned)pk, __ATOMIC_RELAXED, __HIP_MEMORY_SCOPE_WORKGROUP);
                __hip_atomic_fetch_or(p + 1, (unsigned)(pk >> 32), __ATOMIC_RELAXED, __HIP_MEMORY_SCOPE_WORKGROUP);
            };
            put(bits[kb], base + (sc[kb] & 0xffffu));
            unsigned m = m2[kb];
            if (m) {
                unsigned nib = 0, sh = 0;
                do {
                    const unsigned bit = __builtin_ctz(m);
                    m &= m - 1u;
                    const unsigned cb = (unsigned char)smem[S_RAW + kb * REC + tid * 16 + ((bit & 7u) - 3u) * 4 + (bit >> 3)] & 0x7fu;
                    nib |= (cb > 39u ? 0u : (39u - cb) >> 1) << sh;
                    sh += 4u;
                } while (m);
                put(nib, base + mine + (sc[kb] >> 16) * 4u);
            }
        }
        lds_barrier();
        if (part == 0) TS(1);
    }

    v16f acc[2][2];
#pragma unroll
    for (int a = 0; a < 2; ++a)
#pragma unroll
        for (int b = 0; b < 2; ++b)
#pragma unroll
            for (int i = 0; i < 16; ++i) acc[a][b][i] = 0.f;

    const gchar* xg = gptr(ws_me) + par_off + wave * 8192 + lane * 16;
    unsigned spin = 0u;
    auto wait_pub = [&](unsigned q, unsigned need) {
        unsigned c;
#pragma unroll 1
        for (;;) {
            c = __builtin_amdgcn_readfirstlane(lds_get(lds0 + S_PUB + q * 4));
            if (c & need) break;
            if ((c & C_DEAD) || ++spin > SPIN_MAX) {
                spin = SPIN_MAX;
                c = C_ITEM | C_RDY | C_OWN;
                break;
            }
            __builtin_amdgcn_s_sleep(1);
        }
        return c;
    };
    auto load_x = [&](v4i (&xf)[8], unsigned c) {
        const gchar* xp = xg + (c & 31u) * (unsigned)XITEM;
#pragma unroll
        for (int i = 0; i < 8; ++i) { if (ABL & 32) asm volatile("" : "=v"(xf[i])); else xf[i] = *(const gv4i*)((ABL & 1) ? xg + i * 1024 : xp + i * 1024); }
        asm volatile("" ::: "memory");
    };
    auto unpack = [&](unsigned c, unsigned q) {
        if ((c & C_OWN) | (ABL & 4)) return;
        const char* rec = smem + S_SLOT + (q & (NS - 1)) * REC;
        char* dec = smem + S_DEC + (q & 1u) * REC + tid * 16;
        if (c & C_PLAIN) {
            *reinterpret_cast<v4u*>(dec) = *reinterpret_cast<const v4u*>(rec + tid * 16);
            return;
        }
        const v2u nb = *reinterpret_cast<const v2u*>(rec + tid * 8);
        const unsigned wsg = *reinterpret_cast<const unsigned short*>(rec + 4096 + tid * 2);
        const unsigned hd = *reinterpret_cast<const unsigned short*>(rec + RVAR + wave * 2);
        constexpr unsigned U = 0x01010101u;
        const unsigned T[4] = {(nb[1] >> 4) & 15u * U, nb[1] & 15u * U, (nb[0] >> 4) & 15u * U, nb[0] & 15u * U};
        unsigned sel2[4], sel1[4], cum2[4], cum1[4], mk[4], n2 = 0, n1 = 0;
#pragma unroll
        for (int k = 0; k < 4; ++k) {
            const unsigned a5 = T[k] + 5u * U, E = a5 & 16u * U;
            const unsigned F2 = ((T[k] + 3u * U) >> 2) & 4u * U;
            const unsigned F1 = ((a5 ^ (T[k] + 2u * U)) >> 2) & 4u * U;
            asm("v_mad_u32_u24 %0, %1, %2, %1" : "=v"(sel2[k]) : "v"(F2), "s"(0x404040u));
            asm("v_mad_u32_u24 %0, %1, %2, %1" : "=v"(sel1[k]) : "v"(F1), "s"(0x404040u));
            mk[k] = (E >> 1) - (E >> 4);
            cum2[k] = n2;
            cum1[k] = n1;
            n2 += __builtin_popcount(F2);
            n1 += __builtin_popcount(F1);
        }
        n1 += 2u * n2;
        const unsigned in = hd + wave_scan(n1);
        const unsigned e0 = (in - n1) & 8191u;
        const unsigned* sp = reinterpret_cast<const unsigned*>(rec + RS1 + (e0 >> 5) * 4u);
        const unsigned bits = __builtin_amdgcn_alignbit(sp[1], sp[0], e0 & 31u);
        const unsigned bits1 = bits >> ((2u * n2) & 31u);
        const unsigned X = wsg | (wsg << 17);
        const unsigned G[4] = {X, X << 4, X << 2, X << 6};
        v4u o;
        unsigned m = 0;
#pragma unroll
        for (int k = 0; k < 4; ++k) {
            unsigned b4 = __builtin_amdgcn_ubfe(bits, 2u * cum2[k], 8u);
            b4 = (b4 | (b4 << 12)) & 0x000f000fu;
            b4 = (b4 | (b4 << 6)) & 0x03030303u;
            const unsigned x = ((__builtin_amdgcn_ubfe(bits1, cum1[k], 4u) * 0x204081u) & U) * 3u;
            const unsigned tb = __builtin_amdgcn_perm(0x32201926u, 0x291d2735u, (T[k] - __builtin_amdgcn_perm(x, 0u, sel1[k])) & mk[k]);
            const unsigned mag = T[k] + tb + __builtin_amdgcn_perm(b4, 0u, sel2[k]);
            o[k] = mag | (G[k] & 0x80808080u);
            m |= ((mag + 61u * U) & 0x80808080u) >> (4 - k);
        }
        *reinterpret_cast<v4u*>(dec) = o;
        n2 = __builtin_popcount(m);
        const unsigned i2 = ((unsigned)__builtin_amdgcn_readlane((int)in, 63) + (wave_scan(n2) - n2) * 4u) & 8191u;
        if (m) {
            const unsigned* s2 = reinterpret_cast<const unsigned*>(rec + RS1 + (i2 >> 5) * 4u);
            unsigned nib = __builtin_amdgcn_alignbit(s2[1], s2[0], i2 & 31u);
            do {
                const unsigned bit = __builtin_ctz(m);
                m &= m - 1u;
                const unsigned v = nib & 15u;
                nib >>= 4;
                __hip_atomic_fetch_sub(reinterpret_cast<unsigned*>(dec - 12 + (bit & 7u) * 4u), (v ? 29u + 2u * v : 0u) << (bit & 24u),
                                       __ATOMIC_RELAXED, __HIP_MEMORY_SCOPE_WORKGROUP);
            } while (m);
        }
        if ((unsigned)lane < (hd >> 13)) {
            const unsigned pe = *reinterpret_cast<const unsigned*>(rec + RS1 + lane * 4);
            if ((pe >> 18) == (unsigned)wave) smem[S_DEC + (q & 1u) * REC + (pe >> 8)] = (char)pe;
        }
    };
    auto step = [&](v4i (&xc)[8], v4i (&xn)[8], unsigned q, unsigned& c) {
        TF(0);
#pragma unroll 1
        for (;;) {
            if (__builtin_amdgcn_readfirstlane(lds_get(lds0 + S_CNT)) >= 8u * q + 8u) break;
            if (++spin > SPIN_MAX) break;
        }
        TF(1);
        unsigned cn = q < 31u ? __builtin_amdgcn_readfirstlane(lds_get(lds0 + S_PUB + q * 4 + 4)) : 0u;
        const bool ufirst = (wave & ORDB) && (cn & C_RDY);
        if (ufirst) unpack(cn, q + 1u);
        TF(2);
        asm volatile("" :: "v"(xc[0]), "v"(xc[1]), "v"(xc[2]), "v"(xc[3]), "v"(xc[4]), "v"(xc[5]), "v"(xc[6]), "v"(xc[7]));
        const bool early = cn & C_ITEM;
        if (early) load_x(xn, cn);
        else asm volatile("" : "=v"(xn[0]), "=v"(xn[1]), "=v"(xn[2]), "=v"(xn[3]), "=v"(xn[4]), "=v"(xn[5]), "=v"(xn[6]), "=v"(xn[7]));
        TF(3);
        {
            const char* lb = smem + ((c & C_OWN) ? S_RAW + (c & 3u) * REC : S_DEC + (q & 1u) * REC) + lane * 16;
#pragma unroll
            for (int ks = 0; ks < 2; ++ks) {
                v8i bf[2];
#pragma unroll
                for (int i = 0; i < 2; ++i) {
                    const v4i b0 = *reinterpret_cast<const v4i*>(lb + (i * 4 + ks * 2) * 1024);
                    const v4i b1 = *reinterpret_cast<const v4i*>(lb + (i * 4 + ks * 2 + 1) * 1024);
                    bf[i] = __builtin_shufflevector(b0, b1, 0, 1, 2, 3, 4, 5, 6, 7);
                }
                if (ks == 1 && ufirst) lds_inc(lds0 + S_CNT);
#pragma unroll
                for (int a = 0; a < ((ABL & 8) ? 0 : 2); ++a) {
                    const v8i xa = __builtin_shufflevector(xc[a * 4 + ks * 2], xc[a * 4 + ks * 2 + 1], 0, 1, 2, 3, 4, 5, 6, 7);
#pragma unroll
                    for (int b = 0; b < 2; ++b)
                        acc[a][b] = __builtin_amdgcn_mfma_scale_f32_32x32x64_f8f6f4(xa, bf[b], acc[a][b], 0, 1, 0, 127, 0, 127);
                }
                asm volatile("" : "+v"(acc[0][0]), "+v"(acc[0][1]), "+v"(acc[1][0]), "+v"(acc[1][1]));
            }
        }
        TF(4);
        if (!ufirst && q < 31u) {
            if (!early) {
                cn = wait_pub(q + 1u, C_ITEM);
                load_x(xn, cn);
            }
            if (!(cn & C_RDY)) cn = wait_pub(q + 1u, C_RDY);
            TF(5);
            unpack(cn, q + 1u);
            lds_inc(lds0 + S_CNT);
        }
        TF(6);
        c = cn;
    };
    {
        v4i xa[8], xb[8];
        unsigned c = wait_pub(0u, C_RDY);
        load_x(xa, c);
        lds_inc(lds0 + S_CNT);
#pragma unroll 1
        for (unsigned q = 0; q < 32u; q += 2u) {
            step(xa, xb, q, c);
            step(xb, xa, q + 1u, c);
        }
    }
    const bool dead = spin >= SPIN_MAX;
    TS(4);

    {
        const float scale = dead ? __builtin_nanf("") : sx[0] * sw[0];
        float* op = out + (size_t)(wave * 64 + 4 * h) * GN + strip * BN + r32;
#pragma unroll
        for (int a = 0; a < 2; ++a)
#pragma unroll
            for (int i = 0; i < 16; ++i)
#pragma unroll
                for (int b = 0; b < 2; ++b)
                    __hip_atomic_store(op + (size_t)(a * 32 + (i & 3) + 8 * (i >> 2)) * GN + b * 32, acc[a][b][i] * scale, __ATOMIC_RELAXED, __HIP_MEMORY_SCOPE_SYSTEM);
    }
    if (tid == 63)
        __hip_atomic_fetch_add(ctrl_word(ws_me, L_DONE), 1u, __ATOMIC_RELAXED, __HIP_MEMORY_SCOPE_AGENT);
    TS(5);
}

}

extern "C" void kernel_launch(int rank, int nranks, ncclComm_t comm,
                              hipStream_t stream,
                              void* const* d_in, const int* in_sizes, int n_in,
                              void* d_out, int out_size,
                              const int* in_dtypes, int out_dtype,
                              void* const* d_out_all,
                              void* const* d_ws_all, size_t ws_bytes) {
    (void)comm; (void)in_dtypes; (void)out_dtype; (void)d_out_all;
    if (nranks != NR || (unsigned)rank >= (unsigned)NR || n_in < 4 || ws_bytes < WS_NEED ||
        in_sizes[0] != GM * KS || in_sizes[1] != KS * GN || out_size != MS * GN) {
        fprintf(stderr, "kernel_launch: unexpected problem geometry\n");
        return;
    }
    hipLaunchKernelGGL(fused_gemm_kernel, dim3(NWG), dim3(NTHREADS), 0, stream,
                       (const unsigned char*)d_in[0], (const unsigned char*)d_in[1],
                       (const float*)d_in[2], (const float*)d_in[3], (float*)d_out, d_ws_all, rank);
    HIP_CHECK(hipGetLastError());
}
```

```cpp
#include <hip/hip_runtime.h>
#include <rccl/rccl.h>
#include <cstdio>
#include <cstdint>

#define HIP_CHECK(expr) do { hipError_t _e = (expr); if (_e != hipSuccess) { \
    fprintf(stderr, "HIP error %s at %s:%d\n", hipGetErrorString(_e), __FILE__, __LINE__); \
    return; } } while (0)

typedef int v8i __attribute__((ext_vector_type(8)));
typedef int v4i __attribute__((ext_vector_type(4)));
typedef unsigned v4u __attribute__((ext_vector_type(4)));
typedef unsigned v2u __attribute__((ext_vector_type(2)));
typedef float v16f __attribute__((ext_vector_type(16)));
typedef unsigned __attribute__((aligned(1))) u32u;
#define GAS __attribute__((address_space(1)))
typedef GAS char gchar;
typedef GAS unsigned gu32;
typedef GAS v4i gv4i;

namespace {

constexpr int NR = 8;
constexpr int GM = 4096, GK = 4096, GN = 16384;
constexpr int KS = GK / NR;
constexpr int MS = GM / NR;
constexpr int BN = 64, BK = 128;
constexpr int KBS = KS / BK;
constexpr int NWG = GN / BN;
constexpr int NTHREADS = 640;
constexpr int GW = NWG / 8;

constexpr size_t CTRL_BYTES = 1u << 20;
constexpr int XITEM = MS * BK;
constexpr int REC = BK * BN;
constexpr size_t X_BYTES = (size_t)NR * KBS * XITEM;
constexpr size_t PAR_BYTES = X_BYTES + (size_t)NWG * NR * KBS * REC;
constexpr size_t WS_NEED = CTRL_BYTES + 2 * PAR_BYTES;
constexpr int L_DONE = 0;
constexpr int L_TMO = 1;
constexpr int L_TICK = 4;
constexpr int L_XCNT = 16;
constexpr int L_GO = 256;
constexpr int L_LOC = 64;
constexpr int L_GFL = 128;
constexpr int L_FLAG = 512;
constexpr unsigned long long DEADLINE_TICKS = 100000000ull;

#define CREC 6144
constexpr int RVAR = 5120, RS1 = RVAR + 16;
static_assert(CREC <= 6144 && CREC >= RS1, "record size");

constexpr int S_RAW = 0;
constexpr int S_WIRE = S_RAW + KBS * REC;
constexpr int S_DEC = S_WIRE + KBS * 6144;
constexpr int NS = 8;
constexpr int S_SLOT = S_DEC + 2 * REC;
constexpr int S_SNAP = S_SLOT + NS * REC;
constexpr int S_MISC = S_SNAP + 256;
constexpr int S_PUB = S_MISC + 256;
constexpr int S_CNT = S_PUB + 128;
constexpr int S_GO = S_CNT + 4;
constexpr int LDS_BYTES = S_CNT + 16;
static_assert(LDS_BYTES > 80 * 1024, "LDS");

__device__ __forceinline__ gchar* gptr(void* p) { return (gchar*)p; }
__device__ __forceinline__ gu32* ctrl_word(void* ws, int line) {
    return (gu32*)(gptr(ws) + (size_t)line * 128);
}
__device__ __forceinline__ unsigned bfi(unsigned m, unsigned a, unsigned b) { return (a & m) | (b & ~m); }
__device__ __forceinline__ unsigned wave_scan(unsigned s) {
    s += __builtin_amdgcn_update_dpp(0u, s, 0x111, 0xf, 0xf, false);
    s += __builtin_amdgcn_update_dpp(0u, s, 0x112, 0xf, 0xf, false);
    s += __builtin_amdgcn_update_dpp(0u, s, 0x114, 0xf, 0xf, false);
    s += __builtin_amdgcn_update_dpp(0u, s, 0x118, 0xf, 0xf, false);
    s += __builtin_amdgcn_update_dpp(0u, s, 0x142, 0xa, 0xf, false);
    s += __builtin_amdgcn_update_dpp(0u, s, 0x143, 0xc, 0xf, false);
    return s;
}

#define WAIT_VM_(n) asm volatile("s_waitcnt vmcnt(" #n ")" ::: "memory")
#define WAIT_VM(n) WAIT_VM_(n)
#define NCH 80
#define LB 8
#define STAG 2
#define EARLY 0
#define PNEAR 3
#define PNAP 24
#define TG 2
#define NT 10
#define ABL 0
#define OST 2
#define ROTV 32
#define TAU 633u
#define OFF 450u
#define LEAD 200u
#define WIN 4
#define GAP 8
#define WEND 0
#define SWZ 0
#define NOCH 0
#define XWIN 2
#define TMOP 63u
#define LATE 21
#define LGRAN 32u
#define ORDB 4
#define FLM 14
#define TNAP 2800u
#define NBF 3072
#define TD 0u
#define SEQ0 960
#define PSL 2
#define TSC(c, e) do { } while (0)
#define TS(e) do { } while (0)
#define TM(v) do { } while (0)
#define TF(e) do { } while (0)

__device__ __forceinline__ void st16_wt(gchar* base, unsigned off, v4i v) {
    asm volatile("s_nop 4\n\tglobal_store_dwordx4 %0, %1, %2 sc0 sc1\n\ts_nop 1" :: "v"(off), "v"(v), "s"(base) : "memory");
}
__device__ __forceinline__ void ld4k_lds(const gchar* base, unsigned off, unsigned lds) {
    asm volatile("s_mov_b32 m0, %2\n\ts_nop 4\n\t"
                 "global_load_lds_dwordx4 %0, %1 sc0 sc1\n\t"
                 "global_load_lds_dwordx4 %0, %1 offset:1024 sc0 sc1\n\t"
                 "global_load_lds_dwordx4 %0, %1 offset:2048 sc0 sc1\n\t"
                 "global_load_lds_dwordx4 %0, %1 offset:3072 sc0 sc1"
                 :: "v"(off), "s"(base), "s"(lds) : "memory");
}
__device__ __forceinline__ void ld2k_lds(const gchar* base, unsigned off, unsigned lds) {
    asm volatile("s_mov_b32 m0, %2\n\ts_nop 4\n\t"
                 "global_load_lds_dwordx4 %0, %1 sc0 sc1\n\t"
                 "global_load_lds_dwordx4 %0, %1 offset:1024 sc0 sc1"
                 :: "v"(off), "s"(base), "s"(lds) : "memory");
}
__device__ __forceinline__ void ld4_lds(const gchar* base, unsigned off, unsigned lds, unsigned long long lanes) {
    asm volatile("s_mov_b32 m0, %2\n\ts_mov_b64 exec, %3\n\ts_nop 3\n\t"
                 "global_load_lds_dword %0, %1 sc0 sc1\n\ts_mov_b64 exec, -1"
                 :: "v"(off), "s"(base), "s"(lds), "s"(lanes) : "memory");
}
__device__ __forceinline__ unsigned lds_get(unsigned addr) {
    unsigned v;
    asm volatile("ds_read_b32 %0, %1\n\ts_waitcnt lgkmcnt(0)" : "=v"(v) : "v"(addr) : "memory");
    return v;
}
__device__ __forceinline__ void lds_put(unsigned addr, unsigned v) {
    asm volatile("s_mov_b64 exec, 1\n\tds_write_b32 %0, %1\n\ts_mov_b64 exec, -1" :: "v"(addr), "v"(v) : "memory");
}
__device__ __forceinline__ void lds_inc(unsigned addr) {
    asm volatile("s_mov_b64 exec, 1\n\tds_add_u32 %0, %1\n\ts_mov_b64 exec, -1" :: "v"(addr), "v"(1u) : "memory");
}
constexpr unsigned SPIN_MAX = 1u << 25;
__device__ __forceinline__ void lds_barrier() {
    asm volatile("s_waitcnt lgkmcnt(0)\n\ts_barrier" ::: "memory");
}

enum : unsigned { C_ITEM = 0x100u, C_PLAIN = 0x20u, C_OWN = 0x40u, C_DEAD = 0x200u, C_RDY = 0x400u };

__global__ __launch_bounds__(NTHREADS)
void fused_gemm_kernel(const unsigned char* __restrict__ xs,
                       const unsigned char* __restrict__ wsrc,
                       const float* __restrict__ sx, const float* __restrict__ sw,
                       float* __restrict__ out, void* const* __restrict__ ws_all, int rank) {
    __shared__ __attribute__((aligned(16))) char smem[LDS_BYTES];
    const int tid = threadIdx.x;
    const int lane = tid & 63;
    const int wave = __builtin_amdgcn_readfirstlane(tid >> 6);
    const int r32 = lane & 31, h = lane >> 5;
    const unsigned long long t0 = __builtin_amdgcn_s_memrealtime();

    const int d = (int)blockIdx.x;
    const int grp = d & 7;
    const int strip = grp * GW + (d >> 3);
    constexpr int BS = 1 << LB;
    const int dd = (d >> LB << LB) | ((d - ROTV * rank) & (BS - 1));
    auto slot_of = [&](int src, int r) { return (unsigned)((strip * NR + src) * KBS + ((SWZ & 1) ? (r + (d >> 1)) & 3 : r)) * (unsigned)REC; };
    const unsigned sx4 = (SWZ & 2) ? (unsigned)(d & 1) << 12 : 0u;
    const int ph = (d + wave) % 7;
    auto rot = [&](int i) { int e = i + ph; return e >= 7 ? e - 7 : e; };

    const unsigned lds0 = (unsigned)(size_t)(__attribute__((address_space(3))) char*)smem;
    void* ws_me;
    unsigned call, par_off;
    auto whoami = [&]() {
        ws_me = ws_all[rank];
        unsigned v;
        static_assert(L_DONE == 0, "counter offset");
        asm volatile("s_load_dword %0, %1, 0x0\n\ts_waitcnt lgkmcnt(0)" : "=s"(v) : "s"(ws_me));
        call = v / NWG + 1u;
        par_off = (unsigned)CTRL_BYTES + (call & 1u) * (unsigned)PAR_BYTES;
    };

    if (wave >= 8) {
        __builtin_amdgcn_s_setprio(3);
        const bool xpusher = dd >= NWG / 2;
        const int xb = ((dd & 7) << 2 | (dd >> 6 & 1) << 1 | (wave & 1)) * 2;
        const int xdst = (rank + ((dd >> 3) ^ (wave << 2))) & 7;
        v4i xin[8];
        if (xpusher) {
            const unsigned char* xp = xs + (xdst * MS + (xb & 15) * 32 + r32) * KS + (xb >> 4) * BK + h * 16;
#pragma unroll
            for (int i = 0; i < 8; ++i) xin[i] = *reinterpret_cast<const v4i*>(xp + (i >> 2) * 32 * KS + (i & 3) * 32);
        }
        gchar* xws = gptr(ws_all[xdst]);
        whoami();
        TM(tB);
        if (xpusher) {
            asm volatile("" :: "v"(xin[0]), "v"(xin[1]), "v"(xin[2]), "v"(xin[3]),
                               "v"(xin[4]), "v"(xin[5]), "v"(xin[6]), "v"(xin[7]));
#pragma unroll
            for (int i = 0; i < 8; ++i) {
                WAIT_VM(XWIN);
                st16_wt(xws, par_off + rank * KBS * XITEM + xb * 4096 + i * 1024 + lane * 16, xin[i]);
            }
            WAIT_VM(0);
            if (lane == 0)
                __hip_atomic_fetch_add((gu32*)(xws + (L_XCNT + rank) * 128), 1u, __ATOMIC_RELAXED, __HIP_MEMORY_SCOPE_SYSTEM);
        }
        TM(tC);
        if (wave == 8) {
            const unsigned fline = (L_FLAG + (call & 1u) * NWG + strip) * 128;
            if (lane < 36) *reinterpret_cast<unsigned*>(smem + S_PUB + lane * 4) = 0u;
            lds_barrier();
            lds_barrier();
            lds_barrier();
            lds_barrier();
            lds_barrier();
            static_assert(ROTV % 8 == 0 && SEQ0 % 8 == 0 && SEQ0 >= 3 * NWG && LB == 8, "groups");
            const bool single = (lane & 3) * NWG + ((d - ROTV * (lane >> 2)) & (NWG - 1)) >= SEQ0;
            const unsigned gline = (L_GFL + (call & 1u) * GW + (d >> 3)) * 128;
            const unsigned woff = lane < 32 ? (single ? fline : gline) + lane * 4 : lane < 40 ? (L_XCNT + lane - 32) * 128 : L_TMO * 128;
            const unsigned thr = lane < 32 ? call << 8 : lane < 40 ? 32u * call : call;
            const gchar* wg = gptr(ws_me) + par_off + X_BYTES;
            const unsigned myslot = slot_of((lane >> 2) & 7, lane & 3);
            unsigned long long watch = 0x1ffffffffffull;
            unsigned taken = 15u << (4 * rank), q = 0, polls = 0;
            ld4_lds(gptr(ws_me), woff, lds0 + S_SNAP, watch);
#pragma unroll 1
            while (q < 32u) {
                WAIT_VM(0);
                const unsigned pv = lds_get(lds0 + S_SNAP + lane * 4);
                const unsigned used = __builtin_amdgcn_readfirstlane(lds_get(lds0 + S_CNT)) >> 3;
                const unsigned long long ok = __ballot((int)(pv - thr) >= 0);
                const unsigned pl = (unsigned)__ballot((pv >> grp) & 1u);
                const bool late = __builtin_amdgcn_s_memrealtime() - t0 > DEADLINE_TICKS;
                if (late || ((ok >> 40) & 1u)) {
                    if (late && lane == 0)
                        __hip_atomic_store(ctrl_word(ws_me, L_TMO), call, __ATOMIC_RELAXED, __HIP_MEMORY_SCOPE_SYSTEM);
                    lds_put(lds0 + S_PUB + q * 4, C_DEAD);
                    break;
                }
                if (q == 0u && ((ok >> 32) & 0xffu) == 0xffu) {
                    __builtin_amdgcn_fence(__ATOMIC_ACQUIRE, "");
                    WAIT_VM(0);
                    if (lane < KBS) *reinterpret_cast<volatile unsigned*>(smem + S_PUB + lane * 4) = C_ITEM | C_RDY | C_OWN | (unsigned)(4 * rank + lane);
                    q = KBS;
                    TSC(lane == 0, 6);
                    WAIT_VM(0);
                    watch = 0x100ffffffffull;
#pragma unroll 1
                    while ((unsigned)(__builtin_amdgcn_s_memrealtime() - t0) < TNAP) __builtin_amdgcn_s_sleep(32);
                }
                unsigned rem = q ? (unsigned)ok & ~taken : 0u;
                unsigned n = 0, code[4] = {0u, 0u, 0u, 0u}, anyp = 0;
#pragma unroll
                for (int i = 0; i < 4; ++i) {
                    if (rem && q + i < used + NS) {
                        const unsigned it = __builtin_ctz(rem);
                        rem &= rem - 1u;
                        taken |= 1u << it;
                        const gchar* rp = wg + (unsigned)__builtin_amdgcn_readlane(myslot, (ABL & 2) ? 4 * rank : it);
                        const unsigned sl = lds0 + S_SLOT + ((q + i) & (NS - 1)) * REC;
                        if (!(ABL & 16)) ld4k_lds(rp + sx4, lane * 16, sl);
                        const unsigned p = (pl >> it) & 1u;
                        if (p) ld4k_lds(rp + (4096u ^ sx4), lane * 16, sl + 4096);
                        else if (!(ABL & 16)) ld2k_lds(rp + (4096u ^ sx4), lane * 16, sl + 4096);
                        anyp |= p;
                        code[i] = C_ITEM | it | p * C_PLAIN;
                        lds_put(lds0 + S_PUB + (q + i) * 4, code[i]);
                        n = i + 1;
                    } else {
                        rem = 0u;
                    }
                }
                ld4_lds(gptr(ws_me), woff, lds0 + S_SNAP, (++polls & TMOP) ? watch & ~(1ull << 40) : watch);
                if (anyp) WAIT_VM(1);
#pragma unroll
                for (int i = 0; i < 4; ++i) {
                    if (i < (int)n) {
                        const unsigned left = n - 1u - i;
                        if (left == 3u) { WAIT_VM(19); } else if (left == 2u) { WAIT_VM(13); } else if (left == 1u) { WAIT_VM(7); } else { WAIT_VM(1); }
                        lds_put(lds0 + S_PUB + (q + i) * 4, code[i] | C_RDY);
                    }
                }
                q += n;
            }
            WAIT_VM(0);
            return;
        }
        gchar* dws[NR - 1];
#pragma unroll
        for (int i = 0; i < NR - 1; ++i) dws[i] = gptr(ws_all[(rank + 1 + rot(i)) & 7]);
        gchar* sigb = dws[0];
#pragma unroll
        for (int i = 1; i < NR - 1; ++i) sigb = (lane >> 2) == i ? dws[i] : sigb;
        sigb += (rank * KBS + (lane & 3)) * 4;
        int told = 0;
        __builtin_amdgcn_s_waitcnt(0);
        lds_barrier();
        lds_barrier();
        lds_barrier();
#pragma unroll 1
        for (int r = 0; r < KBS; ++r) {
            const int seq = (dd >> LB << (LB + 2)) | (r << LB) | (dd & (BS - 1));
            if (r == (dd < (NT << TG) ? 1 : 0)) {
                lds_barrier();
                lds_barrier();
            }
            auto due_of = [&](int c) {
                int n = 0;
#pragma unroll
                for (int e = 0; e < 6; ++e) n += min(max(seq + (c - e) * GAP + (e < c ? 1 : 0), 0), NWG * KBS);
                return OFF + ((unsigned)n * TAU - (unsigned)min(n, NBF) * TD) / 384u;
            };
            const unsigned due = due_of(0);
            if (seq >= NCH && seq < NWG * KBS - NOCH) {
#pragma unroll 1
                while ((unsigned)(__builtin_amdgcn_s_memrealtime() - t0) + LEAD < due) __builtin_amdgcn_s_sleep(8);
#pragma unroll 1
                for (;;) {
                    const unsigned cv = __hip_atomic_load(ctrl_word(ws_me, L_GO + dd) + r, __ATOMIC_RELAXED, __HIP_MEMORY_SCOPE_SYSTEM);
                    if (__builtin_amdgcn_readfirstlane(cv) == call) break;
                    if (__builtin_amdgcn_s_memrealtime() - t0 > DEADLINE_TICKS) {
                        if (lane == 0)
                            __hip_atomic_store(ctrl_word(ws_me, L_TMO), call, __ATOMIC_RELAXED, __HIP_MEMORY_SCOPE_SYSTEM);
                        return;
                    }
                    __builtin_amdgcn_s_sleep(PSL);
                }
            }
            const unsigned slot = par_off + (unsigned)X_BYTES + slot_of(rank, r) + lane * 16;
            const unsigned len = __builtin_amdgcn_readfirstlane(*reinterpret_cast<const unsigned*>(smem + S_MISC + 192 + r * 4));
            const char* src = smem + (len == (unsigned)REC ? S_RAW + r * REC : S_WIRE + r * 6144) + lane * 16;
#pragma unroll 1
            for (unsigned c = 0; c < len; c += 1024u) {
                const unsigned duc = due_of(min((int)(c >> 10), 5));
#pragma unroll 1
                while ((unsigned)(__builtin_amdgcn_s_memrealtime() - t0) < duc) __builtin_amdgcn_s_sleep(1);
                if (seq + (int)(c >> 10) * GAP < NWG * KBS + WEND) WAIT_VM(WIN);
                if (c + lane * 16 < len) {
                    const v4i v = *reinterpret_cast<const v4i*>(src + c);
#pragma unroll
                    for (int i = 0; i < 7; ++i) st16_wt(dws[i], slot + (c ^ sx4), v);
                }
            }
            TM(tE);
            if (EARLY < 0) WAIT_VM(LATE);
            if (seq + NCH < NWG * KBS && lane == 0)
                __hip_atomic_store(ctrl_word(ws_me, L_GO + ((seq + NCH) & (NWG - 1))) + ((seq + NCH) >> 8), call, __ATOMIC_RELAXED, __HIP_MEMORY_SCOPE_SYSTEM);
            static_assert((FLM & 12) == 12, "announcements");
            WAIT_VM(0);
            if (seq >= SEQ0) {
                if (lane < (NR - 1) * KBS && (lane & 3) == r)
                    __hip_atomic_store((gu32*)(sigb + (L_FLAG + (call & 1u) * NWG + strip) * 128), call << 8 | (unsigned)(len == (unsigned)REC) << grp,
                                       __ATOMIC_RELAXED, __HIP_MEMORY_SCOPE_SYSTEM);
            } else {
                gu32* loc = ctrl_word(ws_me, L_LOC + (d >> 3));
                if (lane == 0)
                    __hip_atomic_store(loc + r * 8 + grp, call << 1 | (len == (unsigned)REC), __ATOMIC_RELAXED, __HIP_MEMORY_SCOPE_SYSTEM);
                if (grp == 7 && ((FLM >> r) & 1)) {
                    unsigned lv;
#pragma unroll 1
                    for (;;) {
                        lv = __hip_atomic_load(loc + (lane & 31), __ATOMIC_RELAXED, __HIP_MEMORY_SCOPE_SYSTEM);
                        if (((unsigned)__ballot((lv >> 1) == (call & 0x7fffffffu)) | (0xffffff00u << (8 * r))) == ~0u) break;
                        if (__builtin_amdgcn_s_memrealtime() - t0 > DEADLINE_TICKS) {
                            if (lane == 0)
                                __hip_atomic_store(ctrl_word(ws_me, L_TMO), call, __ATOMIC_RELAXED, __HIP_MEMORY_SCOPE_SYSTEM);
                            return;
                        }
                        __builtin_amdgcn_s_sleep(4);
                    }
                    const unsigned pl = (unsigned)__ballot(lv & 1u);
                    if (lane < (NR - 1) * KBS && (lane & 3) >= told && (lane & 3) <= r)
                        __hip_atomic_store((gu32*)(sigb + (L_GFL + (call & 1u) * GW + (d >> 3)) * 128), call << 8 | (pl >> (8 * (lane & 3)) & 255u),
                                           __ATOMIC_RELAXED, __HIP_MEMORY_SCOPE_SYSTEM);
                    told = r + 1;
                }
            }
            TM(tF);
        }
        return;
    }

    unsigned wv[16];
    {
        const unsigned char* wp = wsrc + (size_t)((tid >> 4) * 16) * GN + strip * BN + (tid & 15) * 4;
#pragma unroll
        for (int j = 0; j < 16; ++j) wv[j] = *reinterpret_cast<const unsigned*>(wp + (size_t)j * GN);
    }
    whoami();
    TS(0);

    unsigned* s_sel = reinterpret_cast<unsigned*>(smem + S_MISC);
    unsigned* s_tot = s_sel + 16;
    {
        const int kc = tid >> 4, n0 = (tid & 15) * 4;
        char* dst = smem + S_RAW + (kc >> 3) * REC + (((n0 >> 5) * 4 + ((kc >> 1) & 3)) * 64 + (kc & 1) * 32 + (n0 & 31)) * 16;
        v4u o[4];
#pragma unroll
        for (int q = 0; q < 4; ++q) {
            const unsigned a0 = wv[4 * q], a1 = wv[4 * q + 1], a2 = wv[4 * q + 2], a3 = wv[4 * q + 3];
            const unsigned u0 = __builtin_amdgcn_perm(a1, a0, 0x05010400u);
            const unsigned u1 = __builtin_amdgcn_perm(a1, a0, 0x07030602u);
            const unsigned u2 = __builtin_amdgcn_perm(a3, a2, 0x05010400u);
            const unsigned u3 = __builtin_amdgcn_perm(a3, a2, 0x07030602u);
            o[0][q] = __builtin_amdgcn_perm(u2, u0, 0x05040100u);
            o[1][q] = __builtin_amdgcn_perm(u2, u0, 0x07060302u);
            o[2][q] = __builtin_amdgcn_perm(u3, u1, 0x05040100u);
            o[3][q] = __builtin_amdgcn_perm(u3, u1, 0x07060302u);
        }
#pragma unroll
        for (int i = 0; i < 4; ++i) *reinterpret_cast<v4u*>(dst + i * 16) = o[i];
        if (tid < 256) *reinterpret_cast<v4u*>(smem + S_WIRE + (tid >> 6) * 6144 + RVAR + lane * 16) = v4u{0u, 0u, 0u, 0u};
        if (tid < 16) {
            unsigned sel = 0x0c0c0c0cu, n = 0;
#pragma unroll
            for (int b = 0; b < 4; ++b)
                if ((tid >> b) & 1) { sel = bfi(0xffu << (8 * n), (unsigned)b << (8 * n), sel); ++n; }
            s_sel[tid] = sel;
            if (tid < KBS) s_tot[36 + tid] = 0u;
        }
    }
    lds_barrier();
#pragma unroll
    for (int part = 0; part < 2; ++part) {
        const int kb0 = part, kb1 = part ? KBS : 1;
        unsigned bits[KBS], m2[KBS], sc[KBS];
#pragma unroll
        for (int kb = kb0; kb < kb1; ++kb) {
            const v4u c = *reinterpret_cast<const v4u*>(smem + S_RAW + kb * REC + tid * 16);
            unsigned S[4], n1 = 0, n2 = 0, mp = 0, bb = 0, b = 0;
            m2[kb] = 0;
#pragma unroll
            for (int q = 0; q < 4; ++q) {
                constexpr unsigned U = 0x01010101u;
                unsigned t = ((c[q] & 0x7f7f7f7fu) + (128u - 38u) * U) & 0x7f7f7f7fu;
                const unsigned fl = (t + (128u - 31u) * U) & 0x80808080u;
                const unsigned l3 = (t + (128u - 98u) * U) & 0x80808080u;
                t = bfi((fl << 1) - (fl >> 7), (21u * U + (t & U)) | (l3 >> 4), t);
                const unsigned u = t + (128u - 15u) * U, g = u & 0x80808080u;
                const unsigned h27 = ((t + (128u - 27u) * U) >> 7) & U;
                S[q] = bfi((g >> 3) - (g >> 7), u | (h27 * 3u), __builtin_amdgcn_perm(0x0b0c0e0eu, 0x0d0d0d0du, (t >> 1) & 0x07070707u));
                const unsigned F2 = (S[q] + 3u * U) & 0x10101010u;
                const unsigned F1 = ((S[q] + 5u * U) ^ (S[q] + 2u * U)) & 0x10101010u;
                const unsigned sel2 = s_sel[__builtin_amdgcn_udot4(F2, 0x08040201u, 0u, false) >> 4];
                const unsigned sel1 = s_sel[__builtin_amdgcn_udot4(F1, 0x08040201u, 0u, false) >> 4];
                bb |= __builtin_amdgcn_udot4(__builtin_amdgcn_perm(0u, (t + h27) & 0x03030303u, sel2), 0x40100401u, 0u, false) << (2u * n2);
                b |= __builtin_amdgcn_udot4(__builtin_amdgcn_perm(0u, bfi(t >> 3, t ^ (t >> 4), t >> 2) & U, sel1), 0x08040201u, 0u, false) << n1;
                n2 += __builtin_popcount(F2);
                n1 += __builtin_popcount(F1);
                m2[kb] |= ((t + (128u - 29u) * U) & 0x80808080u) >> (4 - q);
                mp |= (fl & ~l3) >> (4 - q);
            }
            bits[kb] = bb | (b << ((2u * n2) & 31u));
            n1 += 2u * n2;
            char* rec = smem + S_WIRE + kb * 6144;
            *reinterpret_cast<v2u*>(rec + tid * 8) = v2u{S[3] | (S[2] << 4), S[1] | (S[0] << 4)};
            const unsigned Y = (c[0] & 0x80808080u) | ((c[1] & 0x80808080u) >> 4) | ((c[2] & 0x80808080u) >> 2) | ((c[3] & 0x80808080u) >> 6);
            *reinterpret_cast<unsigned short*>(rec + 4096 + tid * 2) = (unsigned short)(Y | (Y >> 17));
            n2 = __builtin_popcount(m2[kb]);
            if (n2 > 8u || n1 > 32u) __hip_atomic_fetch_add(s_tot + 36 + kb, 16u, __ATOMIC_RELAXED, __HIP_MEMORY_SCOPE_WORKGROUP);
            while (mp) {
                const unsigned bit = __builtin_ctz(mp);
                mp &= mp - 1u;
                const unsigned pos = tid * 16 + ((bit & 7u) - 3u) * 4 + (bit >> 3);
                const unsigned ix = __hip_atomic_fetch_add(s_tot + 36 + kb, 1u, __ATOMIC_RELAXED, __HIP_MEMORY_SCOPE_WORKGROUP);
                if (ix < 7u)
                    *reinterpret_cast<unsigned*>(rec + RS1 + ix * 4) = pos << 8 | (unsigned char)smem[S_RAW + kb * REC + pos];
            }
            const unsigned cnt = n1 | (n2 << 16);
            const unsigned in = wave_scan(cnt);
            if (lane == 63) s_tot[kb * 8 + wave] = in;
            sc[kb] = in - cnt;
        }
        lds_barrier();
#pragma unroll
        for (int kb = kb0; kb < kb1; ++kb) {
            const unsigned np = __builtin_amdgcn_readfirstlane(s_tot[36 + kb]);
            unsigned base = np * 32u, tot = base, mine = 0;
#pragma unroll
            for (int w = 0; w < 8; ++w) {
                const unsigned v = __builtin_amdgcn_readfirstlane(s_tot[kb * 8 + w]);
                const unsigned z = (v & 0xffffu) + (v >> 16) * 4u;
                tot += z;
                if (w < wave) base += z;
                if (w == wave) mine = v & 0xffffu;
            }
            const unsigned rlen = np > 7u ? (unsigned)REC : RS1 + ((tot + 7u) >> 3);
            if (tid == 0) s_tot[32 + kb] = rlen > (unsigned)CREC ? (unsigned)REC : (rlen + LGRAN - 1u) & ~(LGRAN - 1u);
            if (rlen > (unsigned)CREC) continue;
            char* rec = smem + S_WIRE + kb * 6144;
            if (lane == 0) *reinterpret_cast<unsigned short*>(rec + RVAR + wave * 2) = (unsigned short)(base | np << 13);
            auto put = [&](unsigned v, unsigned at) {
                const unsigned long long pk = (unsigned long long)v << (at & 31u);
                unsigned* p = reinterpret_cast<unsigned*>(rec + RS1 + (at >> 5) * 4u);
                __hip_atomic_fetch_or(p, (unsigned)pk, __ATOMIC_RELAXED, __HIP_MEMORY_SCOPE_WORKGROUP);
                __hip_atomic_fetch_or(p + 1, (unsigned)(pk >> 32), __ATOMIC_RELAXED, __HIP_MEMORY_SCOPE_WORKGROUP);
            };
            put(bits[kb], base + (sc[kb] & 0xffffu));
            unsigned m = m2[kb];
            if (m) {
                unsigned nib = 0, sh = 0;
                do {
                    const unsigned bit = __builtin_ctz(m);
                    m &= m - 1u;
                    const unsigned cb = (unsigned char)smem[S_RAW + kb * REC + tid * 16 + ((bit & 7u) - 3u) * 4 + (bit >> 3)] & 0x7fu;
                    nib |= (cb > 39u ? 0u : (39u - cb) >> 1) << sh;
                    sh += 4u;
                } while (m);
                put(nib, base + mine + (sc[kb] >> 16) * 4u);
            }
        }
        lds_barrier();
        if (part == 0) TS(1);
    }

    v16f acc[2][2];
#pragma unroll
    for (int a = 0; a < 2; ++a)
#pragma unroll
        for (int b = 0; b < 2; ++b)
#pragma unroll
            for (int i = 0; i < 16; ++i) acc[a][b][i] = 0.f;

    const gchar* xg = gptr(ws_me) + par_off + wave * 8192 + lane * 16;
    unsigned spin = 0u;
    auto wait_pub = [&](unsigned q, unsigned need) {
        unsigned c;
#pragma unroll 1
        for (;;) {
            c = __builtin_amdgcn_readfirstlane(lds_get(lds0 + S_PUB + q * 4));
            if (c & need) break;
            if ((c & C_DEAD) || ++spin > SPIN_MAX) {
                spin = SPIN_MAX;
                c = C_ITEM | C_RDY | C_OWN;
                break;
            }
            __builtin_amdgcn_s_sleep(1);
        }
        return c;
    };
    auto load_x = [&](v4i (&xf)[8], unsigned c) {
        const gchar* xp = xg + (c & 31u) * (unsigned)XITEM;
#pragma unroll
        for (int i = 0; i < 8; ++i) { if (ABL & 32) asm volatile("" : "=v"(xf[i])); else xf[i] = *(const gv4i*)((ABL & 1) ? xg + i * 1024 : xp + i * 1024); }
        asm volatile("" ::: "memory");
    };
    auto unpack = [&](unsigned c, unsigned q) {
        if ((c & C_OWN) | (ABL & 4)) return;
        const char* rec = smem + S_SLOT + (q & (NS - 1)) * REC;
        char* dec = smem + S_DEC + (q & 1u) * REC + tid * 16;
        if (c & C_PLAIN) {
            *reinterpret_cast<v4u*>(dec) = *reinterpret_cast<const v4u*>(rec + tid * 16);
            return;
        }
        const v2u nb = *reinterpret_cast<const v2u*>(rec + tid * 8);
        const unsigned wsg = *reinterpret_cast<const unsigned short*>(rec + 4096 + tid * 2);
        const unsigned hd = *reinterpret_cast<const unsigned short*>(rec + RVAR + wave * 2);
        constexpr unsigned U = 0x01010101u;
        const unsigned T[4] = {(nb[1] >> 4) & 15u * U, nb[1] & 15u * U, (nb[0] >> 4) & 15u * U, nb[0] & 15u * U};
        unsigned sel2[4], sel1[4], cum2[4], cum1[4], mk[4], n2 = 0, n1 = 0;
#pragma unroll
        for (int k = 0; k < 4; ++k) {
            const unsigned a5 = T[k] + 5u * U, E = a5 & 16u * U;
            const unsigned F2 = ((T[k] + 3u * U) >> 2) & 4u * U;
            const unsigned F1 = ((a5 ^ (T[k] + 2u * U)) >> 2) & 4u * U;
            asm("v_mad_u32_u24 %0, %1, %2, %1" : "=v"(sel2[k]) : "v"(F2), "s"(0x404040u));
            asm("v_mad_u32_u24 %0, %1, %2, %1" : "=v"(sel1[k]) : "v"(F1), "s"(0x404040u));
            mk[k] = (E >> 1) - (E >> 4);
            cum2[k] = n2;
            cum1[k] = n1;
            n2 += __builtin_popcount(F2);
            n1 += __builtin_popcount(F1);
        }
        n1 += 2u * n2;
        const unsigned in = hd + wave_scan(n1);
        const unsigned e0 = (in - n1) & 8191u;
        const unsigned* sp = reinterpret_cast<const unsigned*>(rec + RS1 + (e0 >> 5) * 4u);
        const unsigned bits = __builtin_amdgcn_alignbit(sp[1], sp[0], e0 & 31u);
        const unsigned bits1 = bits >> ((2u * n2) & 31u);
        const unsigned X = wsg | (wsg << 17);
        const unsigned G[4] = {X, X << 4, X << 2, X << 6};
        v4u o;
        unsigned m = 0;
#pragma unroll
        for (int k = 0; k < 4; ++k) {
            unsigned b4 = __builtin_amdgcn_ubfe(bits, 2u * cum2[k], 8u);
            b4 = (b4 | (b4 << 12)) & 0x000f000fu;
            b4 = (b4 | (b4 << 6)) & 0x03030303u;
            const unsigned x = ((__builtin_amdgcn_ubfe(bits1, cum1[k], 4u) * 0x204081u) & U) * 3u;
            const unsigned tb = __builtin_amdgcn_perm(0x32201926u, 0x291d2735u, (T[k] - __builtin_amdgcn_perm(x, 0u, sel1[k])) & mk[k]);
            const unsigned mag = T[k] + tb + __builtin_amdgcn_perm(b4, 0u, sel2[k]);
            o[k] = mag | (G[k] & 0x80808080u);
            m |= ((mag + 61u * U) & 0x80808080u) >> (4 - k);
        }
        *reinterpret_cast<v4u*>(dec) = o;
        n2 = __builtin_popcount(m);
        const unsigned i2 = ((unsigned)__builtin_amdgcn_readlane((int)in, 63) + (wave_scan(n2) - n2) * 4u) & 8191u;
        if (m) {
            const unsigned* s2 = reinterpret_cast<const unsigned*>(rec + RS1 + (i2 >> 5) * 4u);
            unsigned nib = __builtin_amdgcn_alignbit(s2[1], s2[0], i2 & 31u);
            do {
                const unsigned bit = __builtin_ctz(m);
                m &= m - 1u;
                const unsigned v = nib & 15u;
                nib >>= 4;
                __hip_atomic_fetch_sub(reinterpret_cast<unsigned*>(dec - 12 + (bit & 7u) * 4u), (v ? 29u + 2u * v : 0u) << (bit & 24u),
                                       __ATOMIC_RELAXED, __HIP_MEMORY_SCOPE_WORKGROUP);
            } while (m);
        }
        if ((unsigned)lane < (hd >> 13)) {
            const unsigned pe = *reinterpret_cast<const unsigned*>(rec + RS1 + lane * 4);
            if ((pe >> 18) == (unsigned)wave) smem[S_DEC + (q & 1u) * REC + (pe >> 8)] = (char)pe;
        }
    };
    auto step = [&](v4i (&xc)[8], v4i (&xn)[8], unsigned q, unsigned& c) {
        TF(0);
#pragma unroll 1
        for (;;) {
            if (__builtin_amdgcn_readfirstlane(lds_get(lds0 + S_CNT)) >= 8u * q + 8u) break;
            if (++spin > SPIN_MAX) break;
        }
        TF(1);
        unsigned cn = q < 31u ? __builtin_amdgcn_readfirstlane(lds_get(lds0 + S_PUB + q * 4 + 4)) : 0u;
        const bool ufirst = (wave & ORDB) && (cn & C_RDY);
        if (ufirst) unpack(cn, q + 1u);
        TF(2);
        asm volatile("" :: "v"(xc[0]), "v"(xc[1]), "v"(xc[2]), "v"(xc[3]), "v"(xc[4]), "v"(xc[5]), "v"(xc[6]), "v"(xc[7]));
        const bool early = cn & C_ITEM;
        if (early) load_x(xn, cn);
        else asm volatile("" : "=v"(xn[0]), "=v"(xn[1]), "=v"(xn[2]), "=v"(xn[3]), "=v"(xn[4]), "=v"(xn[5]), "=v"(xn[6]), "=v"(xn[7]));
        TF(3);
        {
            const char* lb = smem + ((c & C_OWN) ? S_RAW + (c & 3u) * REC : S_DEC + (q & 1u) * REC) + lane * 16;
#pragma unroll
            for (int ks = 0; ks < 2; ++ks) {
                v8i bf[2];
#pragma unroll
                for (int i = 0; i < 2; ++i) {
                    const v4i b0 = *reinterpret_cast<const v4i*>(lb + (i * 4 + ks * 2) * 1024);
                    const v4i b1 = *reinterpret_cast<const v4i*>(lb + (i * 4 + ks * 2 + 1) * 1024);
                    bf[i] = __builtin_shufflevector(b0, b1, 0, 1, 2, 3, 4, 5, 6, 7);
                }
                if (ks == 1 && ufirst) lds_inc(lds0 + S_CNT);
#pragma unroll
                for (int a = 0; a < ((ABL & 8) ? 0 : 2); ++a) {
                    const v8i xa = __builtin_shufflevector(xc[a * 4 + ks * 2], xc[a * 4 + ks * 2 + 1], 0, 1, 2, 3, 4, 5, 6, 7);
#pragma unroll
                    for (int b = 0; b < 2; ++b)
                        acc[a][b] = __builtin_amdgcn_mfma_scale_f32_32x32x64_f8f6f4(xa, bf[b], acc[a][b], 0, 1, 0, 127, 0, 127);
                }
                asm volatile("" : "+v"(acc[0][0]), "+v"(acc[0][1]), "+v"(acc[1][0]), "+v"(acc[1][1]));
            }
        }
        TF(4);
        if (!ufirst && q < 31u) {
            if (!early) {
                cn = wait_pub(q + 1u, C_ITEM);
                load_x(xn, cn);
            }
            if (!(cn & C_RDY)) cn = wait_pub(q + 1u, C_RDY);
            TF(5);
            unpack(cn, q + 1u);
            lds_inc(lds0 + S_CNT);
        }
        TF(6);
        c = cn;
    };
    {
        v4i xa[8], xb[8];
        unsigned c = wait_pub(0u, C_RDY);
        load_x(xa, c);
        lds_inc(lds0 + S_CNT);
#pragma unroll 1
        for (unsigned q = 0; q < 32u; q += 2u) {
            step(xa, xb, q, c);
            step(xb, xa, q + 1u, c);
        }
    }
    const bool dead = spin >= SPIN_MAX;
    TS(4);

    {
        const float scale = dead ? __builtin_nanf("") : sx[0] * sw[0];
        float* op = out + (size_t)(wave * 64 + 4 * h) * GN + strip * BN + r32;
#pragma unroll
        for (int a = 0; a < 2; ++a)
#pragma unroll
            for (int i = 0; i < 16; ++i)
#pragma unroll
                for (int b = 0; b < 2; ++b)
                    __hip_atomic_store(op + (size_t)(a * 32 + (i & 3) + 8 * (i >> 2)) * GN + b * 32, acc[a][b][i] * scale, __ATOMIC_RELAXED, __HIP_MEMORY_SCOPE_SYSTEM);
    }
    if (tid == 63)
        __hip_atomic_fetch_add(ctrl_word(ws_me, L_DONE), 1u, __ATOMIC_RELAXED, __HIP_MEMORY_SCOPE_AGENT);
    TS(5);
}

}

extern "C" void kernel_launch(int rank, int nranks, ncclComm_t comm,
                              hipStream_t stream,
                              void* const* d_in, const int* in_sizes, int n_in,
                              void* d_out, int out_size,
                              const int* in_dtypes, int out_dtype,
                              void* const* d_out_all,
                              void* const* d_ws_all, size_t ws_bytes) {
    (void)comm; (void)in_dtypes; (void)out_dtype; (void)d_out_all;
    if (nranks != NR || (unsigned)rank >= (unsigned)NR || n_in < 4 || ws_bytes < WS_NEED ||
        in_sizes[0] != GM * KS || in_sizes[1] != KS * GN || out_size != MS * GN) {
        fprintf(stderr, "kernel_launch: unexpected problem geometry\n");
        return;
    }
    hipLaunchKernelGGL(fused_gemm_kernel, dim3(NWG), dim3(NTHREADS), 0, stream,
                       (const unsigned char*)d_in[0], (const unsigned char*)d_in[1],
                       (const float*)d_in[2], (const float*)d_in[3], (float*)d_out, d_ws_all, rank);
    HIP_CHECK(hipGetLastError());
}
```
